```python
import numpy as np

kernel_name = "gpu_gemm_rs_m4096_k4096_n4096_bf16_q8"


ABSMAX_RTOL = 0.02

COMPUTE_DTYPE = "bfloat16"
EPILOGUE = "quant_int8"
IN_QUANT = None
OUT_QUANT = "int8"
IN_DTYPES = {"x": "bfloat16", "w_mat": "bfloat16"}
WORLD_SIZE = 8

def _to_bf16(x):
    u = np.ascontiguousarray(x, dtype=np.float32).view(np.uint32)
    return ((u + 0x7FFF + ((u >> 16) & 1)) & 0xFFFF0000).view(np.float32)

def setup_inputs():

    x = _to_bf16(np.random.randn(4096, 4096).astype(np.float32))
    w_mat = _to_bf16((np.random.randn(4096, 4096) / np.sqrt(4096)).astype(np.float32))
    return {"x": x, "w_mat": w_mat}

def _quant_dequant(y, amax):
    scale = amax / 127.0
    q = np.clip(np.round(y / scale), -127.0, 127.0)
    return (q * scale).astype(np.float32)
def _epilogue(y):
    return _quant_dequant(y, np.abs(y).max())

def reference(x, w_mat):
    y = (x @ w_mat).astype(np.float32)
    return _epilogue(y)
```

```cpp
#include <hip/hip_runtime.h>
#include <rccl/rccl.h>
#include <cstdio>
#include <cstdint>
#include <cstring>

#define HIP_CHECK(expr) do { hipError_t _e = (expr); if (_e != hipSuccess) { \
    fprintf(stderr, "HIP error %s at %s:%d\n", hipGetErrorString(_e), __FILE__, __LINE__); \
    return; } } while (0)

namespace {

typedef short s16x4 __attribute__((ext_vector_type(4)));
typedef short s16x8 __attribute__((ext_vector_type(8)));
typedef __bf16 bf16x8 __attribute__((ext_vector_type(8)));
typedef __bf16 bf16x2 __attribute__((ext_vector_type(2)));
typedef float f32x4 __attribute__((ext_vector_type(4)));
typedef float f32x2 __attribute__((ext_vector_type(2)));
typedef unsigned u32x4 __attribute__((ext_vector_type(4)));
typedef __attribute__((address_space(1))) unsigned gu32;
typedef __attribute__((address_space(1))) unsigned long long gu64;
typedef __attribute__((address_space(3))) char lchar;
typedef unsigned long long u64;

constexpr int NR = 8;
constexpr int NWG = 256, NT = 512;
constexpr unsigned ITEM = 65536;
constexpr unsigned PTILE = 32768;
constexpr unsigned STAGE = 32768;
constexpr int NST = 4;

constexpr size_t WS_CTR = 0;
constexpr size_t WS_TMO = 32 << 10;
constexpr size_t WS_FIN = 64 << 10;
constexpr size_t WS_FP = 96 << 10;
constexpr size_t WS_G = 128 << 10;
constexpr size_t WS_STAMP = 256 << 10;
constexpr size_t WS_XST = 1 << 20;
constexpr size_t WS_WST = 7 << 20;
constexpr size_t WS_P = 13 << 20;
constexpr size_t WS_END = 19 << 20;

constexpr u64 DEADLINE = 50000000ull;
constexpr int AUX_SYS = 17;

struct Job {
  unsigned char item;
  unsigned char xs;
  unsigned char n;
  unsigned char first;
  unsigned short turn;
  unsigned short pturn;
};

struct Params {
  const char* x;
  const char* w;
  signed char* out;
  char* ws[NR];
  int rank;
  int lead;
  Job job[NR * 32];
};

__device__ __forceinline__ u64 now() { return __builtin_amdgcn_s_memrealtime(); }
__device__ __forceinline__ void wait_until(u64 t) { while (now() < t) __builtin_amdgcn_s_sleep(1); }
#define VM0() asm volatile("s_waitcnt vmcnt(0)" ::: "memory")
#define BAR() do { asm volatile("s_waitcnt lgkmcnt(0)" ::: "memory"); __builtin_amdgcn_s_barrier(); asm volatile("" ::: "memory"); } while (0)

#define STAMP(i) do {} while (0)

__device__ __forceinline__ unsigned rowbase(unsigned q, unsigned xs) {
  if (xs < 4) return 512u * (q ^ 1u) + 128u * xs;
  if (xs < 8) return 512u * q + 128u * (xs - 4u);
  const unsigned e = xs - 8u;
  return 512u * (q ^ (4u + (e >> 1))) + 256u * ((q >> 1) & 1u) + 128u * (e & 1u);
}

__device__ __forceinline__ void stage_load(const char* a, const char* b, unsigned va0, unsigned va1, unsigned vb0,
                                           unsigned vb1, unsigned lds) {
  unsigned keep;
  asm volatile(
      "s_mov_b32 %0, m0\n\t"
      "s_mov_b32 m0, %7\n\t"
      "s_nop 2\n\t"
      "global_load_lds_dwordx4 %1, %5\n\t"
      "s_add_u32 m0, m0, 0x2000\n\t"
      "s_nop 0\n\t"
      "global_load_lds_dwordx4 %2, %5\n\t"
      "s_add_u32 m0, m0, 0x2000\n\t"
      "s_nop 0\n\t"
      "global_load_lds_dwordx4 %3, %6\n\t"
      "s_add_u32 m0, m0, 0x2000\n\t"
      "s_nop 0\n\t"
      "global_load_lds_dwordx4 %4, %6\n\t"
      "s_mov_b32 m0, %0"
      : "=&s"(keep)
      : "v"(va0), "v"(va1), "v"(vb0), "v"(vb1), "s"(a), "s"(b), "s"(lds)
      : "memory", "scc");
}

struct Frag { unsigned x[2]; unsigned w[4]; };

__device__ __forceinline__ void gemm(f32x4 (&acc)[2][4], lchar* sm, unsigned ldsw, const Frag& fr, const char* a,
                                     const char* b, unsigned sa, unsigned sb, unsigned va0, unsigned va1, unsigned vb0,
                                     unsigned vb1, int nsteps, int skip) {
  auto issue = [&](int s) {
    const unsigned ph = (unsigned)(s + (s >= skip ? 4 : 0));
    stage_load(a + (size_t)ph * sa, b + (size_t)ph * sb, va0, va1, vb0, vb1, ldsw + (unsigned)(s & (NST - 1)) * STAGE);
  };
  BAR();
  issue(0); issue(1); issue(2);
  for (int s = 0; s < nsteps; ++s) {
    if (s + 2 < nsteps) asm volatile("s_waitcnt vmcnt(8)" ::: "memory");
    else if (s + 1 < nsteps) asm volatile("s_waitcnt vmcnt(4)" ::: "memory");
    else asm volatile("s_waitcnt vmcnt(0)" ::: "memory");
    BAR();
    if (s + 3 < nsteps) issue(s + 3);
    lchar* st = sm + (unsigned)(s & (NST - 1)) * STAGE;
#pragma unroll
    for (int kk = 0; kk < 2; ++kk) {
      s16x8 xf[2], wf[4];
#pragma unroll
      for (int mb = 0; mb < 2; ++mb)
        xf[mb] = *(__attribute__((address_space(3))) s16x8*)(st + fr.x[kk] + mb * 2048);
#pragma unroll
      for (int nb = 0; nb < 4; ++nb) {
        s16x4 lo = __builtin_amdgcn_ds_read_tr16_b64_v4i16((__attribute__((address_space(3))) s16x4*)(st + fr.w[nb] + kk * 8192));
        s16x4 hi = __builtin_amdgcn_ds_read_tr16_b64_v4i16((__attribute__((address_space(3))) s16x4*)(st + fr.w[nb] + kk * 8192 + 1024));
        wf[nb] = __builtin_shufflevector(lo, hi, 0, 1, 2, 3, 4, 5, 6, 7);
      }
#pragma unroll
      for (int mb = 0; mb < 2; ++mb)
#pragma unroll
        for (int nb = 0; nb < 4; ++nb)
          acc[mb][nb] = __builtin_amdgcn_mfma_f32_16x16x32_bf16(__builtin_bit_cast(bf16x8, wf[nb]),
                                                                __builtin_bit_cast(bf16x8, xf[mb]), acc[mb][nb], 0, 0, 0);
    }
  }
}

__device__ __forceinline__ void wait_flags(const char* f, unsigned call, gu32* tmo) {
  if (threadIdx.x < 64) {
    const u64 t0 = now();
    for (;;) {
      unsigned v = call;
      if (f) v = __hip_atomic_load((gu32*)f, __ATOMIC_RELAXED, __HIP_MEMORY_SCOPE_SYSTEM);
      if (__all(v >= call)) break;
      if (__hip_atomic_load(tmo, __ATOMIC_RELAXED, __HIP_MEMORY_SCOPE_AGENT) == call || now() - t0 > DEADLINE) {
        __hip_atomic_store(tmo, call, __ATOMIC_RELAXED, __HIP_MEMORY_SCOPE_AGENT);
        break;
      }
      __builtin_amdgcn_s_sleep(2);
    }
    __builtin_amdgcn_fence(__ATOMIC_ACQUIRE, "");
    VM0();
  }
  BAR();
}

__device__ __forceinline__ unsigned pack_bf16(float a, float b) {
  f32x2 v = {a, b};
  return __builtin_bit_cast(unsigned, __builtin_convertvector(v, bf16x2));
}

__device__ __forceinline__ void add_partial(f32x4 (&acc)[2][4], const u32x4 (&v)[4]) {
#pragma unroll
  for (int i = 0; i < 4; ++i) {
#pragma unroll
    for (int h = 0; h < 2; ++h) {
      f32x4& c = acc[i >> 1][(i & 1) * 2 + h];
      const unsigned u0 = v[i][2 * h], u1 = v[i][2 * h + 1];
      c[0] += __builtin_bit_cast(float, u0 << 16);
      c[1] += __builtin_bit_cast(float, u0 & 0xffff0000u);
      c[2] += __builtin_bit_cast(float, u1 << 16);
      c[3] += __builtin_bit_cast(float, u1 & 0xffff0000u);
    }
  }
}

__device__ __forceinline__ float wave_max(float m) {
#pragma unroll
  for (int o = 32; o > 0; o >>= 1) m = fmaxf(m, __shfl_xor(m, o));
  return m;
}

__global__ void __launch_bounds__(NT, 2) fused_gemm_rs_q8(const Params p) {
  __shared__ __attribute__((aligned(1024))) char smem[NST * STAGE];
  lchar* sm = (lchar*)smem;
  const u64 tstart = now();
  const unsigned t = threadIdx.x, lane = t & 63;
  const unsigned wv = __builtin_amdgcn_readfirstlane(t >> 6);
  const unsigned r = (unsigned)p.rank & 7u, d = blockIdx.x & 7u, l = blockIdx.x >> 3;
  const unsigned q = r ^ d;
  char* const me = p.ws[r];
  char* const peer = p.ws[d == 0 ? (r ^ 1u) : q];
  const Job job = p.job[d * 32 + l];
  gu32* const ctr = (gu32*)(me + WS_CTR + blockIdx.x * 128);
  gu32* const tmo = (gu32*)(me + WS_TMO);
  const unsigned call = __builtin_amdgcn_readfirstlane(__hip_atomic_load(ctr, __ATOMIC_RELAXED, __HIP_MEMORY_SCOPE_AGENT)) + 1u;
  STAMP(0);

  const unsigned nx0 = (t >> 3) * 1024u + (((t & 7u) ^ ((t >> 4) & 7u)) << 4), nx1 = nx0 + 64u * 1024u;
  const unsigned kr = t >> 4;
  const unsigned nw0 = kr * 8192u + (((t & 15u) ^ ((kr & 3u) | (kr & 8u))) << 4), nw1 = nw0 + 32u * 8192u;
  const unsigned ln0 = t * 16u, ln1 = ln0 + 8192u;
  const unsigned ldsw = (unsigned)(size_t)sm + wv * 1024u;
  Frag fr;
  {
    const unsigned wm = wv >> 1, wn = wv & 1u, g = lane >> 4, m = lane & 15u;
    fr.x[0] = (32u * wm + m) * 128u + ((g ^ (m >> 1)) << 4);
    fr.x[1] = fr.x[0] ^ 64u;
    const unsigned qq = (lane >> 2) & 3u, pp = lane & 3u;
    const unsigned base = 16384u + (8u * g + qq) * 256u + (((8u * wn + 2u * pp) ^ (qq | ((g & 1u) << 3))) << 4) + ((pp & 1u) << 3);
#pragma unroll
    for (unsigned nb = 0; nb < 4; ++nb) fr.w[nb] = base ^ (nb << 3);
  }

  auto push_item = [&]() {
    const unsigned it = job.item;
    const bool isx = it < 16;
    const unsigned cq = q & 1u, jq = q >> 2;
    const char* src = isx ? p.x + (size_t)rowbase(q, it) * 1024u + cq * 512u
                          : p.w + (size_t)cq * 256u * 8192u + (jq * 2048u + (it - 16u) * 128u) * 2u;
    const unsigned o0 = isx ? nx0 : nw0, o1 = isx ? nx1 : nw1, step = isx ? 128u : 64u * 8192u;
    char* dst = peer + (isx ? WS_XST : WS_WST) + (size_t)((it & 15u) * 6u + (d - 2u)) * ITEM;
    const __amdgpu_buffer_rsrc_t rs = __builtin_amdgcn_make_buffer_rsrc(dst, 0, ITEM, 0x00020000);
    if (job.turn > (unsigned)p.lead) wait_until(tstart + job.turn - (unsigned)p.lead);
    u32x4 v[8];
#pragma unroll
    for (int ks = 0; ks < 4; ++ks) {
      v[2 * ks] = *(const u32x4*)(src + (size_t)ks * step + o0);
      v[2 * ks + 1] = *(const u32x4*)(src + (size_t)ks * step + o1);
    }
    wait_until(tstart + job.turn);
#pragma unroll
    for (int i = 0; i < 8; ++i) __builtin_amdgcn_raw_buffer_store_b128(v[i], rs, ln0 + i * 8192u, 0, AUX_SYS);
    VM0();
    BAR();
    if (t == 0) __hip_atomic_store((gu32*)(peer + WS_FIN + ((d - 2u) * 32u + it) * 128u), call, __ATOMIC_RELAXED, __HIP_MEMORY_SCOPE_SYSTEM);
  };

  const bool has_item = d >= 2 && job.item != 255;
  if (has_item && job.first) push_item();
  STAMP(1);

  f32x4 acc[2][4], acc1[2][4];
#pragma unroll
  for (int i = 0; i < 2; ++i)
#pragma unroll
    for (int k = 0; k < 4; ++k) { acc[i][k] = f32x4{0.f, 0.f, 0.f, 0.f}; acc1[i][k] = acc[i][k]; }

  const unsigned j = r >> 2, xs = job.xs, n = job.n;
  if (d < 2)
    gemm(acc1, sm, ldsw, fr, p.x + (size_t)(512u * r + 128u * xs) * 1024u, p.w + ((j ^ 1u) * 2048u + n * 128u) * 2u, 128u,
         64u * 8192u, nx0, nx1, nw0, nw1, 8, 1000);
  gemm(acc, sm, ldsw, fr, p.x + (size_t)rowbase(r, xs) * 1024u, p.w + (j * 2048u + n * 128u) * 2u, 128u, 64u * 8192u, nx0,
       nx1, nw0, nw1, 8, 1000);
  STAMP(2);
  if (has_item && !job.first) push_item();
  STAMP(3);

  {
    const unsigned s = lane % 6u, isw = lane / 6u;
    const bool act = lane < 12 && !(d >= 4 && s == d - 2u);
    wait_flags(act ? me + WS_FIN + (s * 32u + (isw ? 16u + n : xs)) * 128u : nullptr, call, tmo);
    STAMP(4);
    gemm(acc, sm, ldsw, fr, me + WS_XST + (size_t)xs * 6u * ITEM, me + WS_WST + (size_t)n * 6u * ITEM, 16384u, 16384u, ln0,
         ln1, ln0, ln1, d >= 4 ? 20 : 24, d >= 4 ? (int)(d - 2u) * 4 : 1000);
    STAMP(5);
  }

  if (d < 2 || d >= 4) {
    const unsigned a = d < 2 ? xs : 2u * ((r >> 1) & 1u) + (xs & 1u);
    const unsigned slot = ((d < 2 ? 0u : 1u + (r & 1u)) * 4u + a) * 16u + n;
    const __amdgpu_buffer_rsrc_t rs = __builtin_amdgcn_make_buffer_rsrc(peer + WS_P + (size_t)slot * PTILE, 0, PTILE, 0x00020000);
    wait_until(tstart + job.pturn);
#pragma unroll
    for (int i = 0; i < 4; ++i) {
      const f32x4 c0 = acc[i >> 1][(i & 1) * 2], c1 = acc[i >> 1][(i & 1) * 2 + 1];
      const u32x4 v = {pack_bf16(c0[0], c0[1]), pack_bf16(c0[2], c0[3]), pack_bf16(c1[0], c1[1]), pack_bf16(c1[2], c1[3])};
      __builtin_amdgcn_raw_buffer_store_b128(v, rs, ln0 + i * 8192u, 0, AUX_SYS);
    }
    VM0();
    BAR();
    if (t == 0) __hip_atomic_store((gu32*)(peer + WS_FP + slot * 128u), call, __ATOMIC_RELAXED, __HIP_MEMORY_SCOPE_SYSTEM);
    STAMP(6);
  }
  if (d < 4) {
    const unsigned a = xs & 3u;
    const unsigned slot = ((d < 2 ? 4u : 0u) + a) * 16u + n;
    if (d < 2) {
#pragma unroll
      for (int i = 0; i < 2; ++i)
#pragma unroll
        for (int k = 0; k < 4; ++k) acc[i][k] = acc1[i][k];
    }
    wait_flags(lane == 0 || (lane == 1 && d < 2) ? me + WS_FP + (slot + lane * 64u) * 128u : nullptr, call, tmo);
    STAMP(7);
    const __amdgpu_buffer_rsrc_t rs = __builtin_amdgcn_make_buffer_rsrc(me + WS_P + (size_t)slot * PTILE, 0, 65u * PTILE, 0x00020000);
    u32x4 v[4], v2[4];
#pragma unroll
    for (int i = 0; i < 4; ++i) v[i] = __builtin_amdgcn_raw_buffer_load_b128(rs, ln0 + i * 8192u, 0, AUX_SYS);
    if (d < 2) {
#pragma unroll
      for (int i = 0; i < 4; ++i) v2[i] = __builtin_amdgcn_raw_buffer_load_b128(rs, ln0 + i * 8192u + 64u * PTILE, 0, AUX_SYS);
      add_partial(acc, v2);
    }
    add_partial(acc, v);

    float m = 0.f;
#pragma unroll
    for (int i = 0; i < 2; ++i)
#pragma unroll
      for (int k = 0; k < 4; ++k)
#pragma unroll
        for (int e = 0; e < 4; ++e) m = fmaxf(m, fabsf(acc[i][k][e]));
    m = wave_max(m);
    __attribute__((address_space(3))) float* red = (__attribute__((address_space(3))) float*)sm;
    if (lane == 0) red[wv] = m;
    BAR();
    if (t < NR) {
      float mm = red[0];
#pragma unroll
      for (int i = 1; i < 8; ++i) mm = fmaxf(mm, red[i]);
      const u64 g = ((u64)call << 32) | __builtin_bit_cast(unsigned, mm);
      __hip_atomic_store((gu64*)(p.ws[t] + WS_G + (r * 128u + d * 32u + l) * 8u), g, __ATOMIC_RELAXED, __HIP_MEMORY_SCOPE_SYSTEM);
    }
    STAMP(8);
    const __amdgpu_buffer_rsrc_t rg = __builtin_amdgcn_make_buffer_rsrc(me + WS_G, 0, 8192, 0x00020000);
    u32x4 g;
    {
      const u64 t0 = now();
      for (;;) {
        g = __builtin_amdgcn_raw_buffer_load_b128(rg, ln0, 0, AUX_SYS);
        if (__all(g[1] == call && g[3] == call)) break;
        if (__hip_atomic_load(tmo, __ATOMIC_RELAXED, __HIP_MEMORY_SCOPE_AGENT) == call || now() - t0 > DEADLINE) {
          __hip_atomic_store(tmo, call, __ATOMIC_RELAXED, __HIP_MEMORY_SCOPE_AGENT);
          break;
        }
        __builtin_amdgcn_s_sleep(2);
        asm volatile("" ::: "memory");
      }
    }
    const unsigned g0 = g[0], g2 = g[2];
    m = wave_max(fmaxf(__uint_as_float(g0), __uint_as_float(g2)));
    if (lane == 0) red[16 + wv] = m;
    VM0();
    BAR();
    float amax = red[16];
#pragma unroll
    for (int i = 1; i < 8; ++i) amax = fmaxf(amax, red[16 + i]);
    const bool bad = __hip_atomic_load(tmo, __ATOMIC_RELAXED, __HIP_MEMORY_SCOPE_AGENT) == call;
    STAMP(9);
    const float scale = amax > 0.f ? amax / 127.0f : 1.0f;
    const unsigned jc = d < 2 ? (j ^ 1u) : j;
    const unsigned pp = lane >> 4;
#pragma unroll
    for (int mb = 0; mb < 2; ++mb) {
      unsigned dw[4];
#pragma unroll
      for (int nb = 0; nb < 4; ++nb) {
        unsigned u = 0;
#pragma unroll
        for (int e = 0; e < 4; ++e) {
          float f = rintf(acc[mb][nb][e] / scale);
          f = fminf(fmaxf(f, -127.f), 127.f);
          u |= ((unsigned)(int)f & 0xffu) << (8 * e);
        }
        dw[nb] = bad ? 0x80808080u : u;
      }
      const bool odd = pp & 1u;
      const u32x4 o = {odd ? dw[1] : dw[0], odd ? dw[0] : dw[1], odd ? dw[3] : dw[2], odd ? dw[2] : dw[3]};
      const unsigned row = 128u * a + 32u * (wv >> 1) + 16u * mb + (lane & 15u);
      const unsigned col = jc * 2048u + n * 128u + 64u * (wv & 1u) + 16u * pp;
      *(u32x4*)(p.out + (size_t)row * 4096u + col) = o;
    }
    STAMP(10);
  }
  if (t == 0) __hip_atomic_store(ctr, call, __ATOMIC_RELAXED, __HIP_MEMORY_SCOPE_AGENT);
}

#define TAU 110
#define HEAD 1
#define FIRST_T 450
#define LEAD 150

void make_jobs(Job* job) {
  static const unsigned char W = 16;
  const unsigned char ord[32] = {0, W + 0, 1, W + 1, 2, W + 2, 3, W + 3, W + 4, W + 5, 8, W + 6, 10, W + 7, 12, W + 8,
                                 14, W + 9, 9, W + 10, 11, W + 11, 13, W + 12, 15, W + 13, W + 14, W + 15, 4, 5, 6, 7};
  int pos[32];
  for (int i = 0; i < 32; ++i) pos[ord[i]] = i;
  for (int d = 0; d < 8; ++d) {
    Job* jb = job + d * 32;
    if (d < 4) {
      for (int l = 0; l < 32; ++l) {
        jb[l].xs = (unsigned char)((d < 2 ? 0 : 4) + (l & 3));
        jb[l].n = (unsigned char)(2 * (l >> 2) + (d & 1));
        jb[l].item = d < 2 ? 255 : ord[l];
        const int tn = l > HEAD ? (l - HEAD) * TAU : 0;
        jb[l].turn = (unsigned short)tn;
        jb[l].first = tn < FIRST_T;
        jb[l].pturn = 0;
      }
    } else {
      int key[32], idx[32];
      for (int m = 0; m < 32; ++m) {
        const int xs = 8 + 2 * (d - 4) + (m >> 4), n = m & 15;
        const int a = pos[xs], b = pos[16 + n];
        key[m] = (a > b ? a : b) * 64 + m;
        idx[m] = m;
      }
      for (int a = 0; a < 32; ++a)
        for (int b = a + 1; b < 32; ++b)
          if (key[idx[b]] < key[idx[a]]) { const int tmp = idx[a]; idx[a] = idx[b]; idx[b] = tmp; }
      unsigned char lst[32];
      int cnt = 0;
      for (int i = 0; i < 32; ++i)
        if (!(ord[i] >= 8 && ord[i] < 16 && (ord[i] - 8) / 2 == d - 4)) lst[cnt++] = ord[i];
      for (int l = 0; l < 32; ++l) {
        const int m = idx[l];
        jb[l].xs = (unsigned char)(8 + 2 * (d - 4) + (m >> 4));
        jb[l].n = (unsigned char)(m & 15);
        const int k = l - 2;
        jb[l].item = k >= 0 ? lst[k] : 255;
        const int tn = k > HEAD ? (k - HEAD) * TAU : 0;
        jb[l].turn = (unsigned short)tn;
        jb[l].first = tn < FIRST_T;
        jb[l].pturn = (unsigned short)((30 - HEAD) * TAU + l * TAU / 2);
      }
    }
  }
}

}

extern "C" void kernel_launch(int rank, int nranks, ncclComm_t comm, hipStream_t stream, void* const* d_in,
                              const int* in_sizes, int n_in, void* d_out, int out_size, const int* in_dtypes,
                              int out_dtype, void* const* d_out_all, void* const* d_ws_all, size_t ws_bytes) {
  if (nranks != NR || (unsigned)rank >= (unsigned)NR || n_in < 2 || ws_bytes < WS_END || out_size != 512 * 4096 ||
      in_sizes[0] != 4096 * 512 || in_sizes[1] != 512 * 4096) {
    fprintf(stderr, "kernel_launch: unexpected problem shape\n");
    return;
  }
  Params p;
  memset(&p, 0, sizeof p);
  p.x = (const char*)d_in[0];
  p.w = (const char*)d_in[1];
  p.out = (signed char*)d_out;
  for (int i = 0; i < NR; ++i) p.ws[i] = (char*)d_ws_all[i];
  p.rank = rank;
  p.lead = LEAD;
  make_jobs(p.job);
  hipLaunchKernelGGL(fused_gemm_rs_q8, dim3(NWG), dim3(NT), 0, stream, p);
  HIP_CHECK(hipGetLastError());
}
```

```cpp
#include <hip/hip_runtime.h>
#include <rccl/rccl.h>
#include <cstdio>
#include <cstdint>
#include <cstring>

#define HIP_CHECK(expr) do { hipError_t _e = (expr); if (_e != hipSuccess) { \
    fprintf(stderr, "HIP error %s at %s:%d\n", hipGetErrorString(_e), __FILE__, __LINE__); \
    return; } } while (0)

namespace {

typedef short s16x4 __attribute__((ext_vector_type(4)));
typedef short s16x8 __attribute__((ext_vector_type(8)));
typedef __bf16 bf16x8 __attribute__((ext_vector_type(8)));
typedef __bf16 bf16x2 __attribute__((ext_vector_type(2)));
typedef float f32x4 __attribute__((ext_vector_type(4)));
typedef float f32x2 __attribute__((ext_vector_type(2)));
typedef unsigned u32x4 __attribute__((ext_vector_type(4)));
typedef __attribute__((address_space(1))) unsigned gu32;
typedef __attribute__((address_space(1))) unsigned long long gu64;
typedef __attribute__((address_space(3))) char lchar;
typedef unsigned long long u64;

constexpr int NR = 8;
constexpr int NWG = 256, NT = 512;
constexpr unsigned ITEM = 65536;
constexpr unsigned PTILE = 32768;
constexpr unsigned STAGE = 32768;
constexpr int NST = 4;

constexpr size_t WS_CTR = 0;
constexpr size_t WS_TMO = 32 << 10;
constexpr size_t WS_TICK = 33 << 10;
constexpr size_t WS_FIN = 64 << 10;
constexpr size_t WS_FP = 96 << 10;
constexpr size_t WS_G = 128 << 10;
constexpr size_t WS_STAMP = 256 << 10;
constexpr size_t WS_XST = 1 << 20;
constexpr size_t WS_WST = 7 << 20;
constexpr size_t WS_P = 13 << 20;
constexpr size_t WS_END = 19 << 20;

constexpr u64 DEADLINE = 50000000ull;
constexpr int AUX_SYS = 17;

#define TAU 114
#define T0 80
#define STAG (TAU / 8)
#define ROT 1
#define PTAU 57
#define GATE (T0 + 30 * TAU)
#define OUT_AUX 17
#define PSLEEP 4
#define GSLEEP 2
#define SWEEP_T 5000
#define PRED 1
#define DELAY 300
#define EARLY_N 4

constexpr unsigned char W_ = 16;
#define ORDV 1
constexpr unsigned char ORD[32] = {0, W_ + 0, 1, W_ + 1, 2, 3, W_ + 2, W_ + 3, 8, W_ + 4, 10, 12, W_ + 5, 14, W_ + 6, 9,
                                   W_ + 7, 11, W_ + 8, 13, W_ + 9, 15, W_ + 10, W_ + 11, W_ + 12, 4, W_ + 13, 5, W_ + 14, 6, W_ + 15, 7};
constexpr u64 ord_pack(int o) {
  u64 v = 0;
  for (int i = 0; i < 8; ++i) v |= (u64)ORD[o + i] << (8 * i);
  return v;
}

struct alignas(8) Job {
  unsigned char xs;
  unsigned char n;
  unsigned char pos;
  unsigned char skip0, skip1;
  unsigned char pad0, pad1;
  unsigned char mode;
};

struct Params {
  const char* x;
  const char* w;
  signed char* out;
  char* wsx[NR];
  int rank;
  int pad;
  Job job[NR * 32];
};

__device__ __forceinline__ u64 now() { return __builtin_amdgcn_s_memrealtime(); }
__device__ __forceinline__ void wait_until(u64 t) { while (now() < t) __builtin_amdgcn_s_sleep(1); }
#define VM0() asm volatile("s_waitcnt vmcnt(0)" ::: "memory")
#define BAR() do { asm volatile("" ::: "memory"); __builtin_amdgcn_s_waitcnt(0xC07F);   __builtin_amdgcn_s_barrier(); asm volatile("" ::: "memory"); } while (0)
#define STR2(x) #x
#define STR(x) STR2(x)

#define STAMP(i) do {} while (0)
#define STAMPW(i) do {} while (0)

__device__ __forceinline__ unsigned ord_item(unsigned k) {
  const u64 a = (k & 16u) ? ((k & 8u) ? ord_pack(24) : ord_pack(16)) : ((k & 8u) ? ord_pack(8) : ord_pack(0));
  return (unsigned)(a >> (8u * (k & 7u))) & 0xffu;
}

__device__ __forceinline__ unsigned rowbase(unsigned q, unsigned xs) {
  const unsigned e = xs - 8u;
  const unsigned third = 512u * (q ^ (4u + (e >> 1))) + 256u * ((q >> 1) & 1u) + 128u * (e & 1u);
  const unsigned pair = 512u * (q ^ (xs < 4u ? 1u : 0u)) + 128u * (xs & 3u);
  return xs < 8u ? pair : third;
}

__device__ __forceinline__ void stage_load(const char* a, const char* b, const unsigned (&va)[4], const unsigned (&vb)[4],
                                           unsigned lds) {
  unsigned keep;
  asm volatile(
      "s_mov_b32 %0, m0\n\t"
      "s_mov_b32 m0, %11\n\t"
      "s_nop 2\n\t"
      "global_load_lds_dwordx4 %1, %9\n\t"
      "global_load_lds_dwordx4 %2, %9 offset:1024\n\t"
      "global_load_lds_dwordx4 %3, %9 offset:2048\n\t"
      "global_load_lds_dwordx4 %4, %9 offset:3072\n\t"
      "s_add_u32 m0, m0, 0x4000\n\t"
      "s_nop 0\n\t"
      "global_load_lds_dwordx4 %5, %10\n\t"
      "global_load_lds_dwordx4 %6, %10 offset:1024\n\t"
      "global_load_lds_dwordx4 %7, %10 offset:2048\n\t"
      "global_load_lds_dwordx4 %8, %10 offset:3072\n\t"
      "s_mov_b32 m0, %0"
      : "=&s"(keep)
      : "v"(va[0]), "v"(va[1]), "v"(va[2]), "v"(va[3]), "v"(vb[0]), "v"(vb[1]), "v"(vb[2]), "v"(vb[3]), "s"(a), "s"(b),
        "s"(lds)
      : "memory", "scc");
}

__device__ __forceinline__ unsigned nat_x(unsigned c) { return (c >> 3) * 1024u + (((c & 7u) ^ ((c >> 4) & 7u)) << 4); }
__device__ __forceinline__ unsigned nat_w(unsigned c) {
  const unsigned kr = c >> 4;
  return kr * 8192u + (((c & 15u) ^ ((kr & 3u) | (kr & 8u))) << 4);
}

struct Half { s16x8 x[4], w[4]; };

__device__ __forceinline__ void read_half(Half& f, lchar* st, unsigned fx, const unsigned (&fw)[4], int kk) {
#pragma unroll
  for (int i = 0; i < 4; ++i) {
    f.x[i] = *(__attribute__((address_space(3))) s16x8*)(st + fx + i * 2048);
    s16x4 lo = __builtin_amdgcn_ds_read_tr16_b64_v4i16((__attribute__((address_space(3))) s16x4*)(st + fw[i] + kk * 8192));
    s16x4 hi = __builtin_amdgcn_ds_read_tr16_b64_v4i16((__attribute__((address_space(3))) s16x4*)(st + fw[i] + kk * 8192 + 1024));
    f.w[i] = __builtin_shufflevector(lo, hi, 0, 1, 2, 3, 4, 5, 6, 7);
  }
}
__device__ __forceinline__ void mma(f32x4 (&acc)[4][4], const Half& f) {
#pragma unroll
  for (int mb = 0; mb < 4; ++mb)
#pragma unroll
    for (int nb = 0; nb < 4; ++nb)
      acc[mb][nb] = __builtin_amdgcn_mfma_f32_16x16x32_bf16(__builtin_bit_cast(bf16x8, f.w[nb]),
                                                            __builtin_bit_cast(bf16x8, f.x[mb]), acc[mb][nb], 0, 0, 0);
}
__device__ __forceinline__ void interleave() {
#pragma unroll
  for (int i = 0; i < 12; ++i) {
    __builtin_amdgcn_sched_group_barrier(0x8, 1, 0);
    __builtin_amdgcn_sched_group_barrier(0x100, 1, 0);
  }
  __builtin_amdgcn_sched_group_barrier(0x8, 4, 0);
}

__device__ __forceinline__ void gemm_load(unsigned lds, unsigned lw, unsigned lane, const char* an, const char* bn,
                                          const char* as, const char* bs, int s0, int s1, int skip) {
  unsigned nx[4], nw[4], ln[4];
#pragma unroll
  for (unsigned i = 0; i < 4; ++i) {
    const unsigned c = (lw * 4u + i) * 64u + lane;
    nx[i] = nat_x(c) - i * 1024u;
    nw[i] = nat_w(c) - i * 1024u;
    ln[i] = lw * 4096u + lane * 16u;
  }
  auto issue = [&](int s) {
    const unsigned slot = lds + lw * 4096u + (unsigned)(s & (NST - 1)) * STAGE;
    if (s < 8) {
      stage_load(an + (unsigned)s * 128u, bn + (unsigned)s * (64u * 8192u), nx, nw, slot);
    } else {
      const unsigned o = (unsigned)(s - 8 + (s >= skip ? 4 : 0)) * 16384u;
      stage_load(as + o, bs + o, ln, ln, slot);
    }
  };
  BAR();
  for (int s = s0; s < s0 + 4; ++s) issue(s);
  asm volatile("s_waitcnt vmcnt(24)" ::: "memory");
  BAR();
  for (int s = s0; s + 1 < s1; ++s) {
    const int left = s1 - 2 - s;
    if (left >= 2) asm volatile("s_waitcnt vmcnt(16)" ::: "memory");
    else if (left == 1) asm volatile("s_waitcnt vmcnt(8)" ::: "memory");
    else asm volatile("s_waitcnt vmcnt(0)" ::: "memory");
    BAR();
    if (s + 4 < s1) issue(s + 4);
  }
}
__device__ __forceinline__ void gemm_math(f32x4 (&acc)[4][4], lchar* sm, unsigned wv, unsigned lane, int s0, int s1) {
  const unsigned wm = wv >> 1, wn = wv & 1u, g = lane >> 4, m = lane & 15u;
  const unsigned fx0 = (64u * wm + m) * 128u + ((g ^ (m >> 1)) << 4), fx1 = fx0 ^ 64u;
  const unsigned qq = (lane >> 2) & 3u, pp = lane & 3u;
  const unsigned base = 16384u + (8u * g + qq) * 256u + (((8u * wn + 2u * pp) ^ (qq | ((g & 1u) << 3))) << 4) + ((pp & 1u) << 3);
  const unsigned fw[4] = {base, base ^ 8u, base ^ 16u, base ^ 24u};
  Half fa, fb;
  BAR();
  BAR();
  read_half(fa, sm + (unsigned)(s0 & (NST - 1)) * STAGE, fx0, fw, 0);
  for (int s = s0; s + 1 < s1; ++s) {
    read_half(fb, sm + (unsigned)(s & (NST - 1)) * STAGE, fx1, fw, 1);
    mma(acc, fa);
    interleave();
    BAR();
    read_half(fa, sm + (unsigned)((s + 1) & (NST - 1)) * STAGE, fx0, fw, 0);
    mma(acc, fb);
    interleave();
  }
  read_half(fb, sm + (unsigned)((s1 - 1) & (NST - 1)) * STAGE, fx1, fw, 1);
  mma(acc, fa);
  interleave();
  mma(acc, fb);
}

__device__ __forceinline__ bool give_up(gu32* tmo, unsigned call, u64 t0) {
  if (__hip_atomic_load(tmo, __ATOMIC_RELAXED, __HIP_MEMORY_SCOPE_AGENT) == call || now() - t0 > DEADLINE) {
    __hip_atomic_store(tmo, call, __ATOMIC_RELAXED, __HIP_MEMORY_SCOPE_AGENT);
    return true;
  }
  return false;
}

__device__ __forceinline__ void poll_flags(const char* f, const char* g, unsigned call, gu32* tmo) {
  const u64 t0 = now();
  for (unsigned it = 1;; ++it) {
    unsigned vf = 0, vg = 0;
    if (f) vf = __hip_atomic_load((gu32*)f, __ATOMIC_RELAXED, __HIP_MEMORY_SCOPE_SYSTEM);
    if (g) vg = __hip_atomic_load((gu32*)g, __ATOMIC_RELAXED, __HIP_MEMORY_SCOPE_SYSTEM);
    if (vf >= call) f = nullptr;
    if (vg >= call) g = nullptr;
    if (__all(f == nullptr && g == nullptr)) break;
    if ((it & 63u) == 0 && give_up(tmo, call, t0)) break;
    __builtin_amdgcn_s_sleep(PSLEEP);
  }
}

__device__ __forceinline__ unsigned pack_bf16(float a, float b) {
  f32x2 v = {a, b};
  return __builtin_bit_cast(unsigned, __builtin_convertvector(v, bf16x2));
}

__device__ __forceinline__ void add_partial(f32x4 (&acc)[4][4], const u32x4 (&v)[8]) {
#pragma unroll
  for (int i = 0; i < 8; ++i) {
#pragma unroll
    for (int h = 0; h < 2; ++h) {
      f32x4& c = acc[i >> 1][(i & 1) * 2 + h];
      const unsigned u0 = v[i][2 * h], u1 = v[i][2 * h + 1];
      c[0] += __uint_as_float(u0 << 16);
      c[1] += __uint_as_float(u0 & 0xffff0000u);
      c[2] += __uint_as_float(u1 << 16);
      c[3] += __uint_as_float(u1 & 0xffff0000u);
    }
  }
}

__device__ __forceinline__ float wave_max(float m) {
#pragma unroll
  for (int o = 32; o > 0; o >>= 1) m = fmaxf(m, __shfl_xor(m, o));
  return m;
}

__global__ void __launch_bounds__(NT, 2) fused_gemm_rs_q8(const Params p) {
  __shared__ __attribute__((aligned(1024))) char smem[NST * STAGE];
  lchar* sm = (lchar*)smem;
  const u64 tstart = now();
  const unsigned t = threadIdx.x, lane = t & 63;
  const unsigned wv = __builtin_amdgcn_readfirstlane(t >> 6);
  const unsigned r = (unsigned)p.rank & 7u, d = blockIdx.x & 7u, l = blockIdx.x >> 3;
  const unsigned q = r ^ d;
  char* const me = p.wsx[0];
  char* const peer = p.wsx[d == 0 ? 1u : d];
  const Job job = p.job[d * 32 + l];
  gu32* const ctr = (gu32*)(me + WS_CTR + blockIdx.x * 128);
  gu32* const tmo = (gu32*)(me + WS_TMO);
  const unsigned call_v = __hip_atomic_load(ctr, __ATOMIC_RELAXED, __HIP_MEMORY_SCOPE_AGENT) + 1u;

  if (job.mode & 8u) {
    const unsigned u = job.mode & 7u;
    const unsigned xa = nat_x(lane), xb = nat_x(lane + 64u);
    const unsigned wa = nat_w(lane), wb = nat_w(lane + 128u);
    const unsigned cq = q & 1u, jq = q >> 2;
    for (unsigned k = wv; k < 32u; k += 8u) {
      const unsigned it = ord_item(k), sl = (u + ROT * k) & 7u;
      if (d >= 4u && (it >> 1) == d) continue;
      const unsigned slotk = k - (d >= 4u ? (k > job.skip0 ? 1u : 0u) + (k > job.skip1 ? 1u : 0u) : 0u);
      u32x4 v[8];
      if (it < 16u) {
        const __amdgpu_buffer_rsrc_t rs = __builtin_amdgcn_make_buffer_rsrc((void*)p.x, 0, 4u << 20, 0x00020000);
        const unsigned off = (rowbase(q, it) + (sl & 1u) * 64u) * 1024u + cq * 512u + (sl >> 1) * 128u;
#pragma unroll
        for (unsigned i = 0; i < 8; ++i) v[i] = __builtin_amdgcn_raw_buffer_load_b128(rs, (i & 1u) ? xb : xa, off + (i >> 1) * 16384u, 0);
      } else {
        const __amdgpu_buffer_rsrc_t rs = __builtin_amdgcn_make_buffer_rsrc((void*)p.w, 0, 4u << 20, 0x00020000);
        const unsigned off = (cq * 256u + sl * 32u) * 8192u + (jq * 2048u + (it - 16u) * 128u) * 2u;
#pragma unroll
        for (unsigned i = 0; i < 8; ++i) v[i] = __builtin_amdgcn_raw_buffer_load_b128(rs, (i & 2u) ? wb : wa, off + (i & 5u) * 32768u, 0);
      }
      char* dst = peer + (it < 16u ? WS_XST : WS_WST) + (size_t)(((it & 15u) * 6u + (d - 2u)) * ITEM) + sl * 8192u;
      const __amdgpu_buffer_rsrc_t rd = __builtin_amdgcn_make_buffer_rsrc(dst, 0, 8192u, 0x00020000);
      wait_until(tstart + T0 + slotk * TAU + u * STAG);
#pragma unroll
      for (unsigned i = 0; i < 8; ++i) __builtin_amdgcn_raw_buffer_store_b128(v[i], rd, lane * 16u, i * 1024u, AUX_SYS);
      VM0();
      if (lane == 0)
        __hip_atomic_store((gu32*)(peer + WS_FIN + ((d - 2u) * 32u + it) * 128u + u * 4u), call_v, __ATOMIC_RELAXED, __HIP_MEMORY_SCOPE_SYSTEM);
    }
  } else if (job.mode & 32u) {
    wait_until(tstart + DELAY);
  }
  STAMP(1);
  const unsigned call = __builtin_amdgcn_readfirstlane(call_v);
  const unsigned j = r >> 2, xs = job.xs, n = job.n;
  const unsigned plan0 = d >= 2u ? ((job.mode & 8u) ? 0xf2u : 0xf10u) : ((job.mode & 16u) ? 0xf310u : 0xf103u);
  const bool ships = d < 2u || d >= 4u;
  const int send = d >= 4u ? 28 : 32;
  __attribute__((address_space(3))) float* red = (__attribute__((address_space(3))) float*)sm;

  if (wv >= 4) {
    for (unsigned plan = plan0; (plan & 15u) != 15u; plan >>= 4) {
      const unsigned kind = plan & 15u;
      const bool staged = kind == 1u || kind == 2u;
      if (staged && wv == 4) {
        const unsigned s = lane >> 3;
        const bool act = lane < 48u && !(d >= 4u && s == d - 2u);
        if (PRED) wait_until(tstart + T0 + job.pos * TAU);
        poll_flags(act ? me + WS_FIN + (s * 32u + xs) * 128u + (lane & 7u) * 4u : nullptr,
                   act ? me + WS_FIN + (s * 32u + 16u + n) * 128u + (lane & 7u) * 4u : nullptr, call, tmo);
        STAMPW(12);
        __builtin_amdgcn_fence(__ATOMIC_ACQUIRE, "");
        VM0();
        STAMPW(13);
      }
      gemm_load((unsigned)(size_t)sm, wv - 4u, lane, p.x + (size_t)(kind == 3u ? 512u * r + 128u * xs : rowbase(r, xs)) * 1024u,
                p.w + ((kind == 3u ? j ^ 1u : j) * 2048u + n * 128u) * 2u, me + WS_XST + (size_t)(xs * 6u * ITEM),
                me + WS_WST + (size_t)(n * 6u * ITEM), kind == 1u ? 8 : 0, staged ? send : 8, d >= 4u ? (int)d * 4 : 1000);
      if (staged && ships) {
        if (t == 256u) {
          gu64* const next = (gu64*)(me + WS_TICK + (d < 2u ? 0u : d) * 128u);
          const u64 tn = now(), tg = tstart + GATE;
          const u64 old = __hip_atomic_fetch_max(next, d >= 4u && tg > tn ? tg : tn, __ATOMIC_RELAXED, __HIP_MEMORY_SCOPE_AGENT);
          const u64 mine = __hip_atomic_fetch_add(next, (u64)PTAU + (old == ~0ull), __ATOMIC_RELAXED, __HIP_MEMORY_SCOPE_AGENT);
          const u64 lim = tn + GATE + 64u * PTAU;
          wait_until(mine < lim ? mine : lim);
        }
        BAR();
        BAR();
      }
    }
    if (d < 4u) {
      if (wv == 4) {
        const unsigned slot = ((d < 2u ? 4u : 0u) + (xs & 3u)) * 16u + n;
        poll_flags(lane == 0 || (lane == 1 && d < 2u) ? me + WS_FP + (slot + lane * 64u) * 128u : nullptr, nullptr, call, tmo);
      }
      BAR();
      BAR();
      const __amdgpu_buffer_rsrc_t rg = __builtin_amdgcn_make_buffer_rsrc(me + WS_G, 0, 8192, 0x00020000);
      u32x4 g = {0u, 0u, 0u, 0u}, h = g;
      wait_until(tstart + SWEEP_T);
      const u64 t0 = now();
      for (unsigned it = 1;; ++it) {
        const bool mg = !(g[1] == call && g[3] == call), mh = !(h[1] == call && h[3] == call);
        if (!__any(mg || mh)) break;
        if (it > 1) __builtin_amdgcn_s_sleep(GSLEEP);
        if (mg) g = __builtin_amdgcn_raw_buffer_load_b128(rg, (t - 256u) * 16u, 0, AUX_SYS);
        if (mh) h = __builtin_amdgcn_raw_buffer_load_b128(rg, (t - 256u) * 16u + 4096u, 0, AUX_SYS);
        if ((it & 63u) == 0 && give_up(tmo, call, t0)) break;
        asm volatile("" ::: "memory");
      }
      const unsigned tm = __hip_atomic_load(tmo, __ATOMIC_RELAXED, __HIP_MEMORY_SCOPE_AGENT);
      const unsigned g0 = g[0], g2 = g[2], h0 = h[0], h2 = h[2];
      float m = wave_max(fmaxf(fmaxf(__uint_as_float(g0), __uint_as_float(g2)), fmaxf(__uint_as_float(h0), __uint_as_float(h2))));
      if (tm == call) m = -1.0f;
      if (lane == 0) red[16 + wv - 4u] = m;
      BAR();
    }
  } else {
    f32x4 acc[4][4], acc1[4][4];
#pragma unroll
    for (int i = 0; i < 4; ++i)
#pragma unroll
      for (int k = 0; k < 4; ++k) { acc[i][k] = f32x4{0.f, 0.f, 0.f, 0.f}; acc1[i][k] = acc[i][k]; }
    const unsigned ln0 = t * 16u;
    for (unsigned plan = plan0; (plan & 15u) != 15u; plan >>= 4) {
      const unsigned kind = plan & 15u;
      const bool staged = kind == 1u || kind == 2u;
      if (staged) STAMP(3);
      if (kind == 3u) {
#pragma unroll
        for (int i = 0; i < 4; ++i)
#pragma unroll
          for (int k = 0; k < 4; ++k) { const f32x4 tmp = acc[i][k]; acc[i][k] = acc1[i][k]; acc1[i][k] = tmp; }
      }
      gemm_math(acc, sm, wv, lane, kind == 1u ? 8 : 0, staged ? send : 8);
      if (kind == 3u) {
#pragma unroll
        for (int i = 0; i < 4; ++i)
#pragma unroll
          for (int k = 0; k < 4; ++k) { const f32x4 tmp = acc[i][k]; acc[i][k] = acc1[i][k]; acc1[i][k] = tmp; }
      }
      if (kind == 0u) STAMP(2);
      if (staged) STAMP(5);
      if (staged && ships) {
        const unsigned a = d < 2u ? xs : 2u * ((r >> 1) & 1u) + (xs & 1u);
        const unsigned slot = ((d < 2u ? 0u : 1u + (r & 1u)) * 4u + a) * 16u + n;
        const __amdgpu_buffer_rsrc_t rs = __builtin_amdgcn_make_buffer_rsrc(peer + WS_P + (size_t)slot * PTILE, 0, PTILE, 0x00020000);
        u32x4 v[8];
#pragma unroll
        for (int i = 0; i < 8; ++i) {
          const f32x4 c0 = acc[i >> 1][(i & 1) * 2], c1 = acc[i >> 1][(i & 1) * 2 + 1];
          v[i] = u32x4{pack_bf16(c0[0], c0[1]), pack_bf16(c0[2], c0[3]), pack_bf16(c1[0], c1[1]), pack_bf16(c1[2], c1[3])};
        }
        BAR();
        STAMP(4);
#pragma unroll
        for (int i = 0; i < 8; ++i) __builtin_amdgcn_raw_buffer_store_b128(v[i], rs, ln0 + i * 4096u, 0, AUX_SYS);
        VM0();
        BAR();
        if (t == 0) __hip_atomic_store((gu32*)(peer + WS_FP + slot * 128u), call, __ATOMIC_RELAXED, __HIP_MEMORY_SCOPE_SYSTEM);
        STAMP(6);
      }
    }
    if (d < 4u) {
      const unsigned a = xs & 3u;
      const unsigned slot = ((d < 2u ? 4u : 0u) + a) * 16u + n;
      if (d < 2u) {
#pragma unroll
        for (int i = 0; i < 4; ++i)
#pragma unroll
          for (int k = 0; k < 4; ++k) acc[i][k] = acc1[i][k];
      }
      char* const gdst = p.wsx[lane & 7u] + WS_G + (r * 128u + d * 32u + l) * 8u;
      STAMP(11);
      BAR();
      STAMP(7);
      const __amdgpu_buffer_rsrc_t rs = __builtin_amdgcn_make_buffer_rsrc(me + WS_P + (size_t)slot * PTILE, 0, d < 2u ? 65u * PTILE : PTILE, 0x00020000);
      u32x4 v[8], v2[8];
#pragma unroll
      for (int i = 0; i < 8; ++i) v[i] = __builtin_amdgcn_raw_buffer_load_b128(rs, ln0 + i * 4096u, 0, AUX_SYS);
#pragma unroll
      for (int i = 0; i < 8; ++i) v2[i] = __builtin_amdgcn_raw_buffer_load_b128(rs, ln0 + i * 4096u + 64u * PTILE, 0, AUX_SYS);
      add_partial(acc, v);
      add_partial(acc, v2);

      float m = 0.f;
#pragma unroll
      for (int i = 0; i < 4; ++i)
#pragma unroll
        for (int k = 0; k < 4; ++k)
#pragma unroll
          for (int e = 0; e < 4; ++e) m = fmaxf(m, fabsf(acc[i][k][e]));
      m = wave_max(m);
      if (lane == 0) red[wv] = m;
      BAR();
      if (t < NR) {
        const float mm = fmaxf(fmaxf(red[0], red[1]), fmaxf(red[2], red[3]));
        const u64 g = ((u64)call << 32) | __float_as_uint(mm);
        __hip_atomic_store((gu64*)gdst, g, __ATOMIC_RELAXED, __HIP_MEMORY_SCOPE_SYSTEM);
      }
      STAMP(8);
      BAR();
      const float r0 = red[16], r1 = red[17], r2 = red[18], r3 = red[19];
      const bool bad = fminf(fminf(r0, r1), fminf(r2, r3)) < 0.f;
      const float amax = fmaxf(fmaxf(r0, r1), fmaxf(r2, r3));
      STAMP(9);
      const float inv = amax > 0.f ? 127.0f / amax : 0.f;
      const float magic = 12582912.0f;
      const unsigned jc = d < 2u ? (j ^ 1u) : j;
      const unsigned pp = lane >> 4;
      const __amdgpu_buffer_rsrc_t ro = __builtin_amdgcn_make_buffer_rsrc(p.out, 0, 512u * 4096u, 0x00020000);
      const unsigned oo = (128u * a + 64u * (wv >> 1) + (lane & 15u)) * 4096u + jc * 2048u + n * 128u + 64u * (wv & 1u) + 16u * pp;
#pragma unroll
      for (int mb = 0; mb < 4; ++mb) {
        unsigned dw[4];
#pragma unroll
        for (int nb = 0; nb < 4; ++nb) {
          unsigned b[4];
#pragma unroll
          for (int e = 0; e < 4; ++e)
            b[e] = __float_as_uint(__builtin_amdgcn_fmed3f(__builtin_fmaf(acc[mb][nb][e], inv, magic), magic - 127.f, magic + 127.f));
          const unsigned lo = __builtin_amdgcn_perm(b[1], b[0], 0x0c0c0400u), hi = __builtin_amdgcn_perm(b[3], b[2], 0x04000c0cu);
          dw[nb] = bad ? 0x80808080u : (lo | hi);
        }
        const bool odd = pp & 1u;
        const u32x4 o = {odd ? dw[1] : dw[0], odd ? dw[0] : dw[1], odd ? dw[3] : dw[2], odd ? dw[2] : dw[3]};
        __builtin_amdgcn_raw_buffer_store_b128(o, ro, oo + mb * 65536u, 0, OUT_AUX);
      }
      STAMP(10);
    }
    if (t == 0) __hip_atomic_store(ctr, call, __ATOMIC_RELAXED, __HIP_MEMORY_SCOPE_AGENT);
  }
}

void make_jobs(Job* job) {
  int pos[32];
  for (int i = 0; i < 32; ++i) pos[ORD[i]] = i;
  for (int d = 0; d < 8; ++d) {
    Job* jb = job + d * 32;
    int key[32], idx[32], txs[32], tn[32];
    for (int m = 0; m < 32; ++m) {
      txs[m] = d < 4 ? (d < 2 ? 0 : 4) + (m & 3) : 8 + 2 * (d - 4) + (m >> 4);
      tn[m] = d < 4 ? 2 * (m >> 2) + (d & 1) : (m & 15);
      const int a = pos[txs[m]], b = pos[16 + tn[m]];
      key[m] = (a > b ? a : b) * 64 + m;
      idx[m] = m;
    }
    for (int a = 0; a < 32; ++a)
      for (int b = a + 1; b < 32; ++b)
        if (key[idx[b]] < key[idx[a]]) { const int tmp = idx[a]; idx[a] = idx[b]; idx[b] = tmp; }
    for (int l = 0; l < 32; ++l) {
      const int m = idx[(l + 24) & 31];
      jb[l].xs = (unsigned char)txs[m];
      jb[l].n = (unsigned char)tn[m];
      jb[l].pos = (unsigned char)(key[m] / 64);
      jb[l].skip0 = (unsigned char)(d >= 4 ? pos[2 * d] : 255);
      jb[l].skip1 = (unsigned char)(d >= 4 ? pos[2 * d + 1] : 255);
      if (d >= 2) jb[l].mode = (unsigned char)(l < 8 ? 8 + l : 32);
      else jb[l].mode = (unsigned char)(tn[m] < EARLY_N ? 16 : 32);
    }
  }
}

}

extern "C" void kernel_launch(int rank, int nranks, ncclComm_t comm, hipStream_t stream, void* const* d_in,
                              const int* in_sizes, int n_in, void* d_out, int out_size, const int* in_dtypes,
                              int out_dtype, void* const* d_out_all, void* const* d_ws_all, size_t ws_bytes) {
  if (nranks != NR || (unsigned)rank >= (unsigned)NR || n_in < 2 || ws_bytes < WS_END || out_size != 512 * 4096 ||
      in_sizes[0] != 4096 * 512 || in_sizes[1] != 512 * 4096) {
    fprintf(stderr, "kernel_launch: unexpected problem shape\n");
    return;
  }
  Params p;
  memset(&p, 0, sizeof p);
  p.x = (const char*)d_in[0];
  p.w = (const char*)d_in[1];
  p.out = (signed char*)d_out;
  for (int i = 0; i < NR; ++i) p.wsx[i] = (char*)d_ws_all[rank ^ i];
  p.rank = rank;
  make_jobs(p.job);
  hipLaunchKernelGGL(fused_gemm_rs_q8, dim3(NWG), dim3(NT), 0, stream, p);
  HIP_CHECK(hipGetLastError());
}
```

```cpp
#include <hip/hip_runtime.h>
#include <rccl/rccl.h>
#include <cstdio>
#include <cstdint>
#include <cstring>

#define HIP_CHECK(expr) do { hipError_t _e = (expr); if (_e != hipSuccess) { \
    fprintf(stderr, "HIP error %s at %s:%d\n", hipGetErrorString(_e), __FILE__, __LINE__); \
    return; } } while (0)

namespace {

typedef short s16x4 __attribute__((ext_vector_type(4)));
typedef short s16x8 __attribute__((ext_vector_type(8)));
typedef __bf16 bf16x8 __attribute__((ext_vector_type(8)));
typedef __bf16 bf16x2 __attribute__((ext_vector_type(2)));
typedef float f32x4 __attribute__((ext_vector_type(4)));
typedef float f32x2 __attribute__((ext_vector_type(2)));
typedef unsigned u32x4 __attribute__((ext_vector_type(4)));
typedef __attribute__((address_space(1))) unsigned gu32;
typedef __attribute__((address_space(1))) unsigned long long gu64;
typedef __attribute__((address_space(3))) char lchar;
typedef unsigned long long u64;

constexpr int NR = 8;
constexpr int NWG = 256, NT = 512;
constexpr unsigned ITEM = 65536;
constexpr unsigned PTILE = 32768;
constexpr unsigned STAGE = 32768;
#define NSTAGES 4
constexpr int NST = NSTAGES;
static_assert(NST == 4, "the tile hand-over assumes step counts that are multiples of the ring size");

constexpr size_t WS_CTR = 0;
constexpr size_t WS_TMO = 32 << 10;
constexpr size_t WS_TICK = 33 << 10;
constexpr size_t WS_FIN = 64 << 10;
constexpr size_t WS_FP = 96 << 10;
constexpr size_t WS_G = 128 << 10;
constexpr size_t WS_STAMP = 256 << 10;
constexpr size_t WS_XST = 1 << 20;
constexpr size_t WS_WST = 7 << 20;
constexpr size_t WS_P = 13 << 20;
constexpr size_t WS_PARK = 19 << 20;
constexpr size_t WS_END = 23 << 20;

constexpr u64 DEADLINE = 50000000ull;
constexpr int AUX_SYS = 17;

#define TAU 113
#define T0 100
#define STAG (TAU / 8)
#define ROT 1
#define PTAU 57
#define GATE (T0 + 30 * TAU)
#define OUT_AUX 17
#define PSLEEP 4
#define GSLEEP 2
#define SWEEP_T (GATE + 32 * PTAU + 150)
#define ACQ 0
#define DRAW_LEAD 110
#define TURNX 7u
#define JIT 5000
#define STATIC 1
#define SPEC 1
#define PSTAG (PTAU / 4)
#define DELAY 300
#define EARLY_N 4

constexpr unsigned char W_ = 16;
#define ORDV 1
constexpr unsigned char ORD[32] = {0, W_ + 0, 1, W_ + 1, 2, 3, W_ + 2, W_ + 3, 8, W_ + 4, 10, 12, W_ + 5, 14, W_ + 6, 9,
                                   W_ + 7, 11, W_ + 8, 13, W_ + 9, 15, W_ + 10, W_ + 11, W_ + 12, 4, W_ + 13, 5, W_ + 14, 6, W_ + 15, 7};
constexpr u64 ord_pack(int o) {
  u64 v = 0;
  for (int i = 0; i < 8; ++i) v |= (u64)ORD[o + i] << (8 * i);
  return v;
}

constexpr u64 skip_pack() {
  u64 v = 0;
  for (int i = 0; i < 32; ++i)
    if (ORD[i] >= 8 && ORD[i] < 16) v |= (u64)i << (8 * (ORD[i] - 8));
  return v;
}

struct alignas(8) Job {
  unsigned char xs;
  unsigned char n;
  unsigned char pos;
  unsigned char fin;
  unsigned char turn;
  unsigned char delay;
  unsigned short plan;
};

struct Params {
  const char* x;
  const char* w;
  signed char* out;
  char* wsx[NR];
  int rank;
  int pad;
  Job job[NR * 32];
};

__device__ __forceinline__ u64 now() { return __builtin_amdgcn_s_memrealtime(); }
__device__ __forceinline__ void wait_until(u64 t) { while (now() < t) __builtin_amdgcn_s_sleep(1); }
#define VM0() asm volatile("s_waitcnt vmcnt(0)" ::: "memory")
#define BAR() do { asm volatile("" ::: "memory"); __builtin_amdgcn_s_waitcnt(0xC07F);   __builtin_amdgcn_s_barrier(); asm volatile("" ::: "memory"); } while (0)
#define STR2(x) #x
#define STR(x) STR2(x)

#define STAMP(i) do {} while (0)
#define STAMPW(i) do {} while (0)
#define STAMP7(i) do {} while (0)
#define STAMP3(i) do {} while (0)

__device__ __forceinline__ unsigned ord_item(unsigned k) {
  const u64 a = (k & 16u) ? ((k & 8u) ? ord_pack(24) : ord_pack(16)) : ((k & 8u) ? ord_pack(8) : ord_pack(0));
  return (unsigned)(a >> (8u * (k & 7u))) & 0xffu;
}

__device__ __forceinline__ unsigned rowbase(unsigned q, unsigned xs) {
  const unsigned e = xs - 8u;
  const unsigned third = 512u * (q ^ (4u + (e >> 1))) + 256u * ((q >> 1) & 1u) + 128u * (e & 1u);
  const unsigned pair = 512u * (q ^ (xs < 4u ? 1u : 0u)) + 128u * (xs & 3u);
  return xs < 8u ? pair : third;
}

__device__ __forceinline__ void stage_load(const char* a, const char* b, const unsigned (&va)[4], const unsigned (&vb)[4],
                                           unsigned lds) {
  unsigned keep;
  asm volatile(
      "s_mov_b32 %0, m0\n\t"
      "s_mov_b32 m0, %11\n\t"
      "s_nop 2\n\t"
      "global_load_lds_dwordx4 %1, %9\n\t"
      "global_load_lds_dwordx4 %2, %9 offset:1024\n\t"
      "global_load_lds_dwordx4 %3, %9 offset:2048\n\t"
      "global_load_lds_dwordx4 %4, %9 offset:3072\n\t"
      "s_add_u32 m0, m0, 0x4000\n\t"
      "s_nop 0\n\t"
      "global_load_lds_dwordx4 %5, %10\n\t"
      "global_load_lds_dwordx4 %6, %10 offset:1024\n\t"
      "global_load_lds_dwordx4 %7, %10 offset:2048\n\t"
      "global_load_lds_dwordx4 %8, %10 offset:3072\n\t"
      "s_mov_b32 m0, %0"
      : "=&s"(keep)
      : "v"(va[0]), "v"(va[1]), "v"(va[2]), "v"(va[3]), "v"(vb[0]), "v"(vb[1]), "v"(vb[2]), "v"(vb[3]), "s"(a), "s"(b),
        "s"(lds)
      : "memory", "scc");
}

__device__ __forceinline__ unsigned nat_x(unsigned c) { return (c >> 3) * 1024u + (((c & 7u) ^ ((c >> 4) & 7u)) << 4); }
__device__ __forceinline__ unsigned nat_w(unsigned c) {
  const unsigned kr = c >> 4;
  return kr * 8192u + (((c & 15u) ^ ((kr & 3u) | (kr & 8u))) << 4);
}

struct Half { s16x8 x[4], w[4]; };

__device__ __forceinline__ void read_half(Half& f, lchar* st, unsigned fx, const unsigned (&fw)[4], int kk) {
#pragma unroll
  for (int i = 0; i < 4; ++i) {
    f.x[i] = *(__attribute__((address_space(3))) s16x8*)(st + fx + i * 2048);
    s16x4 lo = __builtin_amdgcn_ds_read_tr16_b64_v4i16((__attribute__((address_space(3))) s16x4*)(st + fw[i] + kk * 8192));
    s16x4 hi = __builtin_amdgcn_ds_read_tr16_b64_v4i16((__attribute__((address_space(3))) s16x4*)(st + fw[i] + kk * 8192 + 1024));
    f.w[i] = __builtin_shufflevector(lo, hi, 0, 1, 2, 3, 4, 5, 6, 7);
  }
}
__device__ __forceinline__ void mma(f32x4 (&acc)[4][4], const Half& f) {
#pragma unroll
  for (int mb = 0; mb < 4; ++mb)
#pragma unroll
    for (int nb = 0; nb < 4; ++nb)
      acc[mb][nb] = __builtin_amdgcn_mfma_f32_16x16x32_bf16(__builtin_bit_cast(bf16x8, f.w[nb]),
                                                            __builtin_bit_cast(bf16x8, f.x[mb]), acc[mb][nb], 0, 0, 0);
}
__device__ __forceinline__ void interleave() {
#pragma unroll
  for (int i = 0; i < 12; ++i) {
    __builtin_amdgcn_sched_group_barrier(0x8, 1, 0);
    __builtin_amdgcn_sched_group_barrier(0x100, 1, 0);
  }
  __builtin_amdgcn_sched_group_barrier(0x8, 4, 0);
}

__device__ __forceinline__ unsigned ring_next(unsigned o) { return o + STAGE == NST * STAGE ? 0u : o + STAGE; }
__device__ __forceinline__ u64 gemm_load(unsigned lds, unsigned lw, unsigned lane, const char* an, const char* bn,
                                         const char* as, const char* bs, int s0, int s1, int skip, gu64* next, u64 earliest) {
  u64 mine = 0;
  unsigned nx[4], nw[4], ln[4];
#pragma unroll
  for (unsigned i = 0; i < 4; ++i) {
    const unsigned c = (lw * 4u + i) * 64u + lane;
    nx[i] = nat_x(c) - i * 1024u;
    nw[i] = nat_w(c) - i * 1024u;
    ln[i] = lw * 4096u + lane * 16u;
  }
  unsigned ring = 0;
  auto issue = [&](int s) {
    const unsigned slot = lds + lw * 4096u + ring;
    if (s < 8) {
      stage_load(an + (unsigned)s * 128u, bn + (unsigned)s * (64u * 8192u), nx, nw, slot);
    } else {
      const unsigned o = (unsigned)(s - 8 + (s >= skip ? 4 : 0)) * 16384u;
      stage_load(as + o, bs + o, ln, ln, slot);
    }
    ring = ring_next(ring);
  };
  BAR();
  for (int s = s0; s < s0 + NST; ++s) issue(s);
  asm volatile("s_waitcnt vmcnt(%0)" ::"n"(8 * (NST - 1)) : "memory");
  BAR();
  for (int s = s0; s + NST < s1; ++s) {
    asm volatile("s_waitcnt vmcnt(16)" ::: "memory");
    BAR();
    issue(s + NST);
  }
  asm volatile("s_waitcnt vmcnt(16)" ::: "memory");
  BAR();
  if (next) {
    const u64 tn = now() + DRAW_LEAD, lo = earliest > tn ? earliest : tn, inc = PTAU;
    asm volatile("global_atomic_umax_x2 %1, %2, off\n\tglobal_atomic_add_x2 %0, %1, %3, off sc0"
                 : "=&v"(mine) : "v"(next), "v"(lo), "v"(inc) : "memory");
  }
  asm volatile("s_waitcnt vmcnt(8)" ::: "memory");
  BAR();
  asm volatile("s_waitcnt vmcnt(0)" : "+v"(mine)::"memory");
  BAR();
  return mine;
}
__device__ __forceinline__ void gemm_math(f32x4 (&acc)[4][4], lchar* sm, unsigned wv, unsigned lane, int s0, int s1) {
  const unsigned wm = wv >> 1, wn = wv & 1u, g = lane >> 4, m = lane & 15u;
  const unsigned fx0 = (64u * wm + m) * 128u + ((g ^ (m >> 1)) << 4), fx1 = fx0 ^ 64u;
  const unsigned qq = (lane >> 2) & 3u, pp = lane & 3u;
  const unsigned base = 16384u + (8u * g + qq) * 256u + (((8u * wn + 2u * pp) ^ (qq | ((g & 1u) << 3))) << 4) + ((pp & 1u) << 3);
  const unsigned fw[4] = {base, base ^ 8u, base ^ 16u, base ^ 24u};
  Half fa, fb;
  unsigned ring = 0;
  BAR();
  BAR();
  read_half(fa, sm, fx0, fw, 0);
  for (int s = s0; s + 1 < s1; ++s) {
    read_half(fb, sm + ring, fx1, fw, 1);
    mma(acc, fa);
    interleave();
    BAR();
    ring = ring_next(ring);
    read_half(fa, sm + ring, fx0, fw, 0);
    mma(acc, fb);
    interleave();
  }
  read_half(fb, sm + ring, fx1, fw, 1);
  mma(acc, fa);
  interleave();
  mma(acc, fb);
}

__device__ __forceinline__ bool give_up(gu32* tmo, unsigned call, u64 t0) {
  const unsigned v = __builtin_amdgcn_readfirstlane(__hip_atomic_load(tmo, __ATOMIC_RELAXED, __HIP_MEMORY_SCOPE_AGENT));
  if (v == call || now() - t0 > DEADLINE) {
    __hip_atomic_store(tmo, call, __ATOMIC_RELAXED, __HIP_MEMORY_SCOPE_AGENT);
    return true;
  }
  return false;
}

__device__ __forceinline__ void poll_flags(const char* f, const char* g, unsigned call, gu32* tmo) {
  const u64 t0 = now();
  for (unsigned it = 1;; ++it) {
    unsigned vf = 0, vg = 0;
    if (f) vf = __hip_atomic_load((gu32*)f, __ATOMIC_RELAXED, __HIP_MEMORY_SCOPE_SYSTEM);
    if (g) vg = __hip_atomic_load((gu32*)g, __ATOMIC_RELAXED, __HIP_MEMORY_SCOPE_SYSTEM);
    if (vf >= call) f = nullptr;
    if (vg >= call) g = nullptr;
    if (__all(f == nullptr && g == nullptr)) break;
    if ((it & 63u) == 0 && give_up(tmo, call, t0)) break;
    __builtin_amdgcn_s_sleep(PSLEEP);
  }
}

__device__ __forceinline__ unsigned pack_bf16(float a, float b) {
  f32x2 v = {a, b};
  return __builtin_bit_cast(unsigned, __builtin_convertvector(v, bf16x2));
}

__device__ __forceinline__ void add_partial(f32x4 (&acc)[4][4], const u32x4 (&v)[8]) {
#pragma unroll
  for (int i = 0; i < 8; ++i) {
#pragma unroll
    for (int h = 0; h < 2; ++h) {
      f32x4& c = acc[i >> 1][(i & 1) * 2 + h];
      const unsigned u0 = v[i][2 * h], u1 = v[i][2 * h + 1];
      c[0] += __uint_as_float(u0 << 16);
      c[1] += __uint_as_float(u0 & 0xffff0000u);
      c[2] += __uint_as_float(u1 << 16);
      c[3] += __uint_as_float(u1 & 0xffff0000u);
    }
  }
}

__device__ __forceinline__ unsigned wave_max(unsigned m) {
  m = max(m, (unsigned)__builtin_amdgcn_update_dpp(0, (int)m, 0x111, 0xf, 0xf, false));
  m = max(m, (unsigned)__builtin_amdgcn_update_dpp(0, (int)m, 0x112, 0xf, 0xf, false));
  m = max(m, (unsigned)__builtin_amdgcn_update_dpp(0, (int)m, 0x114, 0xf, 0xf, false));
  m = max(m, (unsigned)__builtin_amdgcn_update_dpp(0, (int)m, 0x118, 0xf, 0xf, false));
  return max(max((unsigned)__builtin_amdgcn_readlane((int)m, 15), (unsigned)__builtin_amdgcn_readlane((int)m, 31)),
             max((unsigned)__builtin_amdgcn_readlane((int)m, 47), (unsigned)__builtin_amdgcn_readlane((int)m, 63)));
}

__global__ void __launch_bounds__(NT, 2) fused_gemm_rs_q8(const Params p) {
  __shared__ __attribute__((aligned(1024))) char smem[NST * STAGE + 64];
  lchar* sm = (lchar*)smem;
  typedef __attribute__((address_space(3))) unsigned lu32;
  const u64 tstart = now();
  const unsigned t = threadIdx.x, lane = t & 63;
  const unsigned wv = __builtin_amdgcn_readfirstlane(t >> 6);
  const unsigned d = blockIdx.x & 7u, l = blockIdx.x >> 3;
  u64 a_job = ((const u64*)p.job)[blockIdx.x];
  char* a_peer = p.wsx[d == 0 ? 1u : d];
  char* a_me = p.wsx[0];
  const char* a_x = p.x;
  const char* a_w = p.w;
  signed char* a_out = p.out;
  unsigned a_rank = (unsigned)p.rank;
  asm volatile("" : "+s"(a_job), "+s"(a_peer), "+s"(a_me), "+s"(a_x), "+s"(a_w), "+s"(a_out), "+s"(a_rank));
  typedef __attribute__((address_space(1))) char gchar;
  char* const me = (char*)(gchar*)a_me;
  char* const peer = (char*)(gchar*)a_peer;
  const char* const px = (const char*)(const gchar*)a_x;
  const char* const pw = (const char*)(const gchar*)a_w;
  const Job job = __builtin_bit_cast(Job, a_job);
  const unsigned r = a_rank & 7u;
  const unsigned q = r ^ d;
  gu32* const ctr = (gu32*)(me + WS_CTR + blockIdx.x * 128);
  gu32* const tmo = (gu32*)(me + WS_TMO);
  unsigned prev = __hip_atomic_load(ctr, __ATOMIC_RELAXED, __HIP_MEMORY_SCOPE_AGENT);
  const unsigned j = r >> 2, xs = job.xs, n = job.n;
  const unsigned fa = job.fin & 3u, fn = job.fin >> 2;
  const bool ships = d < 2u || d >= 4u;
  const unsigned ln0 = t * 16u;

  lu32* const red = (lu32*)(sm + NST * STAGE);

  auto push = [&](unsigned k0, unsigned k1, unsigned kstep) {
      const unsigned u = l & 7u;
      const unsigned cq = q & 1u, jq = q >> 2;
      const unsigned skp = d >= 4u ? (unsigned)(skip_pack() >> (16u * (d - 4u))) & 0xffffu : 0xffffu;
      for (unsigned k = k0; k < k1; k += kstep) {
        const unsigned it = ord_item(k), sl = (u + ROT * k) & 7u;
        if (d >= 4u && (it >> 1) == d) continue;
        const unsigned slotk = k - (k > (skp & 255u) ? 1u : 0u) - (k > (skp >> 8) ? 1u : 0u);
        u32x4 v[8];
        if (it < 16u) {
          const unsigned xa = nat_x(lane), xb = nat_x(lane + 64u);
          const __amdgpu_buffer_rsrc_t rs = __builtin_amdgcn_make_buffer_rsrc((void*)px, 0, 4u << 20, 0x00020000);
          const unsigned off = (rowbase(q, it) + (sl & 1u) * 64u) * 1024u + cq * 512u + (sl >> 1) * 128u;
#pragma unroll
          for (unsigned i = 0; i < 8; ++i) v[i] = __builtin_amdgcn_raw_buffer_load_b128(rs, (i & 1u) ? xb : xa, off + (i >> 1) * 16384u, 0);
        } else {
          const unsigned wa = nat_w(lane), wb = nat_w(lane + 128u);
          const __amdgpu_buffer_rsrc_t rs = __builtin_amdgcn_make_buffer_rsrc((void*)pw, 0, 4u << 20, 0x00020000);
          const unsigned off = (cq * 256u + sl * 32u) * 8192u + (jq * 2048u + (it - 16u) * 128u) * 2u;
#pragma unroll
          for (unsigned i = 0; i < 8; ++i) v[i] = __builtin_amdgcn_raw_buffer_load_b128(rs, (i & 2u) ? wb : wa, off + (i & 5u) * 32768u, 0);
        }
        char* dst = peer + (it < 16u ? WS_XST : WS_WST) + (size_t)(((it & 15u) * 6u + (d - 2u)) * ITEM) + sl * 8192u;
        const __amdgpu_buffer_rsrc_t rd = __builtin_amdgcn_make_buffer_rsrc(dst, 0, 8192u, 0x00020000);
        wait_until(tstart + T0 + slotk * TAU + u * STAG);
#pragma unroll
        for (unsigned i = 0; i < 8; ++i) __builtin_amdgcn_raw_buffer_store_b128(v[i], rd, lane * 16u, i * 1024u, AUX_SYS);
        VM0();
        unsigned pv = prev;
        asm volatile("" : "+v"(pv));
        if (lane == 0)
          __hip_atomic_store((gu32*)(peer + WS_FIN + ((d - 2u) * 32u + it) * 128u + u * 4u), pv + 1u, __ATOMIC_RELAXED, __HIP_MEMORY_SCOPE_SYSTEM);
      }
      STAMPW(1);
  };
  if (d >= 2u && l < 8u) push(wv, 16u, 8u);

  if (wv >= 4) {
    for (unsigned plan = job.plan | 0xf0000u; (plan & 15u) != 15u; plan >>= 4) {
      const unsigned kind = plan & 15u;
      if (kind == 6u) { wait_until(tstart + job.delay * 16u); continue; }
      if (kind == 5u) { push(12u + wv, 32u, 4u); continue; }
      const unsigned call = __builtin_amdgcn_readfirstlane(prev) + 1u;
      const bool staged = kind == 1u || kind == 2u;
      const int s0 = kind == 1u ? 8 : 0, s1 = staged ? (d >= 4u ? 28 : 32) : 8;
      if (staged && wv == 4) {
        const unsigned s = lane >> 3;
        const bool act = lane < 48u && !(d >= 4u && s == d - 2u);
        wait_until(tstart + T0 + job.pos * TAU);
        poll_flags(act ? me + WS_FIN + (s * 32u + xs) * 128u + (lane & 7u) * 4u : nullptr,
                   act ? me + WS_FIN + (s * 32u + 16u + n) * 128u + (lane & 7u) * 4u : nullptr, call, tmo);
        STAMPW(12);
        if (ACQ) {
          __builtin_amdgcn_fence(__ATOMIC_ACQUIRE, "");
          VM0();
        }
      }
      const u64 mine = gemm_load((unsigned)(size_t)sm, wv - 4u, lane,
                px + (size_t)(kind == 3u ? 512u * r + 128u * fa : rowbase(r, xs)) * 1024u,
                pw + (kind == 3u ? (j ^ 1u) * 2048u + fn * 128u : j * 2048u + n * 128u) * 2u, me + WS_XST + (size_t)(xs * 6u * ITEM),
                me + WS_WST + (size_t)(n * 6u * ITEM), s0, s1, d >= 4u ? (int)d * 4 : 1000,
                staged && ships && t == 256u ? (gu64*)(me + WS_TICK + (d < 2u ? 0u : d) * 128u) : nullptr,
                d >= 4u ? tstart + GATE : 0ull);
      if (staged && ships) {
        if (t == 256u) red[12] = (unsigned)(mine - tstart);
        BAR();
        u32x4 v[8];
#pragma unroll
        for (int i = 0; i < 8; ++i) v[i] = *(__attribute__((address_space(3))) u32x4*)(sm + (wv - 4u) * 8192u + i * 1024u + lane * 16u);
        int go = (int)__builtin_amdgcn_readfirstlane(red[12]);
        if (go > (int)(now() - tstart) + GATE + 64 * PTAU) go = 0;
        if (STATIC && d >= 4u) go = GATE + (int)(job.turn ^ ((r & 1u) * TURNX)) * PTAU;
        const unsigned a = d < 2u ? xs : 2u * ((r >> 1) & 1u) + (xs & 1u);
        const unsigned slot = ((d < 2u ? 0u : 1u + (r & 1u)) * 4u + a) * 16u + n;
        const __amdgpu_buffer_rsrc_t rs = __builtin_amdgcn_make_buffer_rsrc(peer + WS_P + (size_t)slot * PTILE + (wv - 4u) * 8192u, 0, 8192u, 0x00020000);
        if (go > 0) wait_until(tstart + (unsigned)go + (wv - 4u) * PSTAG);
        STAMPW(4);
#pragma unroll
        for (int i = 0; i < 8; ++i) __builtin_amdgcn_raw_buffer_store_b128(v[i], rs, lane * 16u, i * 1024u, AUX_SYS);
        VM0();
        if (lane == 0) __hip_atomic_store((gu32*)(peer + WS_FP + slot * 128u + (wv - 4u) * 4u), call, __ATOMIC_RELAXED, __HIP_MEMORY_SCOPE_SYSTEM);
        STAMP7(6);
      }
    }
    if (d < 4u) {
      const unsigned call = __builtin_amdgcn_readfirstlane(prev) + 1u;
      const __amdgpu_buffer_rsrc_t rg = __builtin_amdgcn_make_buffer_rsrc(me + WS_G, 0, 8192, 0x00020000);
      u32x4 g = {0u, 0u, 0u, 0u}, h = g;
      wait_until(tstart + SWEEP_T);
      const u64 t0 = now();
      unsigned state = 1u;
      for (;;) {
        if (!(g[1] == call && g[3] == call)) g = __builtin_amdgcn_raw_buffer_load_b128(rg, ln0 - 4096u, 0, AUX_SYS);
        if (!(h[1] == call && h[3] == call)) h = __builtin_amdgcn_raw_buffer_load_b128(rg, ln0, 0, AUX_SYS);
        if (give_up(tmo, call, t0)) { state = 2u; break; }
        const unsigned m = wave_max(max(max(g[1] == call ? g[0] : 0u, g[3] == call ? g[2] : 0u),
                                        max(h[1] == call ? h[0] : 0u, h[3] == call ? h[2] : 0u)));
        if (lane == 0) __hip_atomic_store(red + wv, m, __ATOMIC_RELAXED, __HIP_MEMORY_SCOPE_WORKGROUP);
        if (__all(g[1] == call && g[3] == call && h[1] == call && h[3] == call)) break;
        __builtin_amdgcn_s_sleep(GSLEEP);
        asm volatile("" ::: "memory");
      }
      if (lane == 0) __hip_atomic_store(red + 4u + wv, state, __ATOMIC_RELEASE, __HIP_MEMORY_SCOPE_WORKGROUP);
      STAMPW(13);
    }
  } else {
    f32x4 acc[4][4];
#pragma unroll
    for (int i = 0; i < 4; ++i)
#pragma unroll
      for (int k = 0; k < 4; ++k) acc[i][k] = f32x4{0.f, 0.f, 0.f, 0.f};
    if (t < 12u) red[t] = 0u;
    char* gdst = p.wsx[lane & 7u] + WS_G + (r * 128u + d * 32u + l) * 8u;
    asm volatile("" : "+v"(gdst));
    const bool parks = d < 2u && (job.plan >> 12) == 1u;
    const __amdgpu_buffer_rsrc_t rpark = __builtin_amdgcn_make_buffer_rsrc(me + WS_PARK + (size_t)((d & 1u) * 32u + l) * 65536u, 0, 65536u, 0x00020000);
    for (unsigned plan = job.plan | 0xf0000u; (plan & 15u) != 15u; plan >>= 4) {
      const unsigned kind = plan & 15u;
      if (kind >= 4u) continue;
      const bool staged = kind == 1u || kind == 2u;
      const int s0 = kind == 1u ? 8 : 0, s1 = staged ? (d >= 4u ? 28 : 32) : 8;
      if (staged) STAMP(3);
      if (kind != 1u) {
#pragma unroll
        for (int i = 0; i < 4; ++i)
#pragma unroll
          for (int k = 0; k < 4; ++k) acc[i][k] = f32x4{0.f, 0.f, 0.f, 0.f};
      }
      gemm_math(acc, sm, wv, lane, s0, s1);
      if (kind == 3u && parks) {
#pragma unroll
        for (int i = 0; i < 16; ++i)
          __builtin_amdgcn_raw_buffer_store_b128(__builtin_bit_cast(u32x4, acc[i >> 2][i & 3]), rpark, ln0, i * 4096u, 0);
      }
      if (kind == 0u) STAMP(2);
      if (staged) STAMP(5);
      if (staged && ships) {
#pragma unroll
        for (int i = 0; i < 8; ++i) {
          const f32x4 c0 = acc[i >> 1][(i & 1) * 2], c1 = acc[i >> 1][(i & 1) * 2 + 1];
          *(__attribute__((address_space(3))) u32x4*)(sm + wv * 8192u + i * 1024u + lane * 16u) =
              u32x4{pack_bf16(c0[0], c0[1]), pack_bf16(c0[2], c0[3]), pack_bf16(c1[0], c1[1]), pack_bf16(c1[2], c1[3])};
        }
        BAR();
        if (parks) {
#pragma unroll
          for (int i = 0; i < 16; ++i)
            acc[i >> 2][i & 3] = __builtin_bit_cast(f32x4, __builtin_amdgcn_raw_buffer_load_b128(rpark, ln0, i * 4096u, 0));
        }
      }
    }
    const unsigned call = __builtin_amdgcn_readfirstlane(prev) + 1u;
    if (d < 4u) {
      const unsigned slot = ((d < 2u ? 4u : 0u) + fa) * 16u + fn;
      STAMP3(11);
      unsigned pend = d < 2u ? 3u : 1u;
      do {
        unsigned got;
        const u64 t0 = now();
        for (unsigned it = 1;; ++it) {
          unsigned f = 0;
          if (lane < 2u && ((pend >> lane) & 1u))
            f = __hip_atomic_load((gu32*)(me + WS_FP + (slot + lane * 64u) * 128u + wv * 4u), __ATOMIC_RELAXED, __HIP_MEMORY_SCOPE_SYSTEM);
          got = (unsigned)__ballot(f >= call) & pend;
          if (got) break;
          if ((it & 63u) == 0 && give_up(tmo, call, t0)) { got = pend; break; }
          __builtin_amdgcn_s_sleep(PSLEEP);
        }
        got = __builtin_amdgcn_readfirstlane(got);
        STAMP3(7);
        char* const pb = me + WS_P + (size_t)slot * PTILE + wv * 8192u;
        const __amdgpu_buffer_rsrc_t r0 = __builtin_amdgcn_make_buffer_rsrc(pb, 0, got & 1u ? 8192u : 0u, 0x00020000);
        const __amdgpu_buffer_rsrc_t r1 = __builtin_amdgcn_make_buffer_rsrc(pb + 64u * PTILE, 0, got & 2u ? 8192u : 0u, 0x00020000);
        u32x4 v[8], w[8];
        asm volatile("" ::: "memory");
#pragma unroll
        for (int i = 0; i < 8; ++i) v[i] = __builtin_amdgcn_raw_buffer_load_b128(r0, lane * 16u, i * 1024u, AUX_SYS);
#pragma unroll
        for (int i = 0; i < 8; ++i) w[i] = __builtin_amdgcn_raw_buffer_load_b128(r1, lane * 16u, i * 1024u, AUX_SYS);
        add_partial(acc, v);
        add_partial(acc, w);
        pend &= ~got;
      } while (pend);
      float mf = 0.f;
#pragma unroll
      for (int i = 0; i < 4; ++i)
#pragma unroll
        for (int k = 0; k < 4; ++k)
#pragma unroll
          for (int e = 0; e < 4; ++e) mf = fmaxf(mf, fabsf(acc[i][k][e]));
      const unsigned mw = wave_max(__float_as_uint(mf));
      unsigned arr = 0;
      if (lane == 0) {
        __hip_atomic_fetch_max(red, mw, __ATOMIC_RELAXED, __HIP_MEMORY_SCOPE_WORKGROUP);
        arr = __hip_atomic_fetch_add(red + 1, 1u, __ATOMIC_ACQ_REL, __HIP_MEMORY_SCOPE_WORKGROUP);
      }
      if (__builtin_amdgcn_readfirstlane(arr) == 3u && lane < NR) {
        const u64 g = ((u64)call << 32) | __hip_atomic_load(red, __ATOMIC_RELAXED, __HIP_MEMORY_SCOPE_WORKGROUP);
        __hip_atomic_store((gu64*)gdst, g, __ATOMIC_RELAXED, __HIP_MEMORY_SCOPE_SYSTEM);
      }
      STAMP3(8);
      const unsigned jc = d < 2u ? (j ^ 1u) : j;
      const unsigned pp = lane >> 4;
      const __amdgpu_buffer_rsrc_t ro = __builtin_amdgcn_make_buffer_rsrc(a_out, 0, 512u * 4096u, 0x00020000);
      const unsigned oo = (128u * fa + 64u * (wv >> 1) + (lane & 15u)) * 4096u + jc * 2048u + fn * 128u + 64u * (wv & 1u) + 16u * pp;
      unsigned used = 0xffffffffu;
      for (;;) {
        const unsigned s0 = __hip_atomic_load(red + 8, __ATOMIC_ACQUIRE, __HIP_MEMORY_SCOPE_WORKGROUP), s1 = __hip_atomic_load(red + 9, __ATOMIC_ACQUIRE, __HIP_MEMORY_SCOPE_WORKGROUP),
                       s2 = __hip_atomic_load(red + 10, __ATOMIC_ACQUIRE, __HIP_MEMORY_SCOPE_WORKGROUP), s3 = __hip_atomic_load(red + 11, __ATOMIC_ACQUIRE, __HIP_MEMORY_SCOPE_WORKGROUP);
        const unsigned m0 = __hip_atomic_load(red + 4, __ATOMIC_RELAXED, __HIP_MEMORY_SCOPE_WORKGROUP), m1 = __hip_atomic_load(red + 5, __ATOMIC_RELAXED, __HIP_MEMORY_SCOPE_WORKGROUP),
                       m2 = __hip_atomic_load(red + 6, __ATOMIC_RELAXED, __HIP_MEMORY_SCOPE_WORKGROUP), m3 = __hip_atomic_load(red + 7, __ATOMIC_RELAXED, __HIP_MEMORY_SCOPE_WORKGROUP);
        const bool fin = __builtin_amdgcn_readfirstlane(min(min(s0, s1), min(s2, s3))) != 0u;
        const bool bad = __builtin_amdgcn_readfirstlane(max(max(s0, s1), max(s2, s3))) == 2u;
        unsigned cur = __builtin_amdgcn_readfirstlane(max(max(m0, m1), max(m2, m3)));
        const bool seen = __builtin_amdgcn_readfirstlane(min(min(m0, m1), min(m2, m3))) != 0u;
        if (bad) cur = 0xfffffffeu;
        if (cur != used && (fin || (SPEC && seen && cur >= mw))) {
          const float amax = __uint_as_float(cur);
          const float inv = amax > 0.f ? 127.0f / amax : 0.f;
          const float magic = 12582912.0f;
          VM0();
#pragma unroll
          for (int mb = 0; mb < 4; ++mb) {
            unsigned dw[4];
#pragma unroll
            for (int nb = 0; nb < 4; ++nb) {
              unsigned b[4];
#pragma unroll
              for (int e = 0; e < 4; ++e)
                b[e] = __float_as_uint(__builtin_amdgcn_fmed3f(__builtin_fmaf(acc[mb][nb][e], inv, magic), magic - 127.f, magic + 127.f));
              const unsigned lo = __builtin_amdgcn_perm(b[1], b[0], 0x0c0c0400u), hi = __builtin_amdgcn_perm(b[3], b[2], 0x04000c0cu);
              dw[nb] = bad ? 0x80808080u : (lo | hi);
            }
            const bool odd = pp & 1u;
            const u32x4 o = {odd ? dw[1] : dw[0], odd ? dw[0] : dw[1], odd ? dw[3] : dw[2], odd ? dw[2] : dw[3]};
            __builtin_amdgcn_raw_buffer_store_b128(o, ro, oo + mb * 65536u, 0, OUT_AUX);
          }
          used = cur;
          STAMP3(9);
        }
        if (fin) break;
        __builtin_amdgcn_s_sleep(1);
      }
      STAMP(10);
    }
  if (t == 0) __hip_atomic_store(ctr, call, __ATOMIC_RELAXED, __HIP_MEMORY_SCOPE_AGENT);
  }
}

void make_jobs(Job* job) {
  int pos[32];
  for (int i = 0; i < 32; ++i) pos[ORD[i]] = i;
  for (int d = 0; d < 8; ++d) {
    Job jb[32];
    int key[32], idx[32], txs[32], tn[32];
    for (int m = 0; m < 32; ++m) {
      txs[m] = d < 4 ? (d < 2 ? 0 : 4) + (m & 3) : 8 + 2 * (d - 4) + (m >> 4);
      tn[m] = d < 4 ? 2 * (m >> 2) + (d & 1) : (m & 15);
      const int a = pos[txs[m]], b = pos[16 + tn[m]];
      key[m] = (a > b ? a : b) * 64 + m;
      idx[m] = m;
    }
    for (int a = 0; a < 32; ++a)
      for (int b = a + 1; b < 32; ++b)
        if (key[idx[b]] < key[idx[a]]) { const int tmp = idx[a]; idx[a] = idx[b]; idx[b] = tmp; }
    for (int l = 0; l < 32; ++l) {
      const int m = idx[(l + 16) & 31];
      jb[l].xs = (unsigned char)txs[m];
      jb[l].n = (unsigned char)tn[m];
      jb[l].pos = (unsigned char)(key[m] / 64);
      const int fm = d < 2 ? idx[31 - ((l + 16) & 31)] : m;
      jb[l].fin = (unsigned char)((txs[fm] & 3) | (tn[fm] << 2));
      if (d >= 2) jb[l].plan = l < 8 ? 0xff10 : l < 16 ? 0x1506 : 0xf106;
      else jb[l].plan = tn[m] < EARLY_N ? 0xf310 : 0x1036;
      jb[l].turn = (unsigned char)((l + 16) & 31);
      int dl = T0 + (d >= 2 && l < 16 ? 16 : key[m] / 64) * TAU - (d < 2 ? 2 : 1) * JIT;
      dl = dl < DELAY ? DELAY : dl;
      jb[l].delay = (unsigned char)(dl / 16);
      job[l * 8 + d] = jb[l];
    }
  }
}

}

extern "C" void kernel_launch(int rank, int nranks, ncclComm_t comm, hipStream_t stream, void* const* d_in,
                              const int* in_sizes, int n_in, void* d_out, int out_size, const int* in_dtypes,
                              int out_dtype, void* const* d_out_all, void* const* d_ws_all, size_t ws_bytes) {
  if (nranks != NR || (unsigned)rank >= (unsigned)NR || n_in < 2 || ws_bytes < WS_END || out_size != 512 * 4096 ||
      in_sizes[0] != 4096 * 512 || in_sizes[1] != 512 * 4096) {
    fprintf(stderr, "kernel_launch: unexpected problem shape\n");
    return;
  }
  Params p;
  memset(&p, 0, sizeof p);
  p.x = (const char*)d_in[0];
  p.w = (const char*)d_in[1];
  p.out = (signed char*)d_out;
  for (int i = 0; i < NR; ++i) p.wsx[i] = (char*)d_ws_all[rank ^ i];
  p.rank = rank;
  make_jobs(p.job);
  hipLaunchKernelGGL(fused_gemm_rs_q8, dim3(NWG), dim3(NT), 0, stream, p);
  HIP_CHECK(hipGetLastError());
}
```

```cpp
#include <hip/hip_runtime.h>
#include <rccl/rccl.h>
#include <cstdio>
#include <cstdint>
#include <cstring>

#define HIP_CHECK(expr) do { hipError_t _e = (expr); if (_e != hipSuccess) { \
    fprintf(stderr, "HIP error %s at %s:%d\n", hipGetErrorString(_e), __FILE__, __LINE__); \
    return; } } while (0)

namespace {

typedef short s16x4 __attribute__((ext_vector_type(4)));
typedef short s16x8 __attribute__((ext_vector_type(8)));
typedef __bf16 bf16x8 __attribute__((ext_vector_type(8)));
typedef __bf16 bf16x2 __attribute__((ext_vector_type(2)));
typedef float f32x4 __attribute__((ext_vector_type(4)));
typedef float f32x2 __attribute__((ext_vector_type(2)));
typedef unsigned u32x4 __attribute__((ext_vector_type(4)));
typedef __attribute__((address_space(1))) unsigned gu32;
typedef __attribute__((address_space(1))) unsigned long long gu64;
typedef __attribute__((address_space(3))) char lchar;
typedef unsigned long long u64;

constexpr int NR = 8;
constexpr int NWG = 256, NT = 512;
constexpr unsigned ITEM = 65536;
constexpr unsigned PTILE = 32768;
constexpr unsigned STAGE = 32768;
#define NSTAGES 4
constexpr int NST = NSTAGES;
static_assert(NST == 4, "the tile hand-over assumes step counts that are multiples of the ring size");

constexpr size_t WS_CTR = 0;
constexpr size_t WS_TMO = 32 << 10;
constexpr size_t WS_TICK = 33 << 10;
constexpr size_t WS_FIN = 64 << 10;
constexpr size_t WS_FP = 96 << 10;
constexpr size_t WS_G = 128 << 10;
constexpr size_t WS_STAMP = 256 << 10;
constexpr size_t WS_SPARE = 512 << 10;
constexpr size_t WS_IN = (2 << 20) + 65536;
constexpr unsigned SRC_STRIDE = (2u << 20) + 65536u;
constexpr size_t WS_P = 16 << 20;
constexpr size_t WS_PARK = 24 << 20;
constexpr size_t WS_END = 28 << 20;
static_assert(WS_IN + 6 * (size_t)SRC_STRIDE <= WS_P, "workspace layout");

constexpr u64 DEADLINE = 50000000ull;
constexpr int AUX_SYS = 17;

#define TAU 113
#define T0 160
#define STAG (TAU / 8)
#define ROT 1
#define PTAU 57
#define GATE (T0 + NCR * TAU)
#define OUT_AUX 17
#define PSLEEP 4
#define GSLEEP 2
#define SWEEP_T (GATE + 32 * PTAU + 150)
#define ACQ 0
#define DRAW_LEAD 110
#define TURNX 3u
#define RELAY_N 12
#define RELAY_R1 4
#define RGATE (T0 + NIN * TAU)
#define PRETOUCH 7
#define RESTART 100
#define SRC_T 124
#define SPLIT_GAIN 100
#define SPEC 1
#define PSTAG (PTAU / 4)
#define DELAY 300
#define EARLY_N 4

#define NF 4
constexpr unsigned char W_ = 16, F_ = 32, G_ = 40;
constexpr int NIN = 32 + NF, NCR = 30 - NF / 2;
#define ORD_HEAD 0, W_ + 0, 1, W_ + 1, 2, 3, W_ + 2, W_ + 3, 8, W_ + 4, 10, 12, W_ + 5, 14, W_ + 6, 9, W_ + 7, 11, W_ + 8, 13, W_ + 9, 15
#define ORD_W W_ + 10, W_ + 11, W_ + 12, W_ + 13, W_ + 14, W_ + 15
constexpr unsigned char OIN[40] = {ORD_HEAD, ORD_W, F_ + 0, F_ + 1, F_ + 2, F_ + 3, 4, 5, 6, 7};
constexpr unsigned char OCR[40] = {ORD_HEAD, G_ + 0, G_ + 1, W_ + 10, W_ + 11, 6, 7};
constexpr u64 ord_pack(const unsigned char* o) {
  u64 v = 0;
  for (int i = 0; i < 8; ++i) v |= (u64)o[i] << (8 * i);
  return v;
}
static_assert(ord_pack(OIN) == ord_pack(OCR) && ord_pack(OIN + 8) == ord_pack(OCR + 8), "common head");
__host__ __device__ __forceinline__ unsigned ord_item(unsigned k, unsigned d) {
  const u64 hi = d < 4u ? ((k & 32u) ? ord_pack(OIN + 32) : (k & 8u) ? ord_pack(OIN + 24) : ord_pack(OIN + 16))
                        : ((k & 8u) ? ord_pack(OCR + 24) : ord_pack(OCR + 16));
  const u64 a = k >= 16u ? hi : (k & 8u) ? ord_pack(OIN + 8) : ord_pack(OIN);
  unsigned it = (unsigned)(a >> (8u * (k & 7u))) & 0xffu;
  if (d >= 4u && (it >> 1) == d) it = 4u + (it & 1u);
  if (it >= G_) it = 32u - NF + 2u * (it - G_) + ((d >> 1) & 1u);
  return it;
}

struct alignas(8) Job {
  unsigned char tile;
  unsigned char early;
  unsigned char pos;
  unsigned char fin;
  unsigned char turn;
  unsigned char pos2;
  unsigned short plan;
};

struct Params {
  const char* x;
  const char* w;
  signed char* out;
  char* wsx[NR];
  int rank;
  int pad;
  Job job[NR * 32];
};

__device__ __forceinline__ u64 now() { return __builtin_amdgcn_s_memrealtime(); }
__device__ __forceinline__ void wait_until(u64 t) { while (now() < t) __builtin_amdgcn_s_sleep(1); }
#define VM0() asm volatile("s_waitcnt vmcnt(0)" ::: "memory")
#define BAR() do { asm volatile("" ::: "memory"); __builtin_amdgcn_s_waitcnt(0xC07F);   __builtin_amdgcn_s_barrier(); asm volatile("" ::: "memory"); } while (0)
#define STR2(x) #x
#define STR(x) STR2(x)

#define STAMP(i) do {} while (0)
#define STAMPW(i) do {} while (0)
#define STAMP7(i) do {} while (0)
#define STAMP3(i) do {} while (0)

__device__ __forceinline__ unsigned rowbase(unsigned q, unsigned xs) {
  const unsigned e = xs - 8u;
  const unsigned third = 512u * (q ^ (4u + (e >> 1))) + 256u * ((q >> 1) & 1u) + 128u * (e & 1u);
  const unsigned pair = 512u * (q ^ (xs < 4u ? 1u : 0u)) + 128u * (xs & 3u);
  return xs < 8u ? pair : third;
}

__device__ __forceinline__ void stage_load(const char* a, const char* b, const unsigned (&va)[4], const unsigned (&vb)[4],
                                           unsigned lds) {
  unsigned keep;
  asm volatile(
      "s_mov_b32 %0, m0\n\t"
      "s_mov_b32 m0, %11\n\t"
      "s_nop 2\n\t"
      "global_load_lds_dwordx4 %1, %9\n\t"
      "global_load_lds_dwordx4 %2, %9 offset:1024\n\t"
      "global_load_lds_dwordx4 %3, %9 offset:2048\n\t"
      "global_load_lds_dwordx4 %4, %9 offset:3072\n\t"
      "s_add_u32 m0, m0, 0x4000\n\t"
      "s_nop 0\n\t"
      "global_load_lds_dwordx4 %5, %10\n\t"
      "global_load_lds_dwordx4 %6, %10 offset:1024\n\t"
      "global_load_lds_dwordx4 %7, %10 offset:2048\n\t"
      "global_load_lds_dwordx4 %8, %10 offset:3072\n\t"
      "s_mov_b32 m0, %0"
      : "=&s"(keep)
      : "v"(va[0]), "v"(va[1]), "v"(va[2]), "v"(va[3]), "v"(vb[0]), "v"(vb[1]), "v"(vb[2]), "v"(vb[3]), "s"(a), "s"(b),
        "s"(lds)
      : "memory", "scc");
}

__device__ __forceinline__ unsigned nat_x(unsigned c) { return (c >> 3) * 1024u + (((c & 7u) ^ ((c >> 4) & 7u)) << 4); }
__device__ __forceinline__ unsigned nat_w(unsigned c) {
  const unsigned kr = c >> 4;
  return kr * 8192u + (((c & 15u) ^ ((kr & 3u) | (kr & 8u))) << 4);
}

struct Half { s16x8 x[4], w[4]; };

__device__ __forceinline__ void read_half(Half& f, lchar* st, unsigned fx, const unsigned (&fw)[4], int kk) {
#pragma unroll
  for (int i = 0; i < 4; ++i) {
    f.x[i] = *(__attribute__((address_space(3))) s16x8*)(st + fx + i * 2048);
    s16x4 lo = __builtin_amdgcn_ds_read_tr16_b64_v4i16((__attribute__((address_space(3))) s16x4*)(st + fw[i] + kk * 8192));
    s16x4 hi = __builtin_amdgcn_ds_read_tr16_b64_v4i16((__attribute__((address_space(3))) s16x4*)(st + fw[i] + kk * 8192 + 1024));
    f.w[i] = __builtin_shufflevector(lo, hi, 0, 1, 2, 3, 4, 5, 6, 7);
  }
}
__device__ __forceinline__ void mma(f32x4 (&acc)[4][4], const Half& f) {
#pragma unroll
  for (int mb = 0; mb < 4; ++mb)
#pragma unroll
    for (int nb = 0; nb < 4; ++nb)
      acc[mb][nb] = __builtin_amdgcn_mfma_f32_16x16x32_bf16(__builtin_bit_cast(bf16x8, f.w[nb]),
                                                            __builtin_bit_cast(bf16x8, f.x[mb]), acc[mb][nb], 0, 0, 0);
}
__device__ __forceinline__ void interleave() {
#pragma unroll
  for (int i = 0; i < 12; ++i) {
    __builtin_amdgcn_sched_group_barrier(0x8, 1, 0);
    __builtin_amdgcn_sched_group_barrier(0x100, 1, 0);
  }
  __builtin_amdgcn_sched_group_barrier(0x8, 4, 0);
}

constexpr unsigned LDS_RED = NST * STAGE, LDS_BYTES = LDS_RED + 64;
__device__ __forceinline__ unsigned ring_next(unsigned o) { return o + STAGE == NST * STAGE ? 0u : o + STAGE; }
__device__ __forceinline__ u64 gemm_load(unsigned lds, unsigned lw, unsigned lane, const char* an, const char* bn,
                                         const char* as, const char* bs, int s0, int s1, unsigned srcs, gu64* next) {
  u64 mine = 0;
  unsigned nx[4], nw[4], ln[4];
#pragma unroll
  for (unsigned i = 0; i < 4; ++i) {
    const unsigned c = (lw * 4u + i) * 64u + lane;
    nx[i] = nat_x(c) - i * 1024u;
    nw[i] = nat_w(c) - i * 1024u;
    ln[i] = lw * 4096u + lane * 16u;
  }
  unsigned ring = 0;
  auto issue = [&](int s) {
    const unsigned slot = lds + lw * 4096u + ring;
    if (s < 8) {
      stage_load(an + (unsigned)s * 128u, bn + (unsigned)s * (64u * 8192u), nx, nw, slot);
    } else {
      const unsigned ps = (unsigned)(s - 8), o = ((srcs >> (ps & ~3u)) & 7u) * SRC_STRIDE + (ps & 3u) * 16384u;
      stage_load(as + o, bs + o, ln, ln, slot);
    }
    ring = ring_next(ring);
  };
  BAR();
  for (int s = s0; s < s0 + NST; ++s) issue(s);
  asm volatile("s_waitcnt vmcnt(%0)" ::"n"(8 * (NST - 1)) : "memory");
  BAR();
  for (int s = s0; s + NST < s1; ++s) {
    asm volatile("s_waitcnt vmcnt(16)" ::: "memory");
    BAR();
    issue(s + NST);
  }
  asm volatile("s_waitcnt vmcnt(16)" ::: "memory");
  BAR();
  if (next) {
    const u64 lo = now() + DRAW_LEAD, inc = PTAU;
    asm volatile("global_atomic_umax_x2 %1, %2, off\n\tglobal_atomic_add_x2 %0, %1, %3, off sc0"
                 : "=&v"(mine) : "v"(next), "v"(lo), "v"(inc) : "memory");
  }
  asm volatile("s_waitcnt vmcnt(8)" ::: "memory");
  BAR();
  asm volatile("s_waitcnt vmcnt(0)" : "+v"(mine)::"memory");
  BAR();
  return mine;
}
__device__ __forceinline__ void gemm_math(f32x4 (&acc)[4][4], lchar* sm, unsigned wv, unsigned lane, int s0, int s1) {
  const unsigned wm = wv >> 1, wn = wv & 1u, g = lane >> 4, m = lane & 15u;
  const unsigned fx0 = (64u * wm + m) * 128u + ((g ^ (m >> 1)) << 4), fx1 = fx0 ^ 64u;
  const unsigned qq = (lane >> 2) & 3u, pp = lane & 3u;
  const unsigned base = 16384u + (8u * g + qq) * 256u + (((8u * wn + 2u * pp) ^ (qq | ((g & 1u) << 3))) << 4) + ((pp & 1u) << 3);
  const unsigned fw[4] = {base, base ^ 8u, base ^ 16u, base ^ 24u};
  Half fa, fb;
  unsigned ring = 0;
  BAR();
  BAR();
  read_half(fa, sm, fx0, fw, 0);
  for (int s = s0; s + 1 < s1; ++s) {
    read_half(fb, sm + ring, fx1, fw, 1);
    mma(acc, fa);
    interleave();
    BAR();
    ring = ring_next(ring);
    read_half(fa, sm + ring, fx0, fw, 0);
    mma(acc, fb);
    interleave();
  }
  read_half(fb, sm + ring, fx1, fw, 1);
  mma(acc, fa);
  interleave();
  mma(acc, fb);
}

__device__ __forceinline__ bool give_up(gu32* tmo, unsigned call, u64 t0) {
  const unsigned v = __builtin_amdgcn_readfirstlane(__hip_atomic_load(tmo, __ATOMIC_RELAXED, __HIP_MEMORY_SCOPE_AGENT));
  if (v == call || now() - t0 > DEADLINE) {
    __hip_atomic_store(tmo, call, __ATOMIC_RELAXED, __HIP_MEMORY_SCOPE_AGENT);
    return true;
  }
  return false;
}

__device__ __forceinline__ void poll_flags(const char* f, const char* g, unsigned call, gu32* tmo) {
  const u64 t0 = now();
  for (unsigned it = 1;; ++it) {
    unsigned vf = 0, vg = 0;
    if (f) vf = __hip_atomic_load((gu32*)f, __ATOMIC_RELAXED, __HIP_MEMORY_SCOPE_SYSTEM);
    if (g) vg = __hip_atomic_load((gu32*)g, __ATOMIC_RELAXED, __HIP_MEMORY_SCOPE_SYSTEM);
    if (vf >= call) f = nullptr;
    if (vg >= call) g = nullptr;
    if (__all(f == nullptr && g == nullptr)) break;
    if ((it & 63u) == 0 && give_up(tmo, call, t0)) break;
    __builtin_amdgcn_s_sleep(PSLEEP);
  }
}

__device__ __forceinline__ unsigned pack_bf16(float a, float b) {
  f32x2 v = {a, b};
  return __builtin_bit_cast(unsigned, __builtin_convertvector(v, bf16x2));
}

__device__ __forceinline__ void add_partial(f32x4 (&acc)[4][4], const u32x4 (&v)[8]) {
#pragma unroll
  for (int i = 0; i < 8; ++i) {
#pragma unroll
    for (int h = 0; h < 2; ++h) {
      f32x4& c = acc[i >> 1][(i & 1) * 2 + h];
      const unsigned u0 = v[i][2 * h], u1 = v[i][2 * h + 1];
      c[0] += __uint_as_float(u0 << 16);
      c[1] += __uint_as_float(u0 & 0xffff0000u);
      c[2] += __uint_as_float(u1 << 16);
      c[3] += __uint_as_float(u1 & 0xffff0000u);
    }
  }
}

__device__ __forceinline__ unsigned wave_max(unsigned m) {
  m = max(m, (unsigned)__builtin_amdgcn_update_dpp(0, (int)m, 0x111, 0xf, 0xf, false));
  m = max(m, (unsigned)__builtin_amdgcn_update_dpp(0, (int)m, 0x112, 0xf, 0xf, false));
  m = max(m, (unsigned)__builtin_amdgcn_update_dpp(0, (int)m, 0x114, 0xf, 0xf, false));
  m = max(m, (unsigned)__builtin_amdgcn_update_dpp(0, (int)m, 0x118, 0xf, 0xf, false));
  return max(max((unsigned)__builtin_amdgcn_readlane((int)m, 15), (unsigned)__builtin_amdgcn_readlane((int)m, 31)),
             max((unsigned)__builtin_amdgcn_readlane((int)m, 47), (unsigned)__builtin_amdgcn_readlane((int)m, 63)));
}

__global__ void __launch_bounds__(NT, 2) fused_gemm_rs_q8(const Params p) {
  __shared__ __attribute__((aligned(1024))) char smem[LDS_BYTES];
  lchar* sm = (lchar*)smem;
  typedef __attribute__((address_space(3))) unsigned lu32;
  const u64 tstart = now();
  const unsigned t = threadIdx.x, lane = t & 63;
  const unsigned wv = __builtin_amdgcn_readfirstlane(t >> 6);
  const unsigned d = blockIdx.x & 7u, l = blockIdx.x >> 3;
  u64 a_job = ((const u64*)p.job)[blockIdx.x];
  char* a_peer = p.wsx[d == 0 ? 1u : d];
  char* a_me = p.wsx[0];
  const char* a_x = p.x;
  const char* a_w = p.w;
  signed char* a_out = p.out;
  unsigned a_rank = (unsigned)p.rank;
  asm volatile("" : "+s"(a_job), "+s"(a_peer), "+s"(a_me), "+s"(a_x), "+s"(a_w), "+s"(a_out), "+s"(a_rank));
  typedef __attribute__((address_space(1))) char gchar;
  char* const me = (char*)(gchar*)a_me;
  char* const peer = (char*)(gchar*)a_peer;
  const char* const px = (const char*)(const gchar*)a_x;
  const char* const pw = (const char*)(const gchar*)a_w;
  const Job job = __builtin_bit_cast(Job, a_job);
  const unsigned r = a_rank & 7u;
  const unsigned q = r ^ d;
  gu32* const ctr = (gu32*)(me + WS_CTR + blockIdx.x * 128);
  gu32* const tmo = (gu32*)(me + WS_TMO);
  unsigned prev = __hip_atomic_load(ctr, __ATOMIC_RELAXED, __HIP_MEMORY_SCOPE_AGENT);
  const unsigned j = r >> 2, xs = job.tile & 15u, n = job.tile >> 4;
  const unsigned fa = job.fin & 3u, fn = job.fin >> 2;
  const bool ships = d < 2u || d >= 4u;
  const bool detour = d < 2u && job.turn != 255u;
  const unsigned allsrc = d >= 4u ? 63u & ~(1u << (d - 2u)) : 63u;
  const unsigned ln0 = t * 16u;

  lu32* const red = (lu32*)(sm + LDS_RED);

  auto callno = [&]() -> unsigned {
    unsigned pv = prev;
    asm volatile("" : "+v"(pv));
    return __builtin_amdgcn_readfirstlane(pv) + 1u;
  };
  auto push = [&](unsigned k0, unsigned k1, unsigned kstep) {
      const unsigned u = l & 7u;
      const unsigned cq = (q ^ (d >= 4u ? (r & (q >> 1)) : 0u)) & 1u, jq = q >> 2;
      for (unsigned k = k0; k < k1; k += kstep) {
        unsigned it = ord_item(k, d), src = d - 2u;
        const unsigned sl = (u + ROT * k) & 7u, slotk = k;
        u32x4 v[8];
        if (it < 16u) {
          const unsigned xa = nat_x(lane), xb = nat_x(lane + 64u);
          const __amdgpu_buffer_rsrc_t rs = __builtin_amdgcn_make_buffer_rsrc((void*)px, 0, 4u << 20, 0x00020000);
          const unsigned off = (rowbase(q, it) + (sl & 1u) * 64u) * 1024u + cq * 512u + (sl >> 1) * 128u;
#pragma unroll
          for (unsigned i = 0; i < 8; ++i) v[i] = __builtin_amdgcn_raw_buffer_load_b128(rs, (i & 1u) ? xb : xa, off + (i >> 1) * 16384u, 0);
        } else if (NF && k1 > 16u && it >= F_) {
          const unsigned ds = 4u | ((it & 1u) << 1) | ((r ^ d) & 1u);
          it -= NF;
          src = (ds ^ d) - 2u;
          wait_until(tstart + T0 + (slotk - 2u) * TAU);
          poll_flags(lane < 8u ? me + WS_FIN + ((ds - 2u) * 32u + it) * 128u + lane * 4u : nullptr, nullptr, callno(), tmo);
          const __amdgpu_buffer_rsrc_t rs = __builtin_amdgcn_make_buffer_rsrc(me + WS_IN + (size_t)((ds - 2u) * SRC_STRIDE + it * ITEM + sl * 8192u), 0, 8192u, 0x00020000);
          asm volatile("" ::: "memory");
#pragma unroll
          for (unsigned i = 0; i < 8; ++i) v[i] = __builtin_amdgcn_raw_buffer_load_b128(rs, lane * 16u, i * 1024u, AUX_SYS);
        } else {
          const unsigned wa = nat_w(lane), wb = nat_w(lane + 128u);
          const __amdgpu_buffer_rsrc_t rs = __builtin_amdgcn_make_buffer_rsrc((void*)pw, 0, 4u << 20, 0x00020000);
          const unsigned off = (cq * 256u + sl * 32u) * 8192u + (jq * 2048u + (it - 16u) * 128u) * 2u;
#pragma unroll
          for (unsigned i = 0; i < 8; ++i) v[i] = __builtin_amdgcn_raw_buffer_load_b128(rs, (i & 2u) ? wb : wa, off + (i & 5u) * 32768u, 0);
        }
        char* dst = peer + WS_IN + (size_t)(src * SRC_STRIDE + it * ITEM + sl * 8192u);
        const __amdgpu_buffer_rsrc_t rd = __builtin_amdgcn_make_buffer_rsrc(dst, 0, 8192u, 0x00020000);
        wait_until(tstart + T0 + slotk * TAU + u * STAG);
#pragma unroll
        for (unsigned i = 0; i < 8; ++i) __builtin_amdgcn_raw_buffer_store_b128(v[i], rd, lane * 16u, i * 1024u, AUX_SYS);
        VM0();
        const unsigned cn = callno();
        if (lane == 0)
          __hip_atomic_store((gu32*)(peer + WS_FIN + (src * 32u + it) * 128u + u * 4u), cn, __ATOMIC_RELAXED, __HIP_MEMORY_SCOPE_SYSTEM);
      }
      STAMPW(1);
  };
  if (t < 16u) red[t] = 0u;
  if (PRETOUCH && wv == 3u) {
    const unsigned po = (r * 256u + blockIdx.x) * 128u;
    const char* a = nullptr;
    if (d >= 2u && l < 8u && (PRETOUCH & 1)) {
      if (lane == 0u) a = peer + WS_SPARE + po;
      else if (lane < 3u) a = peer + WS_IN - 65536u + (size_t)((d - 2u + lane - 1u) * SRC_STRIDE + (po & 65535u));
    }
    if (d < 2u && (PRETOUCH & 2) && lane >= 8u && lane < 15u) a = me + WS_IN - 65536u + (size_t)((lane - 8u) * SRC_STRIDE + (po & 65535u));
    if (d < 4u && (PRETOUCH & 4) && lane == 16u) a = (const char*)a_out + (128u * fa * 4096u + fn * 128u);
    if (a) {
      const unsigned v = __hip_atomic_load((gu32*)a, __ATOMIC_RELAXED, __HIP_MEMORY_SCOPE_SYSTEM);
      asm volatile("" ::"v"(v));
    }
  }
  if (d >= 2u && l < 8u) push(wv, 16u, 8u);

  if (wv >= 4) {
    if ((job.plan & 15u) == 3u || (d >= 2u && l >= 8u)) wait_until(tstart + DELAY);
    for (unsigned plan = job.plan | 0xf0000u; (plan & 15u) != 15u; plan >>= 4) {
      const unsigned kind = plan & 15u;
      if (kind == 5u || kind == 6u) { push((kind == 5u ? 12u : NIN - 8u) + wv, d >= 4u ? NCR : kind == 5u ? NIN - 4u : NIN, 4u); continue; }
      const unsigned call = callno();
      const bool staged = kind == 1u || kind >= 7u;
      const unsigned mask = kind == 7u ? job.early : kind == 8u ? allsrc & ~job.early : allsrc;
      const int s0 = staged ? 8 : 0, s1 = staged ? 8 + 4 * __builtin_popcount(mask) : 8;
      const bool last = staged && kind != 7u && ships;
      if (staged && wv == 4) {
        const unsigned s = lane >> 3;
        const bool act = (mask >> s) & 1u;
        wait_until(tstart + T0 + (kind == 7u ? job.pos2 : job.pos) * TAU);
        poll_flags(act ? me + WS_FIN + (s * 32u + xs) * 128u + (lane & 7u) * 4u : nullptr,
                   act ? me + WS_FIN + (s * 32u + 16u + n) * 128u + (lane & 7u) * 4u : nullptr, call, tmo);
        STAMPW(12);
        if (ACQ) {
          __builtin_amdgcn_fence(__ATOMIC_ACQUIRE, "");
          VM0();
        }
      }
      unsigned srcs = 0;
      for (unsigned s = 6u; s-- > 0u;) if ((mask >> s) & 1u) srcs = (srcs << 4) | s;
      const u64 mine = gemm_load((unsigned)(size_t)sm, wv - 4u, lane,
                px + (size_t)(kind == 3u ? 512u * r + 128u * fa : rowbase(r, xs)) * 1024u,
                pw + (kind == 3u ? (j ^ 1u) * 2048u + fn * 128u : j * 2048u + n * 128u) * 2u, me + WS_IN + (size_t)(xs * ITEM),
                me + WS_IN + (size_t)((16u + n) * ITEM), s0, s1, srcs,
                last && d < 2u && !detour && t == 256u ? (gu64*)(me + WS_TICK) : nullptr);
      if (last) {
        if (t == 256u) red[12] = (unsigned)(mine - tstart);
        BAR();
        u32x4 v[8];
#pragma unroll
        for (int i = 0; i < 8; ++i) v[i] = *(__attribute__((address_space(3))) u32x4*)(sm + (wv - 4u) * 8192u + i * 1024u + lane * 16u);
        int go = (int)__builtin_amdgcn_readfirstlane(red[12]);
        if (go > (int)(now() - tstart) + GATE + 64 * PTAU) go = 0;
        if (d >= 4u) go = GATE + (int)(job.turn ^ ((r & 1u) * TURNX)) * PTAU;
        if (detour) go = RGATE + (int)job.turn * PTAU;
        const unsigned a = d < 2u ? xs : 2u * ((r >> 1) & 1u) + (xs & 1u);
        const unsigned slot = ((d < 2u ? (detour ? 3u : 0u) : 1u + (r & 1u)) * 4u + a) * 16u + n;
        char* const to = detour ? (char*)(gchar*)p.wsx[2u + d] : peer;
        const __amdgpu_buffer_rsrc_t rs = __builtin_amdgcn_make_buffer_rsrc(to + WS_P + (size_t)slot * PTILE + (wv - 4u) * 8192u, 0, 8192u, 0x00020000);
        if (go > 0) wait_until(tstart + (unsigned)go + (wv - 4u) * PSTAG);
        STAMPW(4);
#pragma unroll
        for (int i = 0; i < 8; ++i) __builtin_amdgcn_raw_buffer_store_b128(v[i], rs, lane * 16u, i * 1024u, AUX_SYS);
        VM0();
        if (lane == 0) __hip_atomic_store((gu32*)(to + WS_FP + slot * 128u + (wv - 4u) * 4u), call, __ATOMIC_RELAXED, __HIP_MEMORY_SCOPE_SYSTEM);
        STAMP7(6);
      }
    }
    if (d < 4u) {
      const unsigned call = callno();
      if (!ships && job.turn != 255u) {
        const unsigned slot = (job.turn & 3u) * 16u + ((job.turn >> 2) & 15u);
        poll_flags(lane == 0 ? me + WS_FP + (192u + slot) * 128u + (wv - 4u) * 4u : nullptr, nullptr, call, tmo);
        const __amdgpu_buffer_rsrc_t ri = __builtin_amdgcn_make_buffer_rsrc(me + WS_P + (size_t)(192u + slot) * PTILE + (wv - 4u) * 8192u, 0, 8192u, 0x00020000);
        const __amdgpu_buffer_rsrc_t rs = __builtin_amdgcn_make_buffer_rsrc(peer + WS_P + (size_t)slot * PTILE + (wv - 4u) * 8192u, 0, 8192u, 0x00020000);
        u32x4 v[8];
        asm volatile("" ::: "memory");
#pragma unroll
        for (int i = 0; i < 8; ++i) v[i] = __builtin_amdgcn_raw_buffer_load_b128(ri, lane * 16u, i * 1024u, AUX_SYS);
        wait_until(tstart + RGATE + (RELAY_R1 + 1u + (l & 7u)) * PTAU + (wv - 4u) * PSTAG);
        STAMPW(4);
#pragma unroll
        for (int i = 0; i < 8; ++i) __builtin_amdgcn_raw_buffer_store_b128(v[i], rs, lane * 16u, i * 1024u, AUX_SYS);
        VM0();
        if (lane == 0) __hip_atomic_store((gu32*)(peer + WS_FP + slot * 128u + (wv - 4u) * 4u), call, __ATOMIC_RELAXED, __HIP_MEMORY_SCOPE_SYSTEM);
        STAMP7(6);
      }
      const __amdgpu_buffer_rsrc_t rg = __builtin_amdgcn_make_buffer_rsrc(me + WS_G, 0, 8192, 0x00020000);
      u32x4 g = {0u, 0u, 0u, 0u}, h = g;
      wait_until(tstart + SWEEP_T);
      const u64 t0 = now();
      unsigned state = 1u;
      for (;;) {
        if (!(g[1] == call && g[3] == call)) g = __builtin_amdgcn_raw_buffer_load_b128(rg, ln0 - 4096u, 0, AUX_SYS);
        if (!(h[1] == call && h[3] == call)) h = __builtin_amdgcn_raw_buffer_load_b128(rg, ln0, 0, AUX_SYS);
        if (give_up(tmo, call, t0)) { state = 2u; break; }
        const unsigned m = wave_max(max(max(g[1] == call ? g[0] : 0u, g[3] == call ? g[2] : 0u),
                                        max(h[1] == call ? h[0] : 0u, h[3] == call ? h[2] : 0u)));
        if (lane == 0) __hip_atomic_store(red + wv, m, __ATOMIC_RELAXED, __HIP_MEMORY_SCOPE_WORKGROUP);
        if (__all(g[1] == call && g[3] == call && h[1] == call && h[3] == call)) break;
        __builtin_amdgcn_s_sleep(GSLEEP);
        asm volatile("" ::: "memory");
      }
      if (lane == 0) __hip_atomic_store(red + 4u + wv, state, __ATOMIC_RELEASE, __HIP_MEMORY_SCOPE_WORKGROUP);
      STAMPW(13);
    }
  } else {
    f32x4 acc[4][4];
#pragma unroll
    for (int i = 0; i < 4; ++i)
#pragma unroll
      for (int k = 0; k < 4; ++k) acc[i][k] = f32x4{0.f, 0.f, 0.f, 0.f};
    char* gdst = p.wsx[lane & 7u] + WS_G + (r * 128u + d * 32u + l) * 8u;
    asm volatile("" : "+v"(gdst));
    const bool parks = (job.plan & 15u) == 3u;
    const __amdgpu_buffer_rsrc_t rpark = __builtin_amdgcn_make_buffer_rsrc(me + WS_PARK + (size_t)((d & 1u) * 32u + l) * 65536u, 0, 65536u, 0x00020000);
    for (unsigned plan = job.plan | 0xf0000u; (plan & 15u) != 15u; plan >>= 4) {
      const unsigned kind = plan & 15u;
      if (kind == 5u || kind == 6u) continue;
      const bool staged = kind == 1u || kind >= 7u;
      const unsigned mask = kind == 7u ? job.early : kind == 8u ? allsrc & ~job.early : allsrc;
      const int s0 = staged ? 8 : 0, s1 = staged ? 8 + 4 * __builtin_popcount(mask) : 8;
      const bool last = staged && kind != 7u && ships;
      if (staged) STAMP(3);
      if (!staged) {
#pragma unroll
        for (int i = 0; i < 4; ++i)
#pragma unroll
          for (int k = 0; k < 4; ++k) acc[i][k] = f32x4{0.f, 0.f, 0.f, 0.f};
      }
      gemm_math(acc, sm, wv, lane, s0, s1);
      if (kind == 3u && parks) {
#pragma unroll
        for (int i = 0; i < 16; ++i)
          __builtin_amdgcn_raw_buffer_store_b128(__builtin_bit_cast(u32x4, acc[i >> 2][i & 3]), rpark, ln0, i * 4096u, 0);
      }
      if (kind == 0u) STAMP(2);
      if (staged) STAMP(5);
      if (last) {
#pragma unroll
        for (int i = 0; i < 8; ++i) {
          const f32x4 c0 = acc[i >> 1][(i & 1) * 2], c1 = acc[i >> 1][(i & 1) * 2 + 1];
          *(__attribute__((address_space(3))) u32x4*)(sm + wv * 8192u + i * 1024u + lane * 16u) =
              u32x4{pack_bf16(c0[0], c0[1]), pack_bf16(c0[2], c0[3]), pack_bf16(c1[0], c1[1]), pack_bf16(c1[2], c1[3])};
        }
        BAR();
        if (parks) {
#pragma unroll
          for (int i = 0; i < 16; ++i)
            acc[i >> 2][i & 3] = __builtin_bit_cast(f32x4, __builtin_amdgcn_raw_buffer_load_b128(rpark, ln0, i * 4096u, 0));
        }
      }
    }
    const unsigned call = callno();
    if (d < 4u) {
      const unsigned slot = ((d < 2u ? 4u : 0u) + fa) * 16u + fn;
      STAMP3(11);
      unsigned pend = d < 2u ? 3u : 1u;
      do {
        unsigned got;
        const u64 t0 = now();
        for (unsigned it = 1;; ++it) {
          unsigned f = 0;
          if (lane < 2u && ((pend >> lane) & 1u))
            f = __hip_atomic_load((gu32*)(me + WS_FP + (slot + lane * 64u) * 128u + wv * 4u), __ATOMIC_RELAXED, __HIP_MEMORY_SCOPE_SYSTEM);
          got = (unsigned)__ballot(f >= call) & pend;
          if (got) break;
          if ((it & 63u) == 0 && give_up(tmo, call, t0)) { got = pend; break; }
          __builtin_amdgcn_s_sleep(PSLEEP);
        }
        got = __builtin_amdgcn_readfirstlane(got);
        STAMP3(7);
        char* const pb = me + WS_P + (size_t)slot * PTILE + wv * 8192u;
        const __amdgpu_buffer_rsrc_t r0 = __builtin_amdgcn_make_buffer_rsrc(pb, 0, got & 1u ? 8192u : 0u, 0x00020000);
        const __amdgpu_buffer_rsrc_t r1 = __builtin_amdgcn_make_buffer_rsrc(pb + 64u * PTILE, 0, got & 2u ? 8192u : 0u, 0x00020000);
        u32x4 v[8], w[8];
        asm volatile("" ::: "memory");
#pragma unroll
        for (int i = 0; i < 8; ++i) v[i] = __builtin_amdgcn_raw_buffer_load_b128(r0, lane * 16u, i * 1024u, AUX_SYS);
#pragma unroll
        for (int i = 0; i < 8; ++i) w[i] = __builtin_amdgcn_raw_buffer_load_b128(r1, lane * 16u, i * 1024u, AUX_SYS);
        add_partial(acc, v);
        add_partial(acc, w);
        pend &= ~got;
      } while (pend);
      float mf = 0.f;
#pragma unroll
      for (int i = 0; i < 4; ++i)
#pragma unroll
        for (int k = 0; k < 4; ++k)
#pragma unroll
          for (int e = 0; e < 4; ++e) mf = fmaxf(mf, fabsf(acc[i][k][e]));
      const unsigned mw = wave_max(__float_as_uint(mf));
      unsigned arr = 0;
      if (lane == 0) {
        __hip_atomic_fetch_max(red, mw, __ATOMIC_RELAXED, __HIP_MEMORY_SCOPE_WORKGROUP);
        arr = __hip_atomic_fetch_add(red + 1, 1u, __ATOMIC_ACQ_REL, __HIP_MEMORY_SCOPE_WORKGROUP);
      }
      if (__builtin_amdgcn_readfirstlane(arr) == 3u && lane < NR) {
        const u64 g = ((u64)call << 32) | __hip_atomic_load(red, __ATOMIC_RELAXED, __HIP_MEMORY_SCOPE_WORKGROUP);
        __hip_atomic_store((gu64*)gdst, g, __ATOMIC_RELAXED, __HIP_MEMORY_SCOPE_SYSTEM);
      }
      STAMP3(8);
      const unsigned jc = d < 2u ? (j ^ 1u) : j;
      const unsigned pp = lane >> 4;
      const __amdgpu_buffer_rsrc_t ro = __builtin_amdgcn_make_buffer_rsrc(a_out, 0, 512u * 4096u, 0x00020000);
      const unsigned oo = (128u * fa + 64u * (wv >> 1) + (lane & 15u)) * 4096u + jc * 2048u + fn * 128u + 64u * (wv & 1u) + 16u * pp;
      unsigned used = 0xffffffffu;
      for (;;) {
        const unsigned s0 = __hip_atomic_load(red + 8, __ATOMIC_ACQUIRE, __HIP_MEMORY_SCOPE_WORKGROUP), s1 = __hip_atomic_load(red + 9, __ATOMIC_ACQUIRE, __HIP_MEMORY_SCOPE_WORKGROUP),
                       s2 = __hip_atomic_load(red + 10, __ATOMIC_ACQUIRE, __HIP_MEMORY_SCOPE_WORKGROUP), s3 = __hip_atomic_load(red + 11, __ATOMIC_ACQUIRE, __HIP_MEMORY_SCOPE_WORKGROUP);
        const unsigned m0 = __hip_atomic_load(red + 4, __ATOMIC_RELAXED, __HIP_MEMORY_SCOPE_WORKGROUP), m1 = __hip_atomic_load(red + 5, __ATOMIC_RELAXED, __HIP_MEMORY_SCOPE_WORKGROUP),
                       m2 = __hip_atomic_load(red + 6, __ATOMIC_RELAXED, __HIP_MEMORY_SCOPE_WORKGROUP), m3 = __hip_atomic_load(red + 7, __ATOMIC_RELAXED, __HIP_MEMORY_SCOPE_WORKGROUP);
        const bool fin = __builtin_amdgcn_readfirstlane(min(min(s0, s1), min(s2, s3))) != 0u;
        const bool bad = __builtin_amdgcn_readfirstlane(max(max(s0, s1), max(s2, s3))) == 2u;
        unsigned cur = __builtin_amdgcn_readfirstlane(max(max(m0, m1), max(m2, m3)));
        const bool seen = __builtin_amdgcn_readfirstlane(min(min(m0, m1), min(m2, m3))) != 0u;
        if (bad) cur = 0xfffffffeu;
        if (cur != used && (fin || (SPEC && seen && cur >= mw))) {
          const float amax = __uint_as_float(cur);
          const float inv = amax > 0.f ? 127.0f / amax : 0.f;
          const float magic = 12582912.0f;
          VM0();
#pragma unroll
          for (int mb = 0; mb < 4; ++mb) {
            unsigned dw[4];
#pragma unroll
            for (int nb = 0; nb < 4; ++nb) {
              unsigned b[4];
#pragma unroll
              for (int e = 0; e < 4; ++e)
                b[e] = __float_as_uint(__builtin_amdgcn_fmed3f(__builtin_fmaf(acc[mb][nb][e], inv, magic), magic - 127.f, magic + 127.f));
              const unsigned lo = __builtin_amdgcn_perm(b[1], b[0], 0x0c0c0400u), hi = __builtin_amdgcn_perm(b[3], b[2], 0x04000c0cu);
              dw[nb] = bad ? 0x80808080u : (lo | hi);
            }
            const bool odd = pp & 1u;
            const u32x4 o = {odd ? dw[1] : dw[0], odd ? dw[0] : dw[1], odd ? dw[3] : dw[2], odd ? dw[2] : dw[3]};
            __builtin_amdgcn_raw_buffer_store_b128(o, ro, oo + mb * 65536u, 0, OUT_AUX);
          }
          used = cur;
          STAMP3(9);
        }
        if (fin) break;
        __builtin_amdgcn_s_sleep(1);
      }
      STAMP(10);
    }
  if (t == 0) __hip_atomic_store(ctr, call, __ATOMIC_RELAXED, __HIP_MEMORY_SCOPE_AGENT);
  }
}

void make_jobs(Job* job) {
  int slot[6][32];
  for (int s = 0; s < 6; ++s) {
    for (int it = 0; it < 32; ++it) slot[s][it] = 0;
    for (int k = 0; k < (s < 2 ? NIN : NCR); ++k)
      if (ord_item(k, s + 2) < F_) slot[s][ord_item(k, s + 2)] = k;
  }
  for (int k = 0; k < NIN; ++k) {
    const int e = (int)ord_item(k, 2) - F_;
    for (int s = 2; s < 6 && e >= 0; ++s)
      if ((((s + 2) >> 1) & 1) != (e & 1)) slot[s][32 - NF + e] = k;
  }
  for (int d = 0; d < 8; ++d) {
    Job jb[32];
    int key[32], key2[32], early[32], srt[32], done[32], idx[32], txs[32], tn[32];
    for (int m = 0; m < 32; ++m) {
      txs[m] = d < 4 ? (d < 2 ? 0 : 4) + (m & 3) : 8 + 2 * (d - 4) + (m >> 4);
      tn[m] = d < 4 ? 2 * (m >> 2) + (d & 1) : (m & 15);
      int at[6], lastk = 0, cnt = 0;
      for (int s = 0; s < 6; ++s) {
        at[s] = slot[s][txs[m]] > slot[s][16 + tn[m]] ? slot[s][txs[m]] : slot[s][16 + tn[m]];
        if (d >= 4 && s == d - 2) at[s] = -1;
        if (at[s] > lastk) lastk = at[s];
        cnt += at[s] >= 0;
      }
      int best = lastk * TAU + RESTART + cnt * SRC_T - SPLIT_GAIN;
      early[m] = 0;
      key2[m] = lastk;
      for (int c = 0; c < lastk; ++c) {
        int mk = 0, na = 0, ka = 0;
        for (int s = 0; s < 6; ++s)
          if (at[s] >= 0 && at[s] <= c) { mk |= 1 << s; ++na; if (at[s] > ka) ka = at[s]; }
        if (!na) continue;
        const int fa = ka * TAU + RESTART + na * SRC_T;
        const int fb = (fa > lastk * TAU ? fa : lastk * TAU) + RESTART + (cnt - na) * SRC_T;
        if (fb < best) { best = fb; early[m] = mk; key2[m] = ka; }
      }
      key[m] = lastk * 64 + m;
      srt[m] = (d == 2 || d == 3 ? key2[m] * TAU : best) * 64 + m;
      done[m] = best * 64 + m;
      idx[m] = m;
    }
    for (int a = 0; a < 32; ++a)
      for (int b = a + 1; b < 32; ++b)
        if (srt[idx[b]] < srt[idx[a]]) { const int tmp = idx[a]; idx[a] = idx[b]; idx[b] = tmp; }
    for (int a = 8; a < 24; ++a)
      for (int b = a + 1; b < 24; ++b)
        if (done[idx[b]] < done[idx[a]]) { const int tmp = idx[a]; idx[a] = idx[b]; idx[b] = tmp; }
    for (int l = 0; l < 32; ++l) {
      const int m = idx[(l + 16) & 31];
      jb[l].tile = (unsigned char)(txs[m] | (tn[m] << 4));
      jb[l].early = (unsigned char)early[m];
      jb[l].pos = (unsigned char)(key[m] / 64);
      const int fm = d < 2 ? idx[31 - ((l + 16) & 31)] : m;
      jb[l].fin = (unsigned char)((txs[fm] & 3) | (tn[fm] << 2));
      const bool sp = early[m] != 0;
      if (d >= 2 && l >= 8 && l < 16) jb[l].plan = sp ? 0x8750 : 0xf150;
      else if (d >= 2 && d < 4 && l >= 16 && l < 24) jb[l].plan = sp ? 0x8670 : 0xf160;
      else if (d >= 2) jb[l].plan = sp ? 0xf870 : 0xff10;
      else if (tn[m] < EARLY_N) jb[l].plan = sp ? 0x3870 : 0xf310;
      else jb[l].plan = sp ? 0x8703 : 0xf103;
      jb[l].pos2 = (unsigned char)key2[m];
      jb[l].turn = (unsigned char)((l + 16) & 31);
      if (d < 2) {
        const int rc = (tn[m] - RELAY_N) / 2;
        jb[l].turn = (unsigned char)(tn[m] >= RELAY_N && rc * 4 + (txs[m] & 3) < RELAY_R1 ? rc * 4 + (txs[m] & 3) : 255);
      } else if (d < 4) {
        const int i = l - 24;
        jb[l].turn = (unsigned char)(i >= 0 && i < RELAY_R1 ? (i & 3) | ((RELAY_N + 2 * (i >> 2) + (3 - d)) << 2) : 255);
      }
      job[l * 8 + d] = jb[l];
    }
  }
}

}

extern "C" void kernel_launch(int rank, int nranks, ncclComm_t comm, hipStream_t stream, void* const* d_in,
                              const int* in_sizes, int n_in, void* d_out, int out_size, const int* in_dtypes,
                              int out_dtype, void* const* d_out_all, void* const* d_ws_all, size_t ws_bytes) {
  if (nranks != NR || (unsigned)rank >= (unsigned)NR || n_in < 2 || ws_bytes < WS_END || out_size != 512 * 4096 ||
      in_sizes[0] != 4096 * 512 || in_sizes[1] != 512 * 4096) {
    fprintf(stderr, "kernel_launch: unexpected problem shape\n");
    return;
  }
  Params p;
  memset(&p, 0, sizeof p);
  p.x = (const char*)d_in[0];
  p.w = (const char*)d_in[1];
  p.out = (signed char*)d_out;
  for (int i = 0; i < NR; ++i) p.wsx[i] = (char*)d_ws_all[rank ^ i];
  p.rank = rank;
  make_jobs(p.job);
  hipLaunchKernelGGL(fused_gemm_rs_q8, dim3(NWG), dim3(NT), 0, stream, p);
  HIP_CHECK(hipGetLastError());
}
```

```cpp
#include <hip/hip_runtime.h>
#include <rccl/rccl.h>
#include <cstdio>
#include <cstdint>
#include <cstring>

#define HIP_CHECK(expr) do { hipError_t _e = (expr); if (_e != hipSuccess) { \
    fprintf(stderr, "HIP error %s at %s:%d\n", hipGetErrorString(_e), __FILE__, __LINE__); \
    return; } } while (0)

namespace {

typedef short s16x4 __attribute__((ext_vector_type(4)));
typedef short s16x8 __attribute__((ext_vector_type(8)));
typedef __bf16 bf16x8 __attribute__((ext_vector_type(8)));
typedef __bf16 bf16x2 __attribute__((ext_vector_type(2)));
typedef float f32x4 __attribute__((ext_vector_type(4)));
typedef float f32x2 __attribute__((ext_vector_type(2)));
typedef unsigned u32x4 __attribute__((ext_vector_type(4)));
typedef __attribute__((address_space(1))) unsigned gu32;
typedef __attribute__((address_space(1))) unsigned long long gu64;
typedef __attribute__((address_space(3))) char lchar;
typedef unsigned long long u64;

constexpr int NR = 8;
constexpr int NWG = 256, NT = 512;
constexpr unsigned ITEM = 65536;
constexpr unsigned PTILE = 32768;
constexpr unsigned STAGE = 32768;
#define NSTAGES 4
constexpr int NST = NSTAGES;
static_assert(NST == 4, "the tile hand-over assumes step counts that are multiples of the ring size");

constexpr size_t WS_CTR = 0;
constexpr size_t WS_TMO = 32 << 10;
constexpr size_t WS_TICK = 33 << 10;
constexpr size_t WS_FIN = 64 << 10;
constexpr size_t WS_FP = 96 << 10;
constexpr size_t WS_G = 128 << 10;
constexpr size_t WS_GL = 136 << 10;
constexpr size_t WS_STAMP = 256 << 10;
constexpr size_t WS_SPARE = 512 << 10;
constexpr size_t WS_IN = (2 << 20) + 65536;
constexpr unsigned SRC_STRIDE = (2u << 20) + 65536u;
constexpr size_t WS_P = 16 << 20;
constexpr size_t WS_PARK = 24 << 20;
constexpr size_t WS_END = 28 << 20;
static_assert(WS_IN + 6 * (size_t)SRC_STRIDE <= WS_P, "workspace layout");

constexpr u64 DEADLINE = 25000000ull;
constexpr int AUX_SYS = 17;

#define TAU 113
#define T0 160
#define STAG (TAU / 8)
#define ROT 1
#define PTAU 57
#define GATE (T0 + NCR * TAU)
#define OUT_AUX 17
#define PSLEEP 4
#define GSLEEP 2
#define SWEEP_T (GATE + 32 * PTAU + SWEEP_D)
#define SWEEP_D 150
#define ACQ 0
#define DRAW_LEAD 110
#define TURNX 3u
#define RELAY_N 12
#define RELAY_R1 4
#define RGATE (T0 + NIN * TAU)
#define PRETOUCH 7
#define RESTART 100
#define SRC_T 124
#define SPLIT_GAIN 100
#define PAIRS 1
#define COLLECT_LEAD 400
#define COLLECTOR (16 * 8 + 4)
#define SWAP 1
#define RELAY_GAP 0u
#define EARLY_TAKE 1
#define TAKE_D 150
#define SETC 32u
#define XPERM 0x76543210u
#define SPEC 1
#define PSTAG (PTAU / 4)
#define DELAY 300
#define EARLY_N 4

#define NF 4
constexpr unsigned char W_ = 16, F_ = 32, G_ = 40;
constexpr int NIN = 32 + NF, NCR = 30 - NF / 2;
#define ORD_HEAD 0, W_ + 0, 1, W_ + 1, 2, 3, W_ + 2, W_ + 3, 8, W_ + 4, 10, 12, W_ + 5, 14, W_ + 6, 9, W_ + 7, 11, W_ + 8, 13, W_ + 9, 15
#define ORD_W W_ + 10, W_ + 11, W_ + 12, W_ + 13, W_ + 14, W_ + 15
constexpr unsigned char OIN[40] = {ORD_HEAD, ORD_W, F_ + 0, F_ + 1, F_ + 2, F_ + 3, 4, 5, 6, 7};
constexpr unsigned char OCR[40] = {ORD_HEAD, G_ + 0, G_ + 1, W_ + 10, W_ + 11, 6, 7};
constexpr u64 ord_pack(const unsigned char* o) {
  u64 v = 0;
  for (int i = 0; i < 8; ++i) v |= (u64)o[i] << (8 * i);
  return v;
}
static_assert(ord_pack(OIN) == ord_pack(OCR) && ord_pack(OIN + 8) == ord_pack(OCR + 8), "common head");
__host__ __device__ __forceinline__ unsigned ord_item(unsigned k, unsigned d) {
  const u64 hi = d < 4u ? ((k & 32u) ? ord_pack(OIN + 32) : (k & 8u) ? ord_pack(OIN + 24) : ord_pack(OIN + 16))
                        : ((k & 8u) ? ord_pack(OCR + 24) : ord_pack(OCR + 16));
  const u64 a = k >= 16u ? hi : (k & 8u) ? ord_pack(OIN + 8) : ord_pack(OIN);
  unsigned it = (unsigned)(a >> (8u * (k & 7u))) & 0xffu;
  if (d >= 4u && (it >> 1) == d) it = 4u + (it & 1u);
  if (it >= G_) it = 32u - NF + 2u * (it - G_) + ((d >> 1) & 1u);
  return it;
}

struct alignas(8) Job {
  unsigned char tile;
  unsigned char early;
  unsigned char pos;
  unsigned char fin;
  unsigned char turn;
  unsigned char pos2;
  unsigned short plan;
};

struct Params {
  const char* x;
  const char* w;
  signed char* out;
  char* wsx[NR];
  int rank;
  int pad;
  Job job[NR * 32];
};

__device__ __forceinline__ u64 now() { return __builtin_amdgcn_s_memrealtime(); }
__device__ __forceinline__ void wait_until(u64 t) { while (now() < t) __builtin_amdgcn_s_sleep(1); }
#define VM0() asm volatile("s_waitcnt vmcnt(0)" ::: "memory")
#define BAR() do { asm volatile("" ::: "memory"); __builtin_amdgcn_s_waitcnt(0xC07F);   __builtin_amdgcn_s_barrier(); asm volatile("" ::: "memory"); } while (0)
#define STR2(x) #x
#define STR(x) STR2(x)

#define STAMP(i) do {} while (0)
#define STAMPW(i) do {} while (0)
#define STAMP7(i) do {} while (0)
#define STAMP3(i) do {} while (0)

__device__ __forceinline__ unsigned rowbase(unsigned q, unsigned xs) {
  const unsigned e = xs - 8u;
  const unsigned third = 512u * (q ^ (4u + (e >> 1))) + 256u * ((q >> 1) & 1u) + 128u * (e & 1u);
  const unsigned pair = 512u * (q ^ (xs < 4u ? 1u : 0u)) + 128u * (xs & 3u);
  return xs < 8u ? pair : third;
}

__device__ __forceinline__ void stage_load(const char* a, const char* b, const unsigned (&va)[4], const unsigned (&vb)[4],
                                           unsigned lds) {
  unsigned keep;
  asm volatile(
      "s_mov_b32 %0, m0\n\t"
      "s_mov_b32 m0, %11\n\t"
      "s_nop 2\n\t"
      "global_load_lds_dwordx4 %1, %9\n\t"
      "global_load_lds_dwordx4 %2, %9 offset:1024\n\t"
      "global_load_lds_dwordx4 %3, %9 offset:2048\n\t"
      "global_load_lds_dwordx4 %4, %9 offset:3072\n\t"
      "s_add_u32 m0, m0, 0x4000\n\t"
      "s_nop 0\n\t"
      "global_load_lds_dwordx4 %5, %10\n\t"
      "global_load_lds_dwordx4 %6, %10 offset:1024\n\t"
      "global_load_lds_dwordx4 %7, %10 offset:2048\n\t"
      "global_load_lds_dwordx4 %8, %10 offset:3072\n\t"
      "s_mov_b32 m0, %0"
      : "=&s"(keep)
      : "v"(va[0]), "v"(va[1]), "v"(va[2]), "v"(va[3]), "v"(vb[0]), "v"(vb[1]), "v"(vb[2]), "v"(vb[3]), "s"(a), "s"(b),
        "s"(lds)
      : "memory", "scc");
}

__device__ __forceinline__ unsigned nat_x(unsigned c) { return (c >> 3) * 1024u + (((c & 7u) ^ ((c >> 4) & 7u)) << 4); }
__device__ __forceinline__ unsigned nat_w(unsigned c) {
  const unsigned kr = c >> 4;
  return kr * 8192u + (((c & 15u) ^ ((kr & 3u) | (kr & 8u))) << 4);
}

struct Half { s16x8 x[4], w[4]; };

__device__ __forceinline__ void read_half(Half& f, lchar* st, unsigned fx, const unsigned (&fw)[4], int kk) {
#pragma unroll
  for (int i = 0; i < 4; ++i) {
    f.x[i] = *(__attribute__((address_space(3))) s16x8*)(st + fx + i * 2048);
    s16x4 lo = __builtin_amdgcn_ds_read_tr16_b64_v4i16((__attribute__((address_space(3))) s16x4*)(st + fw[i] + kk * 8192));
    s16x4 hi = __builtin_amdgcn_ds_read_tr16_b64_v4i16((__attribute__((address_space(3))) s16x4*)(st + fw[i] + kk * 8192 + 1024));
    f.w[i] = __builtin_shufflevector(lo, hi, 0, 1, 2, 3, 4, 5, 6, 7);
  }
}
__device__ __forceinline__ void mma(f32x4 (&acc)[4][4], const Half& f) {
#pragma unroll
  for (int mb = 0; mb < 4; ++mb)
#pragma unroll
    for (int nb = 0; nb < 4; ++nb)
      acc[mb][nb] = __builtin_amdgcn_mfma_f32_16x16x32_bf16(__builtin_bit_cast(bf16x8, f.w[nb]),
                                                            __builtin_bit_cast(bf16x8, f.x[mb]), acc[mb][nb], 0, 0, 0);
}
__device__ __forceinline__ void interleave() {
#pragma unroll
  for (int i = 0; i < 12; ++i) {
    __builtin_amdgcn_sched_group_barrier(0x8, 1, 0);
    __builtin_amdgcn_sched_group_barrier(0x100, 1, 0);
  }
  __builtin_amdgcn_sched_group_barrier(0x8, 4, 0);
}

constexpr unsigned LDS_RED = NST * STAGE, LDS_BYTES = LDS_RED + 64;
__device__ __forceinline__ unsigned ring_next(unsigned o) { return o + STAGE == NST * STAGE ? 0u : o + STAGE; }
__device__ __forceinline__ u64 gemm_load(unsigned lds, unsigned lw, unsigned lane, const char* an, const char* bn,
                                         const char* as, const char* bs, int s0, int s1, unsigned srcs, gu64* next) {
  u64 mine = 0;
  unsigned nx[4], nw[4], ln[4];
#pragma unroll
  for (unsigned i = 0; i < 4; ++i) {
    const unsigned c = (lw * 4u + i) * 64u + lane;
    nx[i] = nat_x(c) - i * 1024u;
    nw[i] = nat_w(c) - i * 1024u;
    ln[i] = lw * 4096u + lane * 16u;
  }
  unsigned ring = 0;
  auto issue = [&](int s) {
    const unsigned slot = lds + lw * 4096u + ring;
    if (s < 8) {
      stage_load(an + (unsigned)s * 128u, bn + (unsigned)s * (64u * 8192u), nx, nw, slot);
    } else {
      const unsigned ps = (unsigned)(s - 8), o = ((srcs >> (ps & ~3u)) & 7u) * SRC_STRIDE + (ps & 3u) * 16384u;
      stage_load(as + o, bs + o, ln, ln, slot);
    }
    ring = ring_next(ring);
  };
  BAR();
  for (int s = s0; s < s0 + NST; ++s) issue(s);
  asm volatile("s_waitcnt vmcnt(%0)" ::"n"(8 * (NST - 1)) : "memory");
  BAR();
  for (int s = s0; s + NST < s1; ++s) {
    asm volatile("s_waitcnt vmcnt(16)" ::: "memory");
    BAR();
    issue(s + NST);
  }
  asm volatile("s_waitcnt vmcnt(16)" ::: "memory");
  BAR();
  if (next) {
    const u64 lo = now() + DRAW_LEAD, inc = PTAU;
    asm volatile("global_atomic_umax_x2 %1, %2, off\n\tglobal_atomic_add_x2 %0, %1, %3, off sc0"
                 : "=&v"(mine) : "v"(next), "v"(lo), "v"(inc) : "memory");
  }
  asm volatile("s_waitcnt vmcnt(8)" ::: "memory");
  BAR();
  asm volatile("s_waitcnt vmcnt(0)" : "+v"(mine)::"memory");
  BAR();
  return mine;
}
__device__ __forceinline__ void gemm_math(f32x4 (&acc)[4][4], lchar* sm, unsigned wv, unsigned lane, int s0, int s1) {
  const unsigned wm = wv >> 1, wn = wv & 1u, g = lane >> 4, m = lane & 15u;
  const unsigned fx0 = (64u * wm + m) * 128u + ((g ^ (m >> 1)) << 4), fx1 = fx0 ^ 64u;
  const unsigned qq = (lane >> 2) & 3u, pp = lane & 3u;
  const unsigned base = 16384u + (8u * g + qq) * 256u + (((8u * wn + 2u * pp) ^ (qq | ((g & 1u) << 3))) << 4) + ((pp & 1u) << 3);
  const unsigned fw[4] = {base, base ^ 8u, base ^ 16u, base ^ 24u};
  Half fa, fb;
  unsigned ring = 0;
  BAR();
  BAR();
  read_half(fa, sm, fx0, fw, 0);
  for (int s = s0; s + 1 < s1; ++s) {
    read_half(fb, sm + ring, fx1, fw, 1);
    mma(acc, fa);
    interleave();
    BAR();
    ring = ring_next(ring);
    read_half(fa, sm + ring, fx0, fw, 0);
    mma(acc, fb);
    interleave();
  }
  read_half(fb, sm + ring, fx1, fw, 1);
  mma(acc, fa);
  interleave();
  mma(acc, fb);
}

__device__ __forceinline__ bool give_up(gu32* tmo, unsigned call, u64 t0) {
  const unsigned v = __builtin_amdgcn_readfirstlane(__hip_atomic_load(tmo, __ATOMIC_RELAXED, __HIP_MEMORY_SCOPE_AGENT));
  if (v == call || now() - t0 > DEADLINE) {
    __hip_atomic_store(tmo, call, __ATOMIC_RELAXED, __HIP_MEMORY_SCOPE_AGENT);
    return true;
  }
  return false;
}

__device__ __forceinline__ void poll_flags(const char* f, const char* g, unsigned call, gu32* tmo) {
  const u64 t0 = now();
  for (unsigned it = 1;; ++it) {
    unsigned vf = 0, vg = 0;
    if (f) vf = __hip_atomic_load((gu32*)f, __ATOMIC_RELAXED, __HIP_MEMORY_SCOPE_SYSTEM);
    if (g) vg = __hip_atomic_load((gu32*)g, __ATOMIC_RELAXED, __HIP_MEMORY_SCOPE_SYSTEM);
    if (vf >= call) f = nullptr;
    if (vg >= call) g = nullptr;
    if (__all(f == nullptr && g == nullptr)) break;
    if ((it & 63u) == 0 && give_up(tmo, call, t0)) break;
    __builtin_amdgcn_s_sleep(PSLEEP);
  }
}

__device__ __forceinline__ unsigned pack_bf16(float a, float b) {
  f32x2 v = {a, b};
  return __builtin_bit_cast(unsigned, __builtin_convertvector(v, bf16x2));
}

__device__ __forceinline__ void add_partial(f32x4 (&acc)[4][4], const u32x4 (&v)[8]) {
#pragma unroll
  for (int i = 0; i < 8; ++i) {
#pragma unroll
    for (int h = 0; h < 2; ++h) {
      const unsigned u0 = v[i][2 * h], u1 = v[i][2 * h + 1];
      acc[i >> 1][(i & 1) * 2 + h] += f32x4{__uint_as_float(u0 << 16), __uint_as_float(u0 & 0xffff0000u),
                                            __uint_as_float(u1 << 16), __uint_as_float(u1 & 0xffff0000u)};
    }
  }
}

__device__ __forceinline__ unsigned wave_max(unsigned m) {
  m = max(m, (unsigned)__builtin_amdgcn_update_dpp(0, (int)m, 0x111, 0xf, 0xf, false));
  m = max(m, (unsigned)__builtin_amdgcn_update_dpp(0, (int)m, 0x112, 0xf, 0xf, false));
  m = max(m, (unsigned)__builtin_amdgcn_update_dpp(0, (int)m, 0x114, 0xf, 0xf, false));
  m = max(m, (unsigned)__builtin_amdgcn_update_dpp(0, (int)m, 0x118, 0xf, 0xf, false));
  return max(max((unsigned)__builtin_amdgcn_readlane((int)m, 15), (unsigned)__builtin_amdgcn_readlane((int)m, 31)),
             max((unsigned)__builtin_amdgcn_readlane((int)m, 47), (unsigned)__builtin_amdgcn_readlane((int)m, 63)));
}

__global__ void __launch_bounds__(NT, 2) fused_gemm_rs_q8(const Params p) {
  __shared__ __attribute__((aligned(1024))) char smem[LDS_BYTES];
  lchar* sm = (lchar*)smem;
  typedef __attribute__((address_space(3))) unsigned lu32;
  const u64 tstart = now();
  const unsigned t = threadIdx.x, lane = t & 63;
  const unsigned wv = __builtin_amdgcn_readfirstlane(t >> 6);
  unsigned vb = (blockIdx.x & ~7u) | ((XPERM >> (4u * (blockIdx.x & 7u))) & 7u);
  if (SWAP && (vb & 6u) == 0u && (vb >> 5) == 2u) vb += 20u * 8u + 2u;
  else if (SWAP && (vb & 6u) == 2u && (vb >> 5) == 7u) vb -= 20u * 8u + 2u;
  const unsigned d = vb & 7u, l = vb >> 3;
  u64 a_job = ((const u64*)p.job)[vb];
  char* a_peer = p.wsx[d == 0 ? 1u : d];
  char* a_me = p.wsx[0];
  const char* a_x = p.x;
  const char* a_w = p.w;
  signed char* a_out = p.out;
  unsigned a_rank = (unsigned)p.rank;
  asm volatile("" : "+s"(a_job), "+s"(a_peer), "+s"(a_me), "+s"(a_x), "+s"(a_w), "+s"(a_out), "+s"(a_rank));
  typedef __attribute__((address_space(1))) char gchar;
  char* const me = (char*)(gchar*)a_me;
  char* const peer = (char*)(gchar*)a_peer;
  const char* const px = (const char*)(const gchar*)a_x;
  const char* const pw = (const char*)(const gchar*)a_w;
  const Job job = __builtin_bit_cast(Job, a_job);
  const unsigned r = a_rank & 7u;
  const unsigned q = r ^ d;
  gu32* const ctr = (gu32*)(me + WS_CTR + blockIdx.x * 128);
  gu32* const tmo = (gu32*)(me + WS_TMO);
  unsigned prev = __hip_atomic_load(ctr, __ATOMIC_RELAXED, __HIP_MEMORY_SCOPE_AGENT);
  const unsigned j = r >> 2, xs = job.tile & 15u, n = job.tile >> 4;
  const unsigned fa = job.fin & 3u, fn = job.fin >> 2;
  const bool ships = d < 2u || d >= 4u;
  const bool detour = d < 2u && job.turn != 255u;
  const unsigned allsrc = d >= 4u ? 63u & ~(1u << (d - 2u)) : 63u;
  const unsigned ln0 = t * 16u;

  lu32* const red = (lu32*)(sm + LDS_RED);

  auto callno = [&]() -> unsigned {
    unsigned pv = prev;
    asm volatile("" : "+v"(pv));
    return __builtin_amdgcn_readfirstlane(pv) + 1u;
  };
  auto push = [&](unsigned k0, unsigned k1, unsigned kstep) {
      const unsigned u = l & 7u;
      const unsigned cq = (q ^ (d >= 4u ? (r & (q >> 1)) : 0u)) & 1u, jq = q >> 2;
      for (unsigned k = k0; k < k1; k += kstep) {
        unsigned it = ord_item(k, d), src = d - 2u;
        const unsigned sl = (u + ROT * k) & 7u, slotk = k;
        u32x4 v[8];
        if (it < 16u) {
          const unsigned xa = nat_x(lane), xb = nat_x(lane + 64u);
          const __amdgpu_buffer_rsrc_t rs = __builtin_amdgcn_make_buffer_rsrc((void*)px, 0, 4u << 20, 0x00020000);
          const unsigned off = (rowbase(q, it) + (sl & 1u) * 64u) * 1024u + cq * 512u + (sl >> 1) * 128u;
#pragma unroll
          for (unsigned i = 0; i < 8; ++i) v[i] = __builtin_amdgcn_raw_buffer_load_b128(rs, (i & 1u) ? xb : xa, off + (i >> 1) * 16384u, 0);
        } else if (NF && k1 > 16u && it >= F_) {
          const unsigned ds = 4u | ((it & 1u) << 1) | ((r ^ d) & 1u);
          it -= NF;
          src = (ds ^ d) - 2u;
          wait_until(tstart + T0 + (slotk - 2u) * TAU);
          poll_flags(lane < 8u ? me + WS_FIN + ((ds - 2u) * 32u + it) * 128u + lane * 4u : nullptr, nullptr, callno(), tmo);
          const __amdgpu_buffer_rsrc_t rs = __builtin_amdgcn_make_buffer_rsrc(me + WS_IN + (size_t)((ds - 2u) * SRC_STRIDE + it * ITEM + sl * 8192u), 0, 8192u, 0x00020000);
          asm volatile("" ::: "memory");
#pragma unroll
          for (unsigned i = 0; i < 8; ++i) v[i] = __builtin_amdgcn_raw_buffer_load_b128(rs, lane * 16u, i * 1024u, AUX_SYS);
        } else {
          const unsigned wa = nat_w(lane), wb = nat_w(lane + 128u);
          const __amdgpu_buffer_rsrc_t rs = __builtin_amdgcn_make_buffer_rsrc((void*)pw, 0, 4u << 20, 0x00020000);
          const unsigned off = (cq * 256u + sl * 32u) * 8192u + (jq * 2048u + (it - 16u) * 128u) * 2u;
#pragma unroll
          for (unsigned i = 0; i < 8; ++i) v[i] = __builtin_amdgcn_raw_buffer_load_b128(rs, (i & 2u) ? wb : wa, off + (i & 5u) * 32768u, 0);
        }
        char* dst = peer + WS_IN + (size_t)(src * SRC_STRIDE + it * ITEM + sl * 8192u);
        const __amdgpu_buffer_rsrc_t rd = __builtin_amdgcn_make_buffer_rsrc(dst, 0, 8192u, 0x00020000);
        wait_until(tstart + T0 + slotk * TAU + u * STAG);
#pragma unroll
        for (unsigned i = 0; i < 8; ++i) __builtin_amdgcn_raw_buffer_store_b128(v[i], rd, lane * 16u, i * 1024u, AUX_SYS);
        VM0();
        const unsigned cn = callno();
        if (lane == 0)
          __hip_atomic_store((gu32*)(peer + WS_FIN + (src * 32u + it) * 128u + u * 4u), cn, __ATOMIC_RELAXED, __HIP_MEMORY_SCOPE_SYSTEM);
      }
      STAMPW(1);
  };
  if (t < 16u) red[t] = 0u;
  if (PRETOUCH && wv == 3u) {
    const unsigned po = (r * 256u + blockIdx.x) * 128u;
    const char* a = nullptr;
    if (d >= 2u && l < 8u && (PRETOUCH & 1)) {
      if (lane == 0u) a = peer + WS_SPARE + po;
      else if (lane < 3u) a = peer + WS_IN - 65536u + (size_t)((d - 2u + lane - 1u) * SRC_STRIDE + (po & 65535u));
    }
    if (d < 2u && (PRETOUCH & 2) && lane >= 8u && lane < 15u) a = me + WS_IN - 65536u + (size_t)((lane - 8u) * SRC_STRIDE + (po & 65535u));
    if (d < 4u && (PRETOUCH & 4) && lane == 16u) a = (const char*)a_out + (128u * fa * 4096u + fn * 128u);
    if (a) {
      const unsigned v = __hip_atomic_load((gu32*)a, __ATOMIC_RELAXED, __HIP_MEMORY_SCOPE_SYSTEM);
      asm volatile("" ::"v"(v));
    }
  }
  if (d >= 2u && l < 8u) push(wv, 16u, 8u);

  if (wv >= 4) {
    if ((job.plan & 15u) == 3u || (d >= 2u && l >= 8u)) wait_until(tstart + DELAY);
    for (unsigned plan = job.plan | 0xf0000u; (plan & 15u) != 15u; plan >>= 4) {
      const unsigned kind = plan & 15u;
      if (kind == 5u || kind == 6u) { push((kind == 5u ? 12u : SETC - 4u) + wv, d >= 4u ? NCR : kind == 5u ? SETC : NIN, 4u); continue; }
      const unsigned call = callno();
      const bool staged = kind == 1u || kind >= 7u;
      const unsigned mask = kind == 7u ? job.early : kind == 8u ? allsrc & ~job.early : allsrc;
      const int s0 = staged ? 8 : 0, s1 = staged ? 8 + 4 * __builtin_popcount(mask) : 8;
      const bool last = staged && kind != 7u && ships;
      if (staged && wv == 4) {
        const unsigned s = lane >> 3;
        const bool act = (mask >> s) & 1u;
        wait_until(tstart + T0 + (kind == 7u ? job.pos2 : job.pos) * TAU);
        poll_flags(act ? me + WS_FIN + (s * 32u + xs) * 128u + (lane & 7u) * 4u : nullptr,
                   act ? me + WS_FIN + (s * 32u + 16u + n) * 128u + (lane & 7u) * 4u : nullptr, call, tmo);
        STAMPW(12);
        if (ACQ) {
          __builtin_amdgcn_fence(__ATOMIC_ACQUIRE, "");
          VM0();
        }
      }
      unsigned srcs = 0;
      for (unsigned s = 6u; s-- > 0u;) if ((mask >> s) & 1u) srcs = (srcs << 4) | s;
      const u64 mine = gemm_load((unsigned)(size_t)sm, wv - 4u, lane,
                px + (size_t)(kind == 3u ? 512u * r + 128u * fa : rowbase(r, xs)) * 1024u,
                pw + (kind == 3u ? (j ^ 1u) * 2048u + fn * 128u : j * 2048u + n * 128u) * 2u, me + WS_IN + (size_t)(xs * ITEM),
                me + WS_IN + (size_t)((16u + n) * ITEM), s0, s1, srcs,
                last && d < 2u && !detour && t == 256u ? (gu64*)(me + WS_TICK) : nullptr);
      if (last) {
        if (t == 256u) red[12] = (unsigned)(mine - tstart);
        BAR();
        u32x4 v[8];
#pragma unroll
        for (int i = 0; i < 8; ++i) v[i] = *(__attribute__((address_space(3))) u32x4*)(sm + (wv - 4u) * 8192u + i * 1024u + lane * 16u);
        int go = (int)__builtin_amdgcn_readfirstlane(red[12]);
        if (go > (int)(now() - tstart) + GATE + 64 * PTAU) go = 0;
        unsigned stag = PSTAG;
        if (d >= 4u) {
          const unsigned tn = job.turn ^ ((r & 1u) * TURNX);
          go = GATE + (int)(PAIRS ? (tn & ~1u) * PTAU + (tn & 1u) * PSTAG : tn * PTAU);
          if (PAIRS) stag = 2 * PSTAG;
        }
        if (detour) go = RGATE + (int)job.turn * PTAU;
        const unsigned a = d < 2u ? xs : 2u * ((r >> 1) & 1u) + (xs & 1u);
        const unsigned slot = ((d < 2u ? (detour ? 3u : 0u) : 1u + (r & 1u)) * 4u + a) * 16u + n;
        char* const to = detour ? (char*)(gchar*)p.wsx[2u + d] : peer;
        const __amdgpu_buffer_rsrc_t rs = __builtin_amdgcn_make_buffer_rsrc(to + WS_P + (size_t)slot * PTILE + (wv - 4u) * 8192u, 0, 8192u, 0x00020000);
        if (go > 0) wait_until(tstart + (unsigned)go + (wv - 4u) * stag);
        STAMPW(4);
#pragma unroll
        for (int i = 0; i < 8; ++i) __builtin_amdgcn_raw_buffer_store_b128(v[i], rs, lane * 16u, i * 1024u, AUX_SYS);
        VM0();
        if (lane == 0) __hip_atomic_store((gu32*)(to + WS_FP + slot * 128u + (wv - 4u) * 4u), call, __ATOMIC_RELAXED, __HIP_MEMORY_SCOPE_SYSTEM);
        STAMP7(6);
      }
    }
    if (d < 4u) {
      const unsigned call = callno();
      if (detour) {
        const unsigned slot = xs * 16u + (n ^ 1u);
        char* const to = (char*)(gchar*)p.wsx[2u + d];
        poll_flags(lane == 0 ? me + WS_FP + (192u + slot) * 128u + (wv - 4u) * 4u : nullptr, nullptr, call, tmo);
        const __amdgpu_buffer_rsrc_t ri = __builtin_amdgcn_make_buffer_rsrc(me + WS_P + (size_t)(192u + slot) * PTILE + (wv - 4u) * 8192u, 0, 8192u, 0x00020000);
        const __amdgpu_buffer_rsrc_t rs = __builtin_amdgcn_make_buffer_rsrc(to + WS_P + (size_t)slot * PTILE + (wv - 4u) * 8192u, 0, 8192u, 0x00020000);
        u32x4 v[8];
        asm volatile("" ::: "memory");
#pragma unroll
        for (int i = 0; i < 8; ++i) v[i] = __builtin_amdgcn_raw_buffer_load_b128(ri, lane * 16u, i * 1024u, AUX_SYS);
        wait_until(tstart + RGATE + (RELAY_R1 + RELAY_GAP + job.turn) * PTAU + (wv - 4u) * PSTAG);
        STAMPW(14);
#pragma unroll
        for (int i = 0; i < 8; ++i) __builtin_amdgcn_raw_buffer_store_b128(v[i], rs, lane * 16u, i * 1024u, AUX_SYS);
        VM0();
        if (lane == 0) __hip_atomic_store((gu32*)(to + WS_FP + slot * 128u + (wv - 4u) * 4u), call, __ATOMIC_RELAXED, __HIP_MEMORY_SCOPE_SYSTEM);
        STAMP7(15);
      }
      u64 g = 0, h = t < 256u + NR ? 0ull : (u64)call << 32;
      wait_until(tstart + SWEEP_T);
      const u64 t0 = now();
      unsigned state = 1u;
      for (;;) {
        if ((unsigned)(g >> 32) != call) g = __hip_atomic_load((gu64*)(me + WS_G + (t - 256u) * 8u), __ATOMIC_RELAXED, __HIP_MEMORY_SCOPE_SYSTEM);
        if ((unsigned)(h >> 32) != call) h = __hip_atomic_load((gu64*)(me + WS_G + t * 8u), __ATOMIC_RELAXED, __HIP_MEMORY_SCOPE_SYSTEM);
        const unsigned tm = __hip_atomic_load(tmo, __ATOMIC_RELAXED, __HIP_MEMORY_SCOPE_AGENT);
        const bool gok = (unsigned)(g >> 32) == call, hok = (unsigned)(h >> 32) == call;
        const unsigned m = wave_max(max(gok ? (unsigned)g : 0u, hok ? (unsigned)h : 0u));
        if (lane == 0) __hip_atomic_store(red + wv, m, __ATOMIC_RELAXED, __HIP_MEMORY_SCOPE_WORKGROUP);
        if (__builtin_amdgcn_readfirstlane(tm) == call) { state = 2u; break; }
        if (__all(gok && hok)) break;
        if (now() - t0 > DEADLINE) {
          __hip_atomic_store(tmo, call, __ATOMIC_RELAXED, __HIP_MEMORY_SCOPE_AGENT);
          state = 2u;
          break;
        }
        asm volatile("" ::: "memory");
      }
      asm volatile("" ::: "memory");
      if (lane == 0) __hip_atomic_store(red + 4u + wv, state, __ATOMIC_RELAXED, __HIP_MEMORY_SCOPE_WORKGROUP);
      STAMPW(13);
    }
  } else {
    f32x4 acc[4][4];
#pragma unroll
    for (int i = 0; i < 4; ++i)
#pragma unroll
      for (int k = 0; k < 4; ++k) acc[i][k] = f32x4{0.f, 0.f, 0.f, 0.f};
    const bool collector = vb == COLLECTOR;
    char* gdst = p.wsx[lane & 7u] + WS_G + (collector ? 256u + r : r * 32u + (d & 2u) * 8u + (fn & 3u) * 4u + fa) * 8u;
    asm volatile("" : "+v"(gdst));
    const bool parks = (job.plan & 15u) == 3u;
    unsigned pend = d < 2u ? 3u : 1u;
    const __amdgpu_buffer_rsrc_t rpark = __builtin_amdgcn_make_buffer_rsrc(me + WS_PARK + (size_t)((d & 1u) * 32u + l) * 65536u, 0, 65536u, 0x00020000);
    for (unsigned plan = job.plan | 0xf0000u; (plan & 15u) != 15u; plan >>= 4) {
      const unsigned kind = plan & 15u;
      if (kind == 5u || kind == 6u) continue;
      const bool staged = kind == 1u || kind >= 7u;
      const unsigned mask = kind == 7u ? job.early : kind == 8u ? allsrc & ~job.early : allsrc;
      const int s0 = staged ? 8 : 0, s1 = staged ? 8 + 4 * __builtin_popcount(mask) : 8;
      const bool last = staged && kind != 7u && ships;
      if (staged) STAMP(3);
      if (EARLY_TAKE && kind == 8u && !ships) {
        const unsigned call = callno();
        bool in = false;
        do {
          in = __builtin_amdgcn_readfirstlane(__hip_atomic_load((gu32*)(me + WS_FP + (fa * 16u + fn) * 128u + wv * 4u), __ATOMIC_RELAXED, __HIP_MEMORY_SCOPE_SYSTEM)) >= call;
          if (!in) __builtin_amdgcn_s_sleep(PSLEEP);
        } while (!in && now() < tstart + T0 + job.pos * TAU + TAKE_D);
        const __amdgpu_buffer_rsrc_t rs = __builtin_amdgcn_make_buffer_rsrc(me + WS_P + (size_t)(fa * 16u + fn) * PTILE + wv * 8192u, 0, in ? 8192u : 0u, 0x00020000);
        u32x4 v[8];
        asm volatile("" ::: "memory");
#pragma unroll
        for (int i = 0; i < 8; ++i) v[i] = __builtin_amdgcn_raw_buffer_load_b128(rs, lane * 16u, i * 1024u, AUX_SYS);
        add_partial(acc, v);
        if (in) pend = 0u;
      }
      if (!staged) {
#pragma unroll
        for (int i = 0; i < 4; ++i)
#pragma unroll
          for (int k = 0; k < 4; ++k) acc[i][k] = f32x4{0.f, 0.f, 0.f, 0.f};
      }
      gemm_math(acc, sm, wv, lane, s0, s1);
      if (kind == 3u && parks) {
#pragma unroll
        for (int i = 0; i < 16; ++i)
          __builtin_amdgcn_raw_buffer_store_b128(__builtin_bit_cast(u32x4, acc[i >> 2][i & 3]), rpark, ln0, i * 4096u, 0);
      }
      if (kind == 0u) STAMP(2);
      if (staged) STAMP(5);
      if (last) {
#pragma unroll
        for (int i = 0; i < 8; ++i) {
          const f32x4 c0 = acc[i >> 1][(i & 1) * 2], c1 = acc[i >> 1][(i & 1) * 2 + 1];
          *(__attribute__((address_space(3))) u32x4*)(sm + wv * 8192u + i * 1024u + lane * 16u) =
              u32x4{pack_bf16(c0[0], c0[1]), pack_bf16(c0[2], c0[3]), pack_bf16(c1[0], c1[1]), pack_bf16(c1[2], c1[3])};
        }
        BAR();
        if (parks) {
#pragma unroll
          for (int i = 0; i < 16; ++i)
            acc[i >> 2][i & 3] = __builtin_bit_cast(f32x4, __builtin_amdgcn_raw_buffer_load_b128(rpark, ln0, i * 4096u, 0));
        }
      }
    }
    const unsigned call = callno();
    if (t == 0) __hip_atomic_store(ctr, call, __ATOMIC_RELAXED, __HIP_MEMORY_SCOPE_AGENT);
    if (collector && wv == 0) {
      u64 g = lane < 48u ? 0ull : (u64)call << 32, h = g;
      wait_until(tstart + SWEEP_T - COLLECT_LEAD);
      const u64 t0 = now();
      for (unsigned it = 1;; ++it) {
        if ((unsigned)(g >> 32) != call) g = __hip_atomic_load((gu64*)(me + WS_GL + lane * 8u), __ATOMIC_RELAXED, __HIP_MEMORY_SCOPE_SYSTEM);
        if ((unsigned)(h >> 32) != call) h = __hip_atomic_load((gu64*)(me + WS_GL + 512u + lane * 8u), __ATOMIC_RELAXED, __HIP_MEMORY_SCOPE_SYSTEM);
        if (__all((unsigned)(g >> 32) == call && (unsigned)(h >> 32) == call)) break;
        if ((it & 63u) == 0 && give_up(tmo, call, t0)) break;
      }
      const unsigned m = wave_max(max((unsigned)g, (unsigned)h));
      if (lane < NR) __hip_atomic_store((gu64*)gdst, ((u64)call << 32) | m, __ATOMIC_RELAXED, __HIP_MEMORY_SCOPE_SYSTEM);
    }
    if (d < 4u) {
      const unsigned slot = ((d < 2u ? 4u : 0u) + fa) * 16u + fn;
      STAMP3(11);
      unsigned got = 0u;
      while (pend) {
        const u64 t0 = now();
        for (unsigned it = 1; !got; ++it) {
          unsigned f = 0;
          if (lane < 2u && ((pend >> lane) & 1u))
            f = __hip_atomic_load((gu32*)(me + WS_FP + (slot + lane * 64u) * 128u + wv * 4u), __ATOMIC_RELAXED, __HIP_MEMORY_SCOPE_SYSTEM);
          got = (unsigned)__ballot(f >= call) & pend;
          if (got) break;
          if ((it & 63u) == 0 && give_up(tmo, call, t0)) { got = pend; break; }
          __builtin_amdgcn_s_sleep(PSLEEP);
        }
        got = __builtin_amdgcn_readfirstlane(got);
        STAMP3(7);
        const unsigned w = got & 1u ? 0u : 1u;
        const __amdgpu_buffer_rsrc_t rs = __builtin_amdgcn_make_buffer_rsrc(me + WS_P + (size_t)(slot + w * 64u) * PTILE + wv * 8192u, 0, 8192u, 0x00020000);
        u32x4 v[8];
        asm volatile("" ::: "memory");
#pragma unroll
        for (int i = 0; i < 8; ++i) v[i] = __builtin_amdgcn_raw_buffer_load_b128(rs, lane * 16u, i * 1024u, AUX_SYS);
        add_partial(acc, v);
        pend &= ~(1u << w);
        got &= pend;
      }
      float mf = 0.f;
#pragma unroll
      for (int i = 0; i < 4; ++i)
#pragma unroll
        for (int k = 0; k < 4; ++k)
#pragma unroll
          for (int e = 0; e < 4; ++e) mf = fmaxf(mf, fabsf(acc[i][k][e]));
      const unsigned mw = wave_max(__float_as_uint(mf));
      unsigned arr = 0;
      if (lane == 0) {
        __hip_atomic_fetch_max(red, mw, __ATOMIC_RELAXED, __HIP_MEMORY_SCOPE_WORKGROUP);
        arr = __hip_atomic_fetch_add(red + 1, 1u, __ATOMIC_ACQ_REL, __HIP_MEMORY_SCOPE_WORKGROUP);
      }
      if (__builtin_amdgcn_readfirstlane(arr) == 3u && lane < (fn >= 12u ? NR : 1u)) {
        const u64 g = ((u64)call << 32) | __hip_atomic_load(red, __ATOMIC_RELAXED, __HIP_MEMORY_SCOPE_WORKGROUP);
        __hip_atomic_store((gu64*)(fn >= 12u ? gdst : me + WS_GL + ((d & 2u) * 32u + fn * 4u + fa) * 8u), g, __ATOMIC_RELAXED, __HIP_MEMORY_SCOPE_SYSTEM);
      }
      STAMP3(8);
      const unsigned jc = d < 2u ? (j ^ 1u) : j;
      const unsigned pp = lane >> 4;
      const __amdgpu_buffer_rsrc_t ro = __builtin_amdgcn_make_buffer_rsrc(a_out, 0, 512u * 4096u, 0x00020000);
      const unsigned oo = (128u * fa + 64u * (wv >> 1) + (lane & 15u)) * 4096u + jc * 2048u + fn * 128u + 64u * (wv & 1u) + 16u * pp;
      unsigned used = 0xffffffffu;
      for (;;) {
        const u32x4 sv = *(volatile __attribute__((address_space(3))) u32x4*)(red + 8);
        const u32x4 mv = *(volatile __attribute__((address_space(3))) u32x4*)(red + 4);
        const unsigned s0 = sv[0], s1 = sv[1], s2 = sv[2], s3 = sv[3], m0 = mv[0], m1 = mv[1], m2 = mv[2], m3 = mv[3];
        const bool fin = __builtin_amdgcn_readfirstlane(min(min(s0, s1), min(s2, s3))) != 0u;
        const bool bad = __builtin_amdgcn_readfirstlane(max(max(s0, s1), max(s2, s3))) == 2u;
        unsigned cur = __builtin_amdgcn_readfirstlane(max(max(m0, m1), max(m2, m3)));
        const bool seen = __builtin_amdgcn_readfirstlane(min(min(m0, m1), min(m2, m3))) != 0u;
        if (bad) cur = 0xfffffffeu;
        if (cur != used && (fin || (SPEC && seen && cur >= mw))) {
          const float amax = __uint_as_float(cur);
          const float inv = amax > 0.f ? 127.0f / amax : 0.f;
          const float magic = 12582912.0f;
          if (used != 0xffffffffu) VM0();
#pragma unroll
          for (int mb = 0; mb < 4; ++mb) {
            unsigned dw[4];
#pragma unroll
            for (int nb = 0; nb < 4; ++nb) {
              unsigned b[4];
#pragma unroll
              for (int e = 0; e < 4; ++e)
                b[e] = __float_as_uint(__builtin_amdgcn_fmed3f(__builtin_fmaf(acc[mb][nb][e], inv, magic), magic - 127.f, magic + 127.f));
              const unsigned lo = __builtin_amdgcn_perm(b[1], b[0], 0x0c0c0400u), hi = __builtin_amdgcn_perm(b[3], b[2], 0x04000c0cu);
              dw[nb] = bad ? 0x80808080u : (lo | hi);
            }
            const bool odd = pp & 1u;
            const u32x4 o = {odd ? dw[1] : dw[0], odd ? dw[0] : dw[1], odd ? dw[3] : dw[2], odd ? dw[2] : dw[3]};
            __builtin_amdgcn_raw_buffer_store_b128(o, ro, oo + mb * 65536u, 0, OUT_AUX);
          }
          used = cur;
          STAMP3(9);
        }
        if (fin) break;
        __builtin_amdgcn_s_sleep(1);
      }
      STAMP(10);
    }
  }
}

void make_jobs(Job* job) {
  int slot[6][32];
  for (int s = 0; s < 6; ++s) {
    for (int it = 0; it < 32; ++it) slot[s][it] = 0;
    for (int k = 0; k < (s < 2 ? NIN : NCR); ++k)
      if (ord_item(k, s + 2) < F_) slot[s][ord_item(k, s + 2)] = k;
  }
  for (int k = 0; k < NIN; ++k) {
    const int e = (int)ord_item(k, 2) - F_;
    for (int s = 2; s < 6 && e >= 0; ++s)
      if ((((s + 2) >> 1) & 1) != (e & 1)) slot[s][32 - NF + e] = k;
  }
  for (int d = 0; d < 8; ++d) {
    Job jb[32];
    int key[32], key2[32], early[32], srt[32], idx[32], txs[32], tn[32];
    for (int m = 0; m < 32; ++m) {
      txs[m] = d < 4 ? (d < 2 ? 0 : 4) + (m & 3) : 8 + 2 * (d - 4) + (m >> 4);
      tn[m] = d < 4 ? 2 * (m >> 2) + (d & 1) : (m & 15);
      int at[6], lastk = 0, cnt = 0;
      for (int s = 0; s < 6; ++s) {
        at[s] = slot[s][txs[m]] > slot[s][16 + tn[m]] ? slot[s][txs[m]] : slot[s][16 + tn[m]];
        if (d >= 4 && s == d - 2) at[s] = -1;
        if (at[s] > lastk) lastk = at[s];
        cnt += at[s] >= 0;
      }
      int best = lastk * TAU + RESTART + cnt * SRC_T - SPLIT_GAIN;
      early[m] = 0;
      key2[m] = lastk;
      for (int c = 0; c < lastk; ++c) {
        int mk = 0, na = 0, ka = 0;
        for (int s = 0; s < 6; ++s)
          if (at[s] >= 0 && at[s] <= c) { mk |= 1 << s; ++na; if (at[s] > ka) ka = at[s]; }
        if (!na) continue;
        const int fa = ka * TAU + RESTART + na * SRC_T;
        const int fb = (fa > lastk * TAU ? fa : lastk * TAU) + RESTART + (cnt - na) * SRC_T;
        if (fb < best) { best = fb; early[m] = mk; key2[m] = ka; }
      }
      key[m] = lastk * 64 + m;
      srt[m] = (d == 2 || d == 3 ? key2[m] * TAU : best) * 64 + m;
      idx[m] = m;
    }
    for (int a = 0; a < 32; ++a)
      for (int b = a + 1; b < 32; ++b)
        if (srt[idx[b]] < srt[idx[a]]) { const int tmp = idx[a]; idx[a] = idx[b]; idx[b] = tmp; }
    for (int l = 0; l < 32; ++l) {
      const int m = idx[(l + 16) & 31];
      jb[l].tile = (unsigned char)(txs[m] | (tn[m] << 4));
      jb[l].early = (unsigned char)early[m];
      jb[l].pos = (unsigned char)(key[m] / 64);
      const int fm = d < 2 ? idx[31 - ((l + 16) & 31)] : m;
      jb[l].fin = (unsigned char)((txs[fm] & 3) | (tn[fm] << 2));
      const bool sp = early[m] != 0;
      if (d >= 2 && l >= 8 && l < 16) jb[l].plan = sp ? 0x8750 : 0xf150;
      else if (d >= 2 && d < 4 && l >= 16 && l < 24) jb[l].plan = sp ? 0x8670 : 0xf160;
      else if (d >= 2) jb[l].plan = sp ? 0xf870 : 0xff10;
      else if (tn[m] < EARLY_N) jb[l].plan = sp ? 0x3870 : 0xf310;
      else jb[l].plan = sp ? 0x8703 : 0xf103;
      jb[l].pos2 = (unsigned char)key2[m];
      jb[l].turn = (unsigned char)((l + 16) & 31);
      if (d < 2) {
        const int rc = (tn[m] - RELAY_N) / 2;
        jb[l].turn = (unsigned char)(tn[m] >= RELAY_N && rc * 4 + (txs[m] & 3) < RELAY_R1 ? rc * 4 + (txs[m] & 3) : 255);
      }
      job[l * 8 + d] = jb[l];
    }
  }
}

}

extern "C" void kernel_launch(int rank, int nranks, ncclComm_t comm, hipStream_t stream, void* const* d_in,
                              const int* in_sizes, int n_in, void* d_out, int out_size, const int* in_dtypes,
                              int out_dtype, void* const* d_out_all, void* const* d_ws_all, size_t ws_bytes) {
  if (nranks != NR || (unsigned)rank >= (unsigned)NR || n_in < 2 || ws_bytes < WS_END || out_size != 512 * 4096 ||
      in_sizes[0] != 4096 * 512 || in_sizes[1] != 512 * 4096) {
    fprintf(stderr, "kernel_launch: unexpected problem shape\n");
    return;
  }
  Params p;
  memset(&p, 0, sizeof p);
  p.x = (const char*)d_in[0];
  p.w = (const char*)d_in[1];
  p.out = (signed char*)d_out;
  for (int i = 0; i < NR; ++i) p.wsx[i] = (char*)d_ws_all[rank ^ i];
  p.rank = rank;
  make_jobs(p.job);
  hipLaunchKernelGGL(fused_gemm_rs_q8, dim3(NWG), dim3(NT), 0, stream, p);
  HIP_CHECK(hipGetLastError());
}
```

```cpp
#include <hip/hip_runtime.h>
#include <rccl/rccl.h>
#include <cstdio>
#include <cstdint>
#include <cstring>

#define HIP_CHECK(expr) do { hipError_t _e = (expr); if (_e != hipSuccess) { \
    fprintf(stderr, "HIP error %s at %s:%d\n", hipGetErrorString(_e), __FILE__, __LINE__); \
    return; } } while (0)

namespace {

typedef short s16x4 __attribute__((ext_vector_type(4)));
typedef short s16x8 __attribute__((ext_vector_type(8)));
typedef __bf16 bf16x8 __attribute__((ext_vector_type(8)));
typedef __bf16 bf16x2 __attribute__((ext_vector_type(2)));
typedef float f32x4 __attribute__((ext_vector_type(4)));
typedef float f32x2 __attribute__((ext_vector_type(2)));
typedef unsigned u32x4 __attribute__((ext_vector_type(4)));
typedef __attribute__((address_space(1))) unsigned gu32;
typedef __attribute__((address_space(1))) unsigned long long gu64;
typedef __attribute__((address_space(3))) char lchar;
typedef unsigned long long u64;

constexpr int NR = 8;
constexpr int NWG = 256, NT = 512;
constexpr unsigned ITEM = 65536;
constexpr unsigned PTILE = 32768;
constexpr unsigned STAGE = 32768;
constexpr int NST = 4;

constexpr size_t WS_CTR = 0;
constexpr size_t WS_TMO = 32 << 10;
constexpr size_t WS_TICK = 33 << 10;
constexpr size_t WS_FIN = 64 << 10;
constexpr size_t WS_FP = 96 << 10;
constexpr size_t WS_G = 128 << 10;
constexpr size_t WS_GL = 136 << 10;
constexpr size_t WS_IN = (2 << 20) + 65536;
constexpr unsigned SRC_STRIDE = (2u << 20) + 65536u;
constexpr size_t WS_P = 16 << 20;
constexpr size_t WS_PARK = 24 << 20;
constexpr size_t WS_END = 28 << 20;
static_assert(WS_IN + 6 * (size_t)SRC_STRIDE <= WS_P, "workspace layout");

constexpr u64 DEADLINE = 100000000ull;
constexpr int AUX_SYS = 17;

constexpr int TAU = 113;
constexpr int PTAU = 57;
constexpr int T0 = 160;
constexpr int NF = 4;
constexpr int NIN = 32 + NF, NCR = 30 - NF / 2;
constexpr int GATE = T0 + NCR * TAU;
constexpr int RGATE = T0 + NIN * TAU;
constexpr int SWEEP_T = GATE + 32 * PTAU + 150;
constexpr int DELAY = 300;
constexpr int DRAW_LEAD = 110;
constexpr int RELAY_N = 12;
constexpr int EARLY_N = 4;
constexpr unsigned COLLECTOR = 16 * 8 + 4;
constexpr int PSLEEP = 4;

constexpr unsigned char W_ = 16, F_ = 32, G_ = 40;
#define ORD_HEAD 0, W_ + 0, 1, W_ + 1, 2, 3, W_ + 2, W_ + 3, 8, W_ + 4, 10, 12, W_ + 5, 14, W_ + 6, 9, W_ + 7, 11, W_ + 8, 13, W_ + 9, 15
constexpr unsigned char OIN[40] = {ORD_HEAD, W_ + 10, W_ + 11, W_ + 12, W_ + 13, W_ + 14, W_ + 15, F_ + 0, F_ + 1, F_ + 2, F_ + 3, 4, 5, 6, 7};
constexpr unsigned char OCR[40] = {ORD_HEAD, G_ + 0, G_ + 1, W_ + 10, W_ + 11, 6, 7};
constexpr u64 ord_pack(const unsigned char* o) {
  u64 v = 0;
  for (int i = 0; i < 8; ++i) v |= (u64)o[i] << (8 * i);
  return v;
}
static_assert(ord_pack(OIN) == ord_pack(OCR) && ord_pack(OIN + 8) == ord_pack(OCR + 8), "common head");
__host__ __device__ __forceinline__ unsigned ord_item(unsigned k, unsigned d) {
  const u64 hi = d < 4u ? ((k & 32u) ? ord_pack(OIN + 32) : (k & 8u) ? ord_pack(OIN + 24) : ord_pack(OIN + 16))
                        : ((k & 8u) ? ord_pack(OCR + 24) : ord_pack(OCR + 16));
  const u64 a = k >= 16u ? hi : (k & 8u) ? ord_pack(OIN + 8) : ord_pack(OIN);
  unsigned it = (unsigned)(a >> (8u * (k & 7u))) & 0xffu;
  if (d >= 4u && (it >> 1) == d) it = 4u + (it & 1u);
  if (it >= G_) it = 32u - NF + 2u * (it - G_) + ((d >> 1) & 1u);
  return it;
}

struct alignas(8) Job {
  unsigned char tile;
  unsigned char early;
  unsigned char pos;
  unsigned char fin;
  unsigned char turn;
  unsigned char pos2;
  unsigned short plan;
};

struct Params {
  const char* x;
  const char* w;
  signed char* out;
  char* wsx[NR];
  int rank;
  int pad;
  Job job[NR * 32];
};

__device__ __forceinline__ u64 now() { return __builtin_amdgcn_s_memrealtime(); }
__device__ __forceinline__ void wait_until(u64 t) { while (now() < t) __builtin_amdgcn_s_sleep(1); }
#define VM0() asm volatile("s_waitcnt vmcnt(0)" ::: "memory")
#define BAR() do { asm volatile("" ::: "memory"); __builtin_amdgcn_s_waitcnt(0xC07F);   __builtin_amdgcn_s_barrier(); asm volatile("" ::: "memory"); } while (0)

__host__ __device__ __forceinline__ unsigned rowbase(unsigned q, unsigned xs) {
  const unsigned e = xs - 8u;
  const unsigned third = 512u * (q ^ (4u + (e >> 1))) + 256u * ((q >> 1) & 1u) + 128u * (e & 1u);
  const unsigned pair = 512u * (q ^ (xs < 4u ? 1u : 0u)) + 128u * (xs & 3u);
  return xs < 8u ? pair : third;
}

#define STAGE_ASM(SC)                                             \
  asm volatile(                                                   \
      "s_mov_b32 %0, m0\n\t"                                      \
      "s_mov_b32 m0, %11\n\t"                                     \
      "s_nop 2\n\t"                                               \
      "global_load_lds_dwordx4 %1, %9" SC "\n\t"                   \
      "global_load_lds_dwordx4 %2, %9 offset:1024" SC "\n\t"       \
      "global_load_lds_dwordx4 %3, %9 offset:2048" SC "\n\t"       \
      "global_load_lds_dwordx4 %4, %9 offset:3072" SC "\n\t"       \
      "s_add_u32 m0, m0, 0x4000\n\t"                              \
      "s_nop 0\n\t"                                               \
      "global_load_lds_dwordx4 %5, %10" SC "\n\t"                  \
      "global_load_lds_dwordx4 %6, %10 offset:1024" SC "\n\t"      \
      "global_load_lds_dwordx4 %7, %10 offset:2048" SC "\n\t"      \
      "global_load_lds_dwordx4 %8, %10 offset:3072" SC "\n\t"      \
      "s_mov_b32 m0, %0"                                          \
      : "=&s"(keep)                                               \
      : "v"(va[0]), "v"(va[1]), "v"(va[2]), "v"(va[3]), "v"(vb[0]), "v"(vb[1]), "v"(vb[2]), "v"(vb[3]), "s"(a), "s"(b), "s"(lds) \
      : "memory", "scc")
template <bool PEER>
__device__ __forceinline__ void stage_load(const char* a, const char* b, const unsigned (&va)[4], const unsigned (&vb)[4],
                                           unsigned lds) {
  unsigned keep;
  if (PEER) STAGE_ASM(" sc0 sc1"); else STAGE_ASM("");
}

__device__ __forceinline__ unsigned nat_x(unsigned c) { return (c >> 3) * 1024u + (((c & 7u) ^ ((c >> 4) & 7u)) << 4); }
__device__ __forceinline__ unsigned nat_w(unsigned c) {
  const unsigned kr = c >> 4;
  return kr * 8192u + (((c & 15u) ^ ((kr & 3u) | (kr & 8u))) << 4);
}

struct Half { s16x8 x[4], w[4]; };

__device__ __forceinline__ void read_half(Half& f, lchar* st, unsigned fx, const unsigned (&fw)[4], int kk) {
#pragma unroll
  for (int i = 0; i < 4; ++i) {
    f.x[i] = *(__attribute__((address_space(3))) s16x8*)(st + fx + i * 2048);
    s16x4 lo = __builtin_amdgcn_ds_read_tr16_b64_v4i16((__attribute__((address_space(3))) s16x4*)(st + fw[i] + kk * 8192));
    s16x4 hi = __builtin_amdgcn_ds_read_tr16_b64_v4i16((__attribute__((address_space(3))) s16x4*)(st + fw[i] + kk * 8192 + 1024));
    f.w[i] = __builtin_shufflevector(lo, hi, 0, 1, 2, 3, 4, 5, 6, 7);
  }
}
__device__ __forceinline__ void mma(f32x4 (&acc)[4][4], const Half& f) {
#pragma unroll
  for (int mb = 0; mb < 4; ++mb)
#pragma unroll
    for (int nb = 0; nb < 4; ++nb)
      acc[mb][nb] = __builtin_amdgcn_mfma_f32_16x16x32_bf16(__builtin_bit_cast(bf16x8, f.w[nb]),
                                                            __builtin_bit_cast(bf16x8, f.x[mb]), acc[mb][nb], 0, 0, 0);
}
__device__ __forceinline__ void interleave() {
#pragma unroll
  for (int i = 0; i < 12; ++i) {
    __builtin_amdgcn_sched_group_barrier(0x8, 1, 0);
    __builtin_amdgcn_sched_group_barrier(0x100, 1, 0);
  }
  __builtin_amdgcn_sched_group_barrier(0x8, 4, 0);
}

constexpr unsigned LDS_RED = NST * STAGE, LDS_BYTES = LDS_RED + 64;
__device__ __forceinline__ unsigned ring_next(unsigned o) { return o + STAGE == NST * STAGE ? 0u : o + STAGE; }
__device__ __forceinline__ u64 gemm_load(unsigned lds, unsigned lw, unsigned lane, const char* an, const char* bn,
                                         const char* as, const char* bs, int s0, int s1, unsigned srcs, gu64* next) {
  u64 mine = 0;
  unsigned nx[4], nw[4], ln[4];
#pragma unroll
  for (unsigned i = 0; i < 4; ++i) {
    const unsigned c = (lw * 4u + i) * 64u + lane;
    nx[i] = nat_x(c) - i * 1024u;
    nw[i] = nat_w(c) - i * 1024u;
    ln[i] = lw * 4096u + lane * 16u;
  }
  unsigned ring = 0;
  auto issue = [&](int s) {
    const unsigned slot = lds + lw * 4096u + ring;
    if (s < 8) {
      stage_load<false>(an + (unsigned)s * 128u, bn + (unsigned)s * (64u * 8192u), nx, nw, slot);
    } else {
      const unsigned ps = (unsigned)(s - 8), o = ((srcs >> (ps & ~3u)) & 7u) * SRC_STRIDE + (ps & 3u) * 16384u;
      stage_load<true>(as + o, bs + o, ln, ln, slot);
    }
    ring = ring_next(ring);
  };
  BAR();
  for (int s = s0; s < s0 + NST; ++s) issue(s);
  asm volatile("s_waitcnt vmcnt(%0)" ::"n"(8 * (NST - 1)) : "memory");
  BAR();
  for (int s = s0; s + NST < s1; ++s) {
    asm volatile("s_waitcnt vmcnt(16)" ::: "memory");
    BAR();
    issue(s + NST);
  }
  asm volatile("s_waitcnt vmcnt(16)" ::: "memory");
  BAR();
  if (next) {
    const u64 lo = now() + DRAW_LEAD, inc = PTAU;
    asm volatile("global_atomic_umax_x2 %1, %2, off\n\tglobal_atomic_add_x2 %0, %1, %3, off sc0"
                 : "=&v"(mine) : "v"(next), "v"(lo), "v"(inc) : "memory");
  }
  asm volatile("s_waitcnt vmcnt(8)" ::: "memory");
  BAR();
  asm volatile("s_waitcnt vmcnt(0)" : "+v"(mine)::"memory");
  BAR();
  return mine;
}
__device__ __forceinline__ void gemm_math(f32x4 (&acc)[4][4], lchar* sm, unsigned wv, unsigned lane, int s0, int s1) {
  const unsigned wm = wv >> 1, wn = wv & 1u, g = lane >> 4, m = lane & 15u;
  const unsigned fx0 = (64u * wm + m) * 128u + ((g ^ (m >> 1)) << 4), fx1 = fx0 ^ 64u;
  const unsigned qq = (lane >> 2) & 3u, pp = lane & 3u;
  const unsigned base = 16384u + (8u * g + qq) * 256u + (((8u * wn + 2u * pp) ^ (qq | ((g & 1u) << 3))) << 4) + ((pp & 1u) << 3);
  const unsigned fw[4] = {base, base ^ 8u, base ^ 16u, base ^ 24u};
  Half fa, fb;
  unsigned ring = 0;
  BAR();
  BAR();
  read_half(fa, sm, fx0, fw, 0);
  for (int s = s0; s + 1 < s1; ++s) {
    read_half(fb, sm + ring, fx1, fw, 1);
    mma(acc, fa);
    interleave();
    BAR();
    ring = ring_next(ring);
    read_half(fa, sm + ring, fx0, fw, 0);
    mma(acc, fb);
    interleave();
  }
  read_half(fb, sm + ring, fx1, fw, 1);
  mma(acc, fa);
  interleave();
  mma(acc, fb);
}

__device__ __forceinline__ bool give_up(gu32* tmo, unsigned call, u64 t0) {
  const unsigned v = __builtin_amdgcn_readfirstlane(__hip_atomic_load(tmo, __ATOMIC_RELAXED, __HIP_MEMORY_SCOPE_AGENT));
  if (v == call || now() - t0 > DEADLINE) {
    __hip_atomic_store(tmo, call, __ATOMIC_RELAXED, __HIP_MEMORY_SCOPE_AGENT);
    return true;
  }
  return false;
}

__device__ __forceinline__ void poll_flags(const char* f, const char* g, unsigned call, gu32* tmo) {
  const u64 t0 = now();
  for (unsigned it = 1;; ++it) {
    unsigned vf = 0, vg = 0;
    if (f) vf = __hip_atomic_load((gu32*)f, __ATOMIC_RELAXED, __HIP_MEMORY_SCOPE_SYSTEM);
    if (g) vg = __hip_atomic_load((gu32*)g, __ATOMIC_RELAXED, __HIP_MEMORY_SCOPE_SYSTEM);
    if (vf >= call) f = nullptr;
    if (vg >= call) g = nullptr;
    if (__all(f == nullptr && g == nullptr)) break;
    if ((it & 63u) == 0 && give_up(tmo, call, t0)) break;
    __builtin_amdgcn_s_sleep(PSLEEP);
  }
}

__device__ __forceinline__ unsigned pack_bf16(float a, float b) {
  f32x2 v = {a, b};
  return __builtin_bit_cast(unsigned, __builtin_convertvector(v, bf16x2));
}

__device__ __forceinline__ void add_partial(f32x4 (&acc)[4][4], const u32x4 (&v)[8]) {
#pragma unroll
  for (int i = 0; i < 8; ++i) {
#pragma unroll
    for (int h = 0; h < 2; ++h) {
      const unsigned u0 = v[i][2 * h], u1 = v[i][2 * h + 1];
      acc[i >> 1][(i & 1) * 2 + h] += f32x4{__uint_as_float(u0 << 16), __uint_as_float(u0 & 0xffff0000u),
                                            __uint_as_float(u1 << 16), __uint_as_float(u1 & 0xffff0000u)};
    }
  }
}

__device__ __forceinline__ unsigned wave_max(unsigned m) {
  m = max(m, (unsigned)__builtin_amdgcn_update_dpp(0, (int)m, 0x111, 0xf, 0xf, false));
  m = max(m, (unsigned)__builtin_amdgcn_update_dpp(0, (int)m, 0x112, 0xf, 0xf, false));
  m = max(m, (unsigned)__builtin_amdgcn_update_dpp(0, (int)m, 0x114, 0xf, 0xf, false));
  m = max(m, (unsigned)__builtin_amdgcn_update_dpp(0, (int)m, 0x118, 0xf, 0xf, false));
  return max(max((unsigned)__builtin_amdgcn_readlane((int)m, 15), (unsigned)__builtin_amdgcn_readlane((int)m, 31)),
             max((unsigned)__builtin_amdgcn_readlane((int)m, 47), (unsigned)__builtin_amdgcn_readlane((int)m, 63)));
}

__global__ void __launch_bounds__(NT, 2) fused_gemm_rs_q8(const Params p) {
  __shared__ __attribute__((aligned(1024))) char smem[LDS_BYTES];
  lchar* sm = (lchar*)smem;
  typedef __attribute__((address_space(3))) unsigned lu32;
  const u64 tstart = now();
  const unsigned t = threadIdx.x, lane = t & 63;
  const unsigned wv = __builtin_amdgcn_readfirstlane(t >> 6);
  unsigned vb = blockIdx.x;
  if ((vb & 6u) == 0u && (vb >> 5) == 2u) vb += 20u * 8u + 2u;
  else if ((vb & 6u) == 2u && (vb >> 5) == 7u) vb -= 20u * 8u + 2u;
  const unsigned d = vb & 7u, l = vb >> 3;
  u64 a_job = ((const u64*)p.job)[vb];
  char* a_peer = p.wsx[d == 0 ? 1u : d];
  char* a_me = p.wsx[0];
  const char* a_x = p.x;
  const char* a_w = p.w;
  signed char* a_out = p.out;
  unsigned a_rank = (unsigned)p.rank;
  asm volatile("" : "+s"(a_job), "+s"(a_peer), "+s"(a_me), "+s"(a_x), "+s"(a_w), "+s"(a_out), "+s"(a_rank));
  typedef __attribute__((address_space(1))) char gchar;
  char* const me = (char*)(gchar*)a_me;
  char* const peer = (char*)(gchar*)a_peer;
  const char* const px = (const char*)(const gchar*)a_x;
  const char* const pw = (const char*)(const gchar*)a_w;
  const Job job = __builtin_bit_cast(Job, a_job);
  const unsigned r = a_rank & 7u;
  const unsigned q = r ^ d;
  gu32* const ctr = (gu32*)(me + WS_CTR + blockIdx.x * 128);
  gu32* const tmo = (gu32*)(me + WS_TMO);
  unsigned prev = __hip_atomic_load(ctr, __ATOMIC_RELAXED, __HIP_MEMORY_SCOPE_AGENT);
  const unsigned j = r >> 2, xs = job.tile & 15u, n = job.tile >> 4;
  const unsigned fa = job.fin & 3u, fn = job.fin >> 2;
  const bool ships = d < 2u || d >= 4u;
  const bool detour = d < 2u && job.turn != 255u;
  const unsigned allsrc = d >= 4u ? 63u & ~(1u << (d - 2u)) : 63u;
  const unsigned ln0 = t * 16u;

  lu32* const red = (lu32*)(sm + LDS_RED);

  auto callno = [&]() -> unsigned {
    unsigned pv = prev;
    asm volatile("" : "+v"(pv));
    return __builtin_amdgcn_readfirstlane(pv) + 1u;
  };
  auto push = [&](unsigned k0, unsigned k1, unsigned kstep) {
      const unsigned u = l & 7u;
      const unsigned cq = (q ^ (d >= 4u ? (r & (q >> 1)) : 0u)) & 1u, jq = q >> 2;
      for (unsigned k = k0; k < k1; k += kstep) {
        unsigned it = ord_item(k, d), src = d - 2u;
        const unsigned sl = (u + k) & 7u;
        u32x4 v[8];
        if (it < 16u) {
          const unsigned xa = nat_x(lane), xb = nat_x(lane + 64u);
          const __amdgpu_buffer_rsrc_t rs = __builtin_amdgcn_make_buffer_rsrc((void*)px, 0, 4u << 20, 0x00020000);
          const unsigned off = (rowbase(q, it) + (sl & 1u) * 64u) * 1024u + cq * 512u + (sl >> 1) * 128u;
#pragma unroll
          for (unsigned i = 0; i < 8; ++i) v[i] = __builtin_amdgcn_raw_buffer_load_b128(rs, (i & 1u) ? xb : xa, off + (i >> 1) * 16384u, 0);
        } else if (k1 > 16u && it >= F_) {
          const unsigned ds = 4u | ((it & 1u) << 1) | ((r ^ d) & 1u);
          it -= NF;
          src = (ds ^ d) - 2u;
          wait_until(tstart + T0 + (k - 2u) * TAU);
          poll_flags(lane < 8u ? me + WS_FIN + ((ds - 2u) * 32u + it) * 128u + lane * 4u : nullptr, nullptr, callno(), tmo);
          const __amdgpu_buffer_rsrc_t rs = __builtin_amdgcn_make_buffer_rsrc(me + WS_IN + (size_t)((ds - 2u) * SRC_STRIDE + it * ITEM + sl * 8192u), 0, 8192u, 0x00020000);
          asm volatile("" ::: "memory");
#pragma unroll
          for (unsigned i = 0; i < 8; ++i) v[i] = __builtin_amdgcn_raw_buffer_load_b128(rs, lane * 16u, i * 1024u, AUX_SYS);
        } else {
          const unsigned wa = nat_w(lane), wb = nat_w(lane + 128u);
          const __amdgpu_buffer_rsrc_t rs = __builtin_amdgcn_make_buffer_rsrc((void*)pw, 0, 4u << 20, 0x00020000);
          const unsigned off = (cq * 256u + sl * 32u) * 8192u + (jq * 2048u + (it - 16u) * 128u) * 2u;
#pragma unroll
          for (unsigned i = 0; i < 8; ++i) v[i] = __builtin_amdgcn_raw_buffer_load_b128(rs, (i & 2u) ? wb : wa, off + (i & 5u) * 32768u, 0);
        }
        char* dst = peer + WS_IN + (size_t)(src * SRC_STRIDE + it * ITEM + sl * 8192u);
        const __amdgpu_buffer_rsrc_t rd = __builtin_amdgcn_make_buffer_rsrc(dst, 0, 8192u, 0x00020000);
        wait_until(tstart + T0 + k * TAU + u * (TAU / 8));
#pragma unroll
        for (unsigned i = 0; i < 8; ++i) __builtin_amdgcn_raw_buffer_store_b128(v[i], rd, lane * 16u, i * 1024u, AUX_SYS);
        VM0();
        const unsigned cn = callno();
        if (lane == 0)
          __hip_atomic_store((gu32*)(peer + WS_FIN + (src * 32u + it) * 128u + u * 4u), cn, __ATOMIC_RELAXED, __HIP_MEMORY_SCOPE_SYSTEM);
      }
  };
  if (t < 16u) red[t] = 0u;
  if (d >= 2u && l < 8u) push(wv, 16u, 8u);

  if (wv >= 4) {
    if ((job.plan & 15u) == 3u || (d >= 2u && l >= 8u)) wait_until(tstart + DELAY);
    for (unsigned plan = job.plan | 0xf0000u; (plan & 15u) != 15u; plan >>= 4) {
      const unsigned kind = plan & 15u;
      if (kind == 5u || kind == 6u) { push((kind == 5u ? 12u : 28u) + wv, d >= 4u ? NCR : kind == 5u ? 32u : NIN, 4u); continue; }
      const unsigned call = callno();
      const bool staged = kind == 1u || kind >= 7u;
      const unsigned mask = kind == 7u ? job.early : kind == 8u ? allsrc & ~job.early : allsrc;
      const int s0 = staged ? 8 : 0, s1 = staged ? 8 + 4 * __builtin_popcount(mask) : 8;
      const bool last = staged && kind != 7u && ships;
      if (staged && wv == 4) {
        const unsigned s = lane >> 3;
        const bool act = (mask >> s) & 1u;
        wait_until(tstart + T0 + (kind == 7u ? job.pos2 : job.pos) * TAU);
        poll_flags(act ? me + WS_FIN + (s * 32u + xs) * 128u + (lane & 7u) * 4u : nullptr,
                   act ? me + WS_FIN + (s * 32u + 16u + n) * 128u + (lane & 7u) * 4u : nullptr, call, tmo);
      }
      unsigned srcs = 0;
      for (unsigned s = 6u; s-- > 0u;) if ((mask >> s) & 1u) srcs = (srcs << 4) | s;
      const u64 mine = gemm_load((unsigned)(size_t)sm, wv - 4u, lane,
                px + (size_t)(kind == 3u ? 512u * r + 128u * fa : rowbase(r, xs)) * 1024u,
                pw + (kind == 3u ? (j ^ 1u) * 2048u + fn * 128u : j * 2048u + n * 128u) * 2u, me + WS_IN + (size_t)(xs * ITEM),
                me + WS_IN + (size_t)((16u + n) * ITEM), s0, s1, srcs,
                last && d < 2u && !detour && t == 256u ? (gu64*)(me + WS_TICK) : nullptr);
      if (last) {
        if (t == 256u) red[12] = (unsigned)(mine - tstart);
        BAR();
        u32x4 v[8];
#pragma unroll
        for (int i = 0; i < 8; ++i) v[i] = *(__attribute__((address_space(3))) u32x4*)(sm + (wv - 4u) * 8192u + i * 1024u + lane * 16u);
        int go = (int)__builtin_amdgcn_readfirstlane(red[12]);
        if (go > (int)(now() - tstart) + GATE + 64 * PTAU) go = 0;
        if (d >= 4u) go = GATE + (int)(job.turn ^ ((r & 1u) * 3u)) * PTAU;
        if (detour) go = RGATE + (int)job.turn * PTAU;
        const unsigned a = d < 2u ? xs : 2u * ((r >> 1) & 1u) + (xs & 1u);
        const unsigned slot = ((d < 2u ? (detour ? 3u : 0u) : 1u + (r & 1u)) * 4u + a) * 16u + n;
        char* const to = detour ? (char*)(gchar*)p.wsx[2u + d] : peer;
        const __amdgpu_buffer_rsrc_t rs = __builtin_amdgcn_make_buffer_rsrc(to + WS_P + (size_t)slot * PTILE + (wv - 4u) * 8192u, 0, 8192u, 0x00020000);
        if (go > 0) wait_until(tstart + (unsigned)go + (wv - 4u) * (PTAU / 4));
#pragma unroll
        for (int i = 0; i < 8; ++i) __builtin_amdgcn_raw_buffer_store_b128(v[i], rs, lane * 16u, i * 1024u, AUX_SYS);
        VM0();
        if (lane == 0) __hip_atomic_store((gu32*)(to + WS_FP + slot * 128u + (wv - 4u) * 4u), call, __ATOMIC_RELAXED, __HIP_MEMORY_SCOPE_SYSTEM);
      }
    }
    if (d < 4u) {
      const unsigned call = callno();
      if (detour) {
        const unsigned slot = xs * 16u + (n ^ 1u);
        char* const to = (char*)(gchar*)p.wsx[2u + d];
        poll_flags(lane == 0 ? me + WS_FP + (192u + slot) * 128u + (wv - 4u) * 4u : nullptr, nullptr, call, tmo);
        const __amdgpu_buffer_rsrc_t ri = __builtin_amdgcn_make_buffer_rsrc(me + WS_P + (size_t)(192u + slot) * PTILE + (wv - 4u) * 8192u, 0, 8192u, 0x00020000);
        const __amdgpu_buffer_rsrc_t rs = __builtin_amdgcn_make_buffer_rsrc(to + WS_P + (size_t)slot * PTILE + (wv - 4u) * 8192u, 0, 8192u, 0x00020000);
        u32x4 v[8];
        asm volatile("" ::: "memory");
#pragma unroll
        for (int i = 0; i < 8; ++i) v[i] = __builtin_amdgcn_raw_buffer_load_b128(ri, lane * 16u, i * 1024u, AUX_SYS);
        wait_until(tstart + RGATE + (4u + job.turn) * PTAU + (wv - 4u) * (PTAU / 4));
#pragma unroll
        for (int i = 0; i < 8; ++i) __builtin_amdgcn_raw_buffer_store_b128(v[i], rs, lane * 16u, i * 1024u, AUX_SYS);
        VM0();
        if (lane == 0) __hip_atomic_store((gu32*)(to + WS_FP + slot * 128u + (wv - 4u) * 4u), call, __ATOMIC_RELAXED, __HIP_MEMORY_SCOPE_SYSTEM);
      }
      u64 g = 0, h = t < 256u + NR ? 0ull : (u64)call << 32;
      wait_until(tstart + SWEEP_T);
      const u64 t0 = now();
      unsigned state = 1u;
      for (;;) {
        if ((unsigned)(g >> 32) != call) g = __hip_atomic_load((gu64*)(me + WS_G + (t - 256u) * 8u), __ATOMIC_RELAXED, __HIP_MEMORY_SCOPE_SYSTEM);
        if ((unsigned)(h >> 32) != call) h = __hip_atomic_load((gu64*)(me + WS_G + t * 8u), __ATOMIC_RELAXED, __HIP_MEMORY_SCOPE_SYSTEM);
        const unsigned tm = __hip_atomic_load(tmo, __ATOMIC_RELAXED, __HIP_MEMORY_SCOPE_AGENT);
        const bool gok = (unsigned)(g >> 32) == call, hok = (unsigned)(h >> 32) == call;
        const unsigned m = wave_max(max(gok ? (unsigned)g : 0u, hok ? (unsigned)h : 0u));
        if (lane == 0) __hip_atomic_store(red + wv, m, __ATOMIC_RELAXED, __HIP_MEMORY_SCOPE_WORKGROUP);
        if (__builtin_amdgcn_readfirstlane(tm) == call) { state = 2u; break; }
        if (__all(gok && hok)) break;
        if (now() - t0 > DEADLINE) {
          __hip_atomic_store(tmo, call, __ATOMIC_RELAXED, __HIP_MEMORY_SCOPE_AGENT);
          state = 2u;
          break;
        }
        asm volatile("" ::: "memory");
      }
      asm volatile("" ::: "memory");
      if (lane == 0) __hip_atomic_store(red + 4u + wv, state, __ATOMIC_RELAXED, __HIP_MEMORY_SCOPE_WORKGROUP);
    }
  } else {
    f32x4 acc[4][4];
#pragma unroll
    for (int i = 0; i < 4; ++i)
#pragma unroll
      for (int k = 0; k < 4; ++k) acc[i][k] = f32x4{0.f, 0.f, 0.f, 0.f};
    const bool collector = vb == COLLECTOR;
    char* gdst = p.wsx[lane & 7u] + WS_G + (collector ? 256u + r : r * 32u + (d & 2u) * 8u + (fn & 3u) * 4u + fa) * 8u;
    asm volatile("" : "+v"(gdst));
    const bool parks = (job.plan & 15u) == 3u;
    unsigned pend = d < 2u ? 3u : 1u;
    const __amdgpu_buffer_rsrc_t rpark = __builtin_amdgcn_make_buffer_rsrc(me + WS_PARK + (size_t)((d & 1u) * 32u + l) * 65536u, 0, 65536u, 0x00020000);
    for (unsigned plan = job.plan | 0xf0000u; (plan & 15u) != 15u; plan >>= 4) {
      const unsigned kind = plan & 15u;
      if (kind == 5u || kind == 6u) continue;
      const bool staged = kind == 1u || kind >= 7u;
      const unsigned mask = kind == 7u ? job.early : kind == 8u ? allsrc & ~job.early : allsrc;
      const int s0 = staged ? 8 : 0, s1 = staged ? 8 + 4 * __builtin_popcount(mask) : 8;
      const bool last = staged && kind != 7u && ships;
      if (!staged) {
#pragma unroll
        for (int i = 0; i < 4; ++i)
#pragma unroll
          for (int k = 0; k < 4; ++k) acc[i][k] = f32x4{0.f, 0.f, 0.f, 0.f};
      }
      gemm_math(acc, sm, wv, lane, s0, s1);
      if (kind == 3u && parks) {
#pragma unroll
        for (int i = 0; i < 16; ++i)
          __builtin_amdgcn_raw_buffer_store_b128(__builtin_bit_cast(u32x4, acc[i >> 2][i & 3]), rpark, ln0, i * 4096u, 0);
      }
      if (last) {
#pragma unroll
        for (int i = 0; i < 8; ++i) {
          const f32x4 c0 = acc[i >> 1][(i & 1) * 2], c1 = acc[i >> 1][(i & 1) * 2 + 1];
          *(__attribute__((address_space(3))) u32x4*)(sm + wv * 8192u + i * 1024u + lane * 16u) =
              u32x4{pack_bf16(c0[0], c0[1]), pack_bf16(c0[2], c0[3]), pack_bf16(c1[0], c1[1]), pack_bf16(c1[2], c1[3])};
        }
        BAR();
        if (parks) {
#pragma unroll
          for (int i = 0; i < 16; ++i)
            acc[i >> 2][i & 3] = __builtin_bit_cast(f32x4, __builtin_amdgcn_raw_buffer_load_b128(rpark, ln0, i * 4096u, 0));
        }
      }
    }
    const unsigned call = callno();
    if (t == 0) __hip_atomic_store(ctr, call, __ATOMIC_RELAXED, __HIP_MEMORY_SCOPE_AGENT);
    if (collector && wv == 0) {
      u64 g = lane < 48u ? 0ull : (u64)call << 32, h = g;
      wait_until(tstart + SWEEP_T - 400);
      const u64 t0 = now();
      for (unsigned it = 1;; ++it) {
        if ((unsigned)(g >> 32) != call) g = __hip_atomic_load((gu64*)(me + WS_GL + lane * 8u), __ATOMIC_RELAXED, __HIP_MEMORY_SCOPE_SYSTEM);
        if ((unsigned)(h >> 32) != call) h = __hip_atomic_load((gu64*)(me + WS_GL + 512u + lane * 8u), __ATOMIC_RELAXED, __HIP_MEMORY_SCOPE_SYSTEM);
        if (__all((unsigned)(g >> 32) == call && (unsigned)(h >> 32) == call)) break;
        if ((it & 63u) == 0 && give_up(tmo, call, t0)) break;
      }
      const unsigned m = wave_max(max((unsigned)g, (unsigned)h));
      if (lane < NR) __hip_atomic_store((gu64*)gdst, ((u64)call << 32) | m, __ATOMIC_RELAXED, __HIP_MEMORY_SCOPE_SYSTEM);
    }
    if (d < 4u) {
      const unsigned slot = ((d < 2u ? 4u : 0u) + fa) * 16u + fn;
      unsigned got = 0u;
      while (pend) {
        const u64 t0 = now();
        for (unsigned it = 1; !got; ++it) {
          unsigned f = 0;
          if (lane < 2u && ((pend >> lane) & 1u))
            f = __hip_atomic_load((gu32*)(me + WS_FP + (slot + lane * 64u) * 128u + wv * 4u), __ATOMIC_RELAXED, __HIP_MEMORY_SCOPE_SYSTEM);
          got = (unsigned)__ballot(f >= call) & pend;
          if (got) break;
          if ((it & 63u) == 0 && give_up(tmo, call, t0)) { got = pend; break; }
          __builtin_amdgcn_s_sleep(PSLEEP);
        }
        got = __builtin_amdgcn_readfirstlane(got);
        const unsigned w = got & 1u ? 0u : 1u;
        const __amdgpu_buffer_rsrc_t rs = __builtin_amdgcn_make_buffer_rsrc(me + WS_P + (size_t)(slot + w * 64u) * PTILE + wv * 8192u, 0, 8192u, 0x00020000);
        u32x4 v[8];
        asm volatile("" ::: "memory");
#pragma unroll
        for (int i = 0; i < 8; ++i) v[i] = __builtin_amdgcn_raw_buffer_load_b128(rs, lane * 16u, i * 1024u, AUX_SYS);
        add_partial(acc, v);
        pend &= ~(1u << w);
        got &= pend;
      }
      float mf = 0.f;
#pragma unroll
      for (int i = 0; i < 4; ++i)
#pragma unroll
        for (int k = 0; k < 4; ++k)
#pragma unroll
          for (int e = 0; e < 4; ++e) mf = fmaxf(mf, fabsf(acc[i][k][e]));
      const unsigned mw = wave_max(__float_as_uint(mf));
      unsigned arr = 0;
      if (lane == 0) {
        __hip_atomic_fetch_max(red, mw, __ATOMIC_RELAXED, __HIP_MEMORY_SCOPE_WORKGROUP);
        arr = __hip_atomic_fetch_add(red + 1, 1u, __ATOMIC_RELAXED, __HIP_MEMORY_SCOPE_WORKGROUP);
      }
      if (__builtin_amdgcn_readfirstlane(arr) == 3u && lane < (fn >= 12u ? NR : 1u)) {
        const u64 g = ((u64)call << 32) | __hip_atomic_load(red, __ATOMIC_RELAXED, __HIP_MEMORY_SCOPE_WORKGROUP);
        __hip_atomic_store((gu64*)(fn >= 12u ? gdst : me + WS_GL + ((d & 2u) * 32u + fn * 4u + fa) * 8u), g, __ATOMIC_RELAXED, __HIP_MEMORY_SCOPE_SYSTEM);
      }
      const unsigned jc = d < 2u ? (j ^ 1u) : j;
      const unsigned pp = lane >> 4;
      const __amdgpu_buffer_rsrc_t ro = __builtin_amdgcn_make_buffer_rsrc(a_out, 0, 512u * 4096u, 0x00020000);
      const unsigned oo = (128u * fa + 64u * (wv >> 1) + (lane & 15u)) * 4096u + jc * 2048u + fn * 128u + 64u * (wv & 1u) + 16u * pp;
      unsigned used = 0xffffffffu;
      for (;;) {
        const u32x4 sv = *(volatile __attribute__((address_space(3))) u32x4*)(red + 8);
        const u32x4 mv = *(volatile __attribute__((address_space(3))) u32x4*)(red + 4);
        const unsigned s0 = sv[0], s1 = sv[1], s2 = sv[2], s3 = sv[3], m0 = mv[0], m1 = mv[1], m2 = mv[2], m3 = mv[3];
        const bool fin = __builtin_amdgcn_readfirstlane(min(min(s0, s1), min(s2, s3))) != 0u;
        const bool bad = __builtin_amdgcn_readfirstlane(max(max(s0, s1), max(s2, s3))) == 2u;
        unsigned cur = __builtin_amdgcn_readfirstlane(max(max(m0, m1), max(m2, m3)));
        const bool seen = __builtin_amdgcn_readfirstlane(min(min(m0, m1), min(m2, m3))) != 0u;
        if (bad) cur = 0xfffffffeu;
        if (cur != used && (fin || (seen && cur >= mw))) {
          const float amax = __uint_as_float(cur);
          const float pre = amax < 1e-30f ? 0x1p64f : 1.0f;
          const float inv = amax > 0.f ? 127.0f / (amax * pre) : 0.f;
          const float magic = 12582912.0f;
          if (used != 0xffffffffu) VM0();
#pragma unroll
          for (int mb = 0; mb < 4; ++mb) {
            unsigned dw[4];
#pragma unroll
            for (int nb = 0; nb < 4; ++nb) {
              unsigned b[4];
#pragma unroll
              for (int e = 0; e < 4; ++e)
                b[e] = __float_as_uint(__builtin_amdgcn_fmed3f(__builtin_fmaf(acc[mb][nb][e] * pre, inv, magic), magic - 127.f, magic + 127.f));
              const unsigned lo = __builtin_amdgcn_perm(b[1], b[0], 0x0c0c0400u), hi = __builtin_amdgcn_perm(b[3], b[2], 0x04000c0cu);
              dw[nb] = bad ? 0x80808080u : (lo | hi);
            }
            const bool odd = pp & 1u;
            const u32x4 o = {odd ? dw[1] : dw[0], odd ? dw[0] : dw[1], odd ? dw[3] : dw[2], odd ? dw[2] : dw[3]};
            __builtin_amdgcn_raw_buffer_store_b128(o, ro, oo + mb * 65536u, 0, AUX_SYS);
          }
          used = cur;
        }
        if (fin) break;
        __builtin_amdgcn_s_sleep(1);
      }
    }
  }
}

void make_jobs(Job* job) {
  constexpr int RESTART = 100, SRC_T = 124;
  int slot[6][32];
  for (int s = 0; s < 6; ++s) {
    for (int it = 0; it < 32; ++it) slot[s][it] = 0;
    for (int k = 0; k < (s < 2 ? NIN : NCR); ++k)
      if (ord_item(k, s + 2) < F_) slot[s][ord_item(k, s + 2)] = k;
  }
  for (int k = 0; k < NIN; ++k) {
    const int e = (int)ord_item(k, 2) - F_;
    for (int s = 2; s < 6 && e >= 0; ++s)
      if ((((s + 2) >> 1) & 1) != (e & 1)) slot[s][32 - NF + e] = k;
  }
  for (int d = 0; d < 8; ++d) {
    Job jb[32];
    int key[32], key2[32], early[32], srt[32], idx[32], txs[32], tn[32];
    for (int m = 0; m < 32; ++m) {
      txs[m] = d < 4 ? (d < 2 ? 0 : 4) + (m & 3) : 8 + 2 * (d - 4) + (m >> 4);
      tn[m] = d < 4 ? 2 * (m >> 2) + (d & 1) : (m & 15);
      int at[6], lastk = 0, cnt = 0;
      for (int s = 0; s < 6; ++s) {
        at[s] = slot[s][txs[m]] > slot[s][16 + tn[m]] ? slot[s][txs[m]] : slot[s][16 + tn[m]];
        if (d >= 4 && s == d - 2) at[s] = -1;
        if (at[s] > lastk) lastk = at[s];
        cnt += at[s] >= 0;
      }
      int best = lastk * TAU + cnt * SRC_T;
      early[m] = 0;
      key2[m] = lastk;
      for (int c = 0; c < lastk; ++c) {
        int mk = 0, na = 0, ka = 0;
        for (int s = 0; s < 6; ++s)
          if (at[s] >= 0 && at[s] <= c) { mk |= 1 << s; ++na; if (at[s] > ka) ka = at[s]; }
        if (!na) continue;
        const int fa = ka * TAU + RESTART + na * SRC_T;
        const int fb = (fa > lastk * TAU ? fa : lastk * TAU) + RESTART + (cnt - na) * SRC_T;
        if (fb < best) { best = fb; early[m] = mk; key2[m] = ka; }
      }
      key[m] = lastk * 64 + m;
      srt[m] = (d == 2 || d == 3 ? key2[m] * TAU : best) * 64 + m;
      idx[m] = m;
    }
    for (int a = 0; a < 32; ++a)
      for (int b = a + 1; b < 32; ++b)
        if (srt[idx[b]] < srt[idx[a]]) { const int tmp = idx[a]; idx[a] = idx[b]; idx[b] = tmp; }
    for (int l = 0; l < 32; ++l) {
      const int m = idx[(l + 16) & 31];
      jb[l].tile = (unsigned char)(txs[m] | (tn[m] << 4));
      jb[l].early = (unsigned char)early[m];
      jb[l].pos = (unsigned char)(key[m] / 64);
      const int fm = d < 2 ? idx[31 - ((l + 16) & 31)] : m;
      jb[l].fin = (unsigned char)((txs[fm] & 3) | (tn[fm] << 2));
      const bool sp = early[m] != 0;
      if (d >= 2 && l >= 8 && l < 16) jb[l].plan = sp ? 0x8750 : 0xf150;
      else if (d >= 2 && d < 4 && l >= 16 && l < 24) jb[l].plan = sp ? 0x8670 : 0xf160;
      else if (d >= 2) jb[l].plan = sp ? 0xf870 : 0xff10;
      else if (tn[m] < EARLY_N) jb[l].plan = sp ? 0x3870 : 0xf310;
      else jb[l].plan = sp ? 0x8703 : 0xf103;
      jb[l].pos2 = (unsigned char)key2[m];
      jb[l].turn = (unsigned char)((l + 16) & 31);
      if (d < 2) {
        jb[l].turn = (unsigned char)(tn[m] == RELAY_N + d ? txs[m] & 3 : 255);
      }
      job[l * 8 + d] = jb[l];
    }
  }
}

}

extern "C" void kernel_launch(int rank, int nranks, ncclComm_t comm, hipStream_t stream, void* const* d_in,
                              const int* in_sizes, int n_in, void* d_out, int out_size, const int* in_dtypes,
                              int out_dtype, void* const* d_out_all, void* const* d_ws_all, size_t ws_bytes) {
  if (nranks != NR || (unsigned)rank >= (unsigned)NR || n_in < 2 || ws_bytes < WS_END || out_size != 512 * 4096 ||
      in_sizes[0] != 4096 * 512 || in_sizes[1] != 512 * 4096) {
    fprintf(stderr, "kernel_launch: unexpected problem shape\n");
    return;
  }
  Params p;
  memset(&p, 0, sizeof p);
  p.x = (const char*)d_in[0];
  p.w = (const char*)d_in[1];
  p.out = (signed char*)d_out;
  for (int i = 0; i < NR; ++i) p.wsx[i] = (char*)d_ws_all[rank ^ i];
  p.rank = rank;
  make_jobs(p.job);
  hipLaunchKernelGGL(fused_gemm_rs_q8, dim3(NWG), dim3(NT), 0, stream, p);
  HIP_CHECK(hipGetLastError());
}
```

```cpp
#include <hip/hip_runtime.h>
#include <rccl/rccl.h>
#include <cstdio>
#include <cstdint>
#include <cstring>
#include <type_traits>

#define HIP_CHECK(expr) do { hipError_t _e = (expr); if (_e != hipSuccess) { \
    fprintf(stderr, "HIP error %s at %s:%d\n", hipGetErrorString(_e), __FILE__, __LINE__); \
    return; } } while (0)

namespace {

typedef short s16x4 __attribute__((ext_vector_type(4)));
typedef short s16x8 __attribute__((ext_vector_type(8)));
typedef __bf16 bf16x8 __attribute__((ext_vector_type(8)));
typedef __bf16 bf16x2 __attribute__((ext_vector_type(2)));
typedef float f32x4 __attribute__((ext_vector_type(4)));
typedef float f32x2 __attribute__((ext_vector_type(2)));
typedef unsigned u32x4 __attribute__((ext_vector_type(4)));
typedef __attribute__((address_space(1))) unsigned gu32;
typedef __attribute__((address_space(1))) unsigned long long gu64;
typedef __attribute__((address_space(3))) char lchar;
typedef unsigned long long u64;

constexpr int NR = 8;
constexpr int NWG = 256, NT = 512;
constexpr unsigned ITEM = 65536;
constexpr unsigned PTILE = 32768;
constexpr unsigned STAGE = 32768;
constexpr int NST = 4;

constexpr size_t WS_CTR = 0;
constexpr size_t WS_TMO = 32 << 10;
constexpr size_t WS_TICK = 33 << 10;
constexpr size_t WS_FIN = 64 << 10;
constexpr size_t WS_FP = 96 << 10;
constexpr size_t WS_G = 128 << 10;
constexpr size_t WS_GL = 136 << 10;
constexpr size_t WS_IN = (2 << 20) + 65536;
constexpr unsigned SRC_STRIDE = (2u << 20) + 65536u;
constexpr size_t WS_P = 16 << 20;
constexpr size_t WS_PARK = 24 << 20;
constexpr size_t WS_END = 28 << 20;
static_assert(WS_IN + 6 * (size_t)SRC_STRIDE <= WS_P, "workspace layout");

constexpr u64 DEADLINE = 100000000ull;
constexpr int AUX_SYS = 17;

constexpr int T0 = 120;
constexpr int TAU = 115, TAU2 = 111, KNEE = 8;
constexpr int TAUI = 114;
constexpr int PTAUX = 56, PTAU = 58;
constexpr unsigned NFIRST = 2u;
constexpr int NF = 4;
constexpr int NIN = 32 + NF, NCR = 30 - NF / 2;
__host__ __device__ constexpr unsigned tslot(unsigned k, bool cross) {
  return T0 + (cross ? k * TAU - (k > KNEE ? (k - KNEE) * (TAU - TAU2) : 0u) : k * TAUI);
}
constexpr int GATE = tslot(NCR, true);
constexpr int RGATE = tslot(NIN, false);
constexpr int SWEEP_T = GATE + 32 * PTAUX + 150;
constexpr int DELAY = 300;
constexpr int DRAW_LEAD = 110;
constexpr int RELAY_N = 12;
constexpr int EARLY_N = 4;
constexpr unsigned COLLECTOR = 16 * 8 + 4;
constexpr int PSLEEP = 4;

constexpr unsigned char W_ = 16, F_ = 32, G_ = 40;
#define ORD_HEAD 0, W_ + 0, 1, W_ + 1, 2, 3, W_ + 2, W_ + 3, 8, W_ + 4, 10, 12, W_ + 5, 14, W_ + 6, 9, W_ + 7, 11, W_ + 8, 13, W_ + 9, 15
constexpr unsigned char OIN[40] = {ORD_HEAD, W_ + 10, W_ + 11, W_ + 12, W_ + 13, W_ + 14, W_ + 15, F_ + 0, F_ + 1, F_ + 2, F_ + 3, 4, 5, 6, 7};
constexpr unsigned char OCR[40] = {ORD_HEAD, G_ + 0, G_ + 1, W_ + 10, W_ + 11, 6, 7};
constexpr u64 ord_pack(const unsigned char* o) {
  u64 v = 0;
  for (int i = 0; i < 8; ++i) v |= (u64)o[i] << (8 * i);
  return v;
}
static_assert(OIN[27] < F_ && OIN[28] == F_ && OIN[32] < F_, "passed-on items are the fourth of a wave");
static_assert(ord_pack(OIN) == ord_pack(OCR) && ord_pack(OIN + 8) == ord_pack(OCR + 8), "common head");
__host__ __device__ __forceinline__ unsigned ord_item(unsigned k, unsigned d) {
  const u64 hi = d < 4u ? ((k & 32u) ? ord_pack(OIN + 32) : (k & 8u) ? ord_pack(OIN + 24) : ord_pack(OIN + 16))
                        : ((k & 8u) ? ord_pack(OCR + 24) : ord_pack(OCR + 16));
  const u64 a = k >= 16u ? hi : (k & 8u) ? ord_pack(OIN + 8) : ord_pack(OIN);
  unsigned it = (unsigned)(a >> (8u * (k & 7u))) & 0xffu;
  if (d >= 4u && (it >> 1) == d) it = 4u + (it & 1u);
  if (it >= G_) it = 32u - NF + 2u * (it - G_) + ((d >> 1) & 1u);
  return it;
}

struct alignas(8) Job {
  unsigned char tile;
  unsigned char early;
  unsigned char pos;
  unsigned char fin;
  unsigned char turn;
  unsigned char pos2;
  unsigned short plan;
};

struct Params {
  const char* x;
  const char* w;
  signed char* out;
  char* wsx[NR];
  int rank;
  int pad;
  Job job[NR * 32];
};

__device__ __forceinline__ u64 now() { return __builtin_amdgcn_s_memrealtime(); }
__device__ __forceinline__ void wait_until(u64 t) { while (now() < t) __builtin_amdgcn_s_sleep(1); }
#define VM0() asm volatile("s_waitcnt vmcnt(0)" ::: "memory")
#define BAR() do { asm volatile("" ::: "memory"); __builtin_amdgcn_s_waitcnt(0xC07F);   __builtin_amdgcn_s_barrier(); asm volatile("" ::: "memory"); } while (0)

__host__ __device__ __forceinline__ unsigned rowbase(unsigned q, unsigned xs) {
  const unsigned e = xs - 8u;
  const unsigned third = 512u * (q ^ (4u + (e >> 1))) + 256u * ((q >> 1) & 1u) + 128u * (e & 1u);
  const unsigned pair = 512u * (q ^ (xs < 4u ? 1u : 0u)) + 128u * (xs & 3u);
  return xs < 8u ? pair : third;
}

#define STAGE_ASM(SC)                                             \
  asm volatile(                                                   \
      "s_mov_b32 %0, m0\n\t"                                      \
      "s_mov_b32 m0, %11\n\t"                                     \
      "s_nop 2\n\t"                                               \
      "global_load_lds_dwordx4 %1, %9" SC "\n\t"                   \
      "global_load_lds_dwordx4 %2, %9 offset:1024" SC "\n\t"       \
      "global_load_lds_dwordx4 %3, %9 offset:2048" SC "\n\t"       \
      "global_load_lds_dwordx4 %4, %9 offset:3072" SC "\n\t"       \
      "s_add_u32 m0, m0, 0x4000\n\t"                              \
      "s_nop 0\n\t"                                               \
      "global_load_lds_dwordx4 %5, %10" SC "\n\t"                  \
      "global_load_lds_dwordx4 %6, %10 offset:1024" SC "\n\t"      \
      "global_load_lds_dwordx4 %7, %10 offset:2048" SC "\n\t"      \
      "global_load_lds_dwordx4 %8, %10 offset:3072" SC "\n\t"      \
      "s_mov_b32 m0, %0"                                          \
      : "=&s"(keep)                                               \
      : "v"(va[0]), "v"(va[1]), "v"(va[2]), "v"(va[3]), "v"(vb[0]), "v"(vb[1]), "v"(vb[2]), "v"(vb[3]), "s"(a), "s"(b), "s"(lds) \
      : "memory", "scc")
template <bool PEER>
__device__ __forceinline__ void stage_load(const char* a, const char* b, const unsigned (&va)[4], const unsigned (&vb)[4],
                                           unsigned lds) {
  unsigned keep;
  if (PEER) STAGE_ASM(" sc0 sc1"); else STAGE_ASM("");
}

__device__ __forceinline__ unsigned nat_x(unsigned c) { return (c >> 3) * 1024u + (((c & 7u) ^ ((c >> 4) & 7u)) << 4); }
__device__ __forceinline__ unsigned nat_w(unsigned c) {
  const unsigned kr = c >> 4;
  return kr * 8192u + (((c & 15u) ^ ((kr & 3u) | (kr & 8u))) << 4);
}

struct Half { s16x8 x[4], w[4]; };

__device__ __forceinline__ void read_half(Half& f, lchar* st, unsigned fx, const unsigned (&fw)[4], int kk) {
#pragma unroll
  for (int i = 0; i < 4; ++i) {
    f.x[i] = *(__attribute__((address_space(3))) s16x8*)(st + fx + i * 2048);
    s16x4 lo = __builtin_amdgcn_ds_read_tr16_b64_v4i16((__attribute__((address_space(3))) s16x4*)(st + fw[i] + kk * 8192));
    s16x4 hi = __builtin_amdgcn_ds_read_tr16_b64_v4i16((__attribute__((address_space(3))) s16x4*)(st + fw[i] + kk * 8192 + 1024));
    f.w[i] = __builtin_shufflevector(lo, hi, 0, 1, 2, 3, 4, 5, 6, 7);
  }
}
__device__ __forceinline__ void mma(f32x4 (&acc)[4][4], const Half& f) {
#pragma unroll
  for (int mb = 0; mb < 4; ++mb)
#pragma unroll
    for (int nb = 0; nb < 4; ++nb)
      acc[mb][nb] = __builtin_amdgcn_mfma_f32_16x16x32_bf16(__builtin_bit_cast(bf16x8, f.w[nb]),
                                                            __builtin_bit_cast(bf16x8, f.x[mb]), acc[mb][nb], 0, 0, 0);
}
__device__ __forceinline__ void interleave() {
#pragma unroll
  for (int i = 0; i < 12; ++i) {
    __builtin_amdgcn_sched_group_barrier(0x8, 1, 0);
    __builtin_amdgcn_sched_group_barrier(0x100, 1, 0);
  }
  __builtin_amdgcn_sched_group_barrier(0x8, 4, 0);
}

constexpr unsigned LDS_RED = NST * STAGE, LDS_BYTES = LDS_RED + 64;
__device__ __forceinline__ unsigned ring_next(unsigned o) { return o + STAGE == NST * STAGE ? 0u : o + STAGE; }
__device__ __forceinline__ u64 gemm_load(unsigned lds, unsigned lw, unsigned lane, const char* an, const char* bn,
                                         const char* as, const char* bs, int s0, int s1, unsigned srcs, gu64* next) {
  u64 mine = 0;
  unsigned nx[4], nw[4], ln[4];
#pragma unroll
  for (unsigned i = 0; i < 4; ++i) {
    const unsigned c = (lw * 4u + i) * 64u + lane;
    nx[i] = nat_x(c) - i * 1024u;
    nw[i] = nat_w(c) - i * 1024u;
    ln[i] = lw * 4096u + lane * 16u;
  }
  unsigned ring = 0;
  auto issue = [&](int s) {
    const unsigned slot = lds + lw * 4096u + ring;
    if (s < 8) {
      stage_load<false>(an + (unsigned)s * 128u, bn + (unsigned)s * (64u * 8192u), nx, nw, slot);
    } else {
      const unsigned ps = (unsigned)(s - 8), o = ((srcs >> (ps & ~3u)) & 7u) * SRC_STRIDE + (ps & 3u) * 16384u;
      stage_load<true>(as + o, bs + o, ln, ln, slot);
    }
    ring = ring_next(ring);
  };
  BAR();
  for (int s = s0; s < s0 + NST; ++s) issue(s);
  asm volatile("s_waitcnt vmcnt(%0)" ::"n"(8 * (NST - 1)) : "memory");
  BAR();
  for (int s = s0; s + NST < s1; ++s) {
    asm volatile("s_waitcnt vmcnt(16)" ::: "memory");
    BAR();
    issue(s + NST);
  }
  asm volatile("s_waitcnt vmcnt(16)" ::: "memory");
  BAR();
  if (next) {
    const u64 lo = now() + DRAW_LEAD, inc = PTAU;
    asm volatile("global_atomic_umax_x2 %1, %2, off\n\tglobal_atomic_add_x2 %0, %1, %3, off sc0"
                 : "=&v"(mine) : "v"(next), "v"(lo), "v"(inc) : "memory");
  }
  asm volatile("s_waitcnt vmcnt(8)" ::: "memory");
  BAR();
  asm volatile("s_waitcnt vmcnt(0)" : "+v"(mine)::"memory");
  BAR();
  return mine;
}
__device__ __forceinline__ void gemm_math(f32x4 (&acc)[4][4], lchar* sm, unsigned wv, unsigned lane, int s0, int s1) {
  const unsigned wm = wv >> 1, wn = wv & 1u, g = lane >> 4, m = lane & 15u;
  const unsigned fx0 = (64u * wm + m) * 128u + ((g ^ (m >> 1)) << 4), fx1 = fx0 ^ 64u;
  const unsigned qq = (lane >> 2) & 3u, pp = lane & 3u;
  const unsigned base = 16384u + (8u * g + qq) * 256u + (((8u * wn + 2u * pp) ^ (qq | ((g & 1u) << 3))) << 4) + ((pp & 1u) << 3);
  const unsigned fw[4] = {base, base ^ 8u, base ^ 16u, base ^ 24u};
  Half fa, fb;
  unsigned ring = 0;
  BAR();
  BAR();
  read_half(fa, sm, fx0, fw, 0);
  for (int s = s0; s + 1 < s1; ++s) {
    read_half(fb, sm + ring, fx1, fw, 1);
    mma(acc, fa);
    interleave();
    BAR();
    ring = ring_next(ring);
    read_half(fa, sm + ring, fx0, fw, 0);
    mma(acc, fb);
    interleave();
  }
  read_half(fb, sm + ring, fx1, fw, 1);
  mma(acc, fa);
  interleave();
  mma(acc, fb);
}

__device__ __forceinline__ bool give_up(gu32* tmo, unsigned call, u64 t0) {
  const unsigned v = __builtin_amdgcn_readfirstlane(__hip_atomic_load(tmo, __ATOMIC_RELAXED, __HIP_MEMORY_SCOPE_AGENT));
  if (v == call || now() - t0 > DEADLINE) {
    __hip_atomic_store(tmo, call, __ATOMIC_RELAXED, __HIP_MEMORY_SCOPE_AGENT);
    return true;
  }
  return false;
}

__device__ __forceinline__ void poll_flags(const char* f, const char* g, unsigned call, gu32* tmo) {
  const u64 t0 = now();
  for (unsigned it = 1;; ++it) {
    unsigned vf = 0, vg = 0;
    if (f) vf = __hip_atomic_load((gu32*)f, __ATOMIC_RELAXED, __HIP_MEMORY_SCOPE_SYSTEM);
    if (g) vg = __hip_atomic_load((gu32*)g, __ATOMIC_RELAXED, __HIP_MEMORY_SCOPE_SYSTEM);
    if (vf >= call) f = nullptr;
    if (vg >= call) g = nullptr;
    if (__all(f == nullptr && g == nullptr)) break;
    if ((it & 63u) == 0 && give_up(tmo, call, t0)) break;
    __builtin_amdgcn_s_sleep(PSLEEP);
  }
}

__device__ __forceinline__ unsigned pack_bf16(float a, float b) {
  f32x2 v = {a, b};
  return __builtin_bit_cast(unsigned, __builtin_convertvector(v, bf16x2));
}

__device__ __forceinline__ void add_partial(f32x4 (&acc)[4][4], const u32x4 (&v)[8]) {
#pragma unroll
  for (int i = 0; i < 8; ++i) {
#pragma unroll
    for (int h = 0; h < 2; ++h) {
      const unsigned u0 = v[i][2 * h], u1 = v[i][2 * h + 1];
      acc[i >> 1][(i & 1) * 2 + h] += f32x4{__uint_as_float(u0 << 16), __uint_as_float(u0 & 0xffff0000u),
                                            __uint_as_float(u1 << 16), __uint_as_float(u1 & 0xffff0000u)};
    }
  }
}

__device__ __forceinline__ unsigned wave_max(unsigned m) {
  m = max(m, (unsigned)__builtin_amdgcn_update_dpp(0, (int)m, 0x111, 0xf, 0xf, false));
  m = max(m, (unsigned)__builtin_amdgcn_update_dpp(0, (int)m, 0x112, 0xf, 0xf, false));
  m = max(m, (unsigned)__builtin_amdgcn_update_dpp(0, (int)m, 0x114, 0xf, 0xf, false));
  m = max(m, (unsigned)__builtin_amdgcn_update_dpp(0, (int)m, 0x118, 0xf, 0xf, false));
  return max(max((unsigned)__builtin_amdgcn_readlane((int)m, 15), (unsigned)__builtin_amdgcn_readlane((int)m, 31)),
             max((unsigned)__builtin_amdgcn_readlane((int)m, 47), (unsigned)__builtin_amdgcn_readlane((int)m, 63)));
}

__global__ void __launch_bounds__(NT, 2) fused_gemm_rs_q8(const Params p) {
  __shared__ __attribute__((aligned(1024))) char smem[LDS_BYTES];
  lchar* sm = (lchar*)smem;
  typedef __attribute__((address_space(3))) unsigned lu32;
  const u64 tstart = now();
  const unsigned t = threadIdx.x, lane = t & 63;
  const unsigned wv = __builtin_amdgcn_readfirstlane(t >> 6);
  unsigned vb = blockIdx.x;
  if ((vb & 6u) == 0u && (vb >> 5) == 2u) vb += 20u * 8u + 2u;
  else if ((vb & 6u) == 2u && (vb >> 5) == 7u) vb -= 20u * 8u + 2u;
  const unsigned d = vb & 7u, l = vb >> 3;
  u64 a_job = ((const u64*)p.job)[vb];
  char* a_peer = p.wsx[d == 0 ? 1u : d];
  char* a_me = p.wsx[0];
  const char* a_x = p.x;
  const char* a_w = p.w;
  signed char* a_out = p.out;
  unsigned a_rank = (unsigned)p.rank;
  asm volatile("" : "+s"(a_job), "+s"(a_peer), "+s"(a_me), "+s"(a_x), "+s"(a_w), "+s"(a_out), "+s"(a_rank));
  typedef __attribute__((address_space(1))) char gchar;
  char* const me = (char*)(gchar*)a_me;
  char* const peer = (char*)(gchar*)a_peer;
  const char* const px = (const char*)(const gchar*)a_x;
  const char* const pw = (const char*)(const gchar*)a_w;
  const Job job = __builtin_bit_cast(Job, a_job);
  const unsigned r = a_rank & 7u;
  const unsigned q = r ^ d;
  gu32* const ctr = (gu32*)(me + WS_CTR + blockIdx.x * 128);
  gu32* const tmo = (gu32*)(me + WS_TMO);
  unsigned prev = __hip_atomic_load(ctr, __ATOMIC_RELAXED, __HIP_MEMORY_SCOPE_AGENT);
  const unsigned j = r >> 2, xs = job.tile & 15u, n = job.tile >> 4;
  const unsigned fa = job.fin & 3u, fn = job.fin >> 2;
  const bool ships = d < 2u || d >= 4u;
  const bool detour = d < 2u && job.turn != 255u;
  const unsigned allsrc = d >= 4u ? 63u & ~(1u << (d - 2u)) : 63u;
  const unsigned ln0 = t * 16u;

  lu32* const red = (lu32*)(sm + LDS_RED);

  auto callno = [&]() -> unsigned {
    unsigned pv = prev;
    asm volatile("" : "+v"(pv));
    return __builtin_amdgcn_readfirstlane(pv) + 1u;
  };
  auto push = [&](auto ni, unsigned k0, unsigned k1, unsigned kstep) {
    constexpr unsigned NI = decltype(ni)::value;
    const unsigned u = l & 7u;
    const unsigned cq = (q ^ (d >= 4u ? (r & (q >> 1)) : 0u)) & 1u, jq = q >> 2;
    const unsigned xa = nat_x(lane), xb = nat_x(lane + 64u);
    const unsigned wa = nat_w(lane), wb = nat_w(lane + 128u);
    u32x4 v[NI][8];
#pragma unroll
    for (unsigned c = 0; c < NI; ++c) {
      const unsigned k = k0 + c * kstep, it = ord_item(k, d), sl = (u + k) & 7u;
      const bool isx = it < 16u;
      const __amdgpu_buffer_rsrc_t rs = __builtin_amdgcn_make_buffer_rsrc((void*)(isx ? px : pw), 0, k < k1 && it < F_ ? 4u << 20 : 0u, 0x00020000);
      const unsigned off = isx ? (rowbase(q, it) + (sl & 1u) * 64u) * 1024u + cq * 512u + (sl >> 1) * 128u
                               : (cq * 256u + sl * 32u) * 8192u + (jq * 2048u + (it - 16u) * 128u) * 2u;
      const unsigned v0 = isx ? xa : wa, v1 = isx ? xb : wa, v2 = isx ? xa : wb, v3 = isx ? xb : wb;
      const unsigned s1 = isx ? 0u : 32768u, s2 = isx ? 16384u : 0u, s4 = isx ? 32768u : 131072u;
#pragma unroll
      for (unsigned i = 0; i < 8; ++i)
        v[c][i] = __builtin_amdgcn_raw_buffer_load_b128(rs, (i & 2u) ? ((i & 1u) ? v3 : v2) : ((i & 1u) ? v1 : v0),
                                                        off + (i & 1u) * s1 + ((i >> 1) & 1u) * s2 + (i >> 2) * s4, 0);
    }
#pragma unroll
    for (unsigned c = 0; c < NI; ++c) {
      const unsigned k = k0 + c * kstep;
      if (k >= k1) break;
      unsigned it = ord_item(k, d), src = d - 2u;
      const unsigned sl = (u + k) & 7u;
      if (NI == 4 && c == 3 && it >= F_) {
        const unsigned ds = 4u | ((it & 1u) << 1) | ((r ^ d) & 1u);
        it -= NF;
        src = (ds ^ d) - 2u;
        wait_until(tstart + tslot(k - 2u, false));
        poll_flags(lane < 8u ? me + WS_FIN + ((ds - 2u) * 32u + it) * 128u + lane * 4u : nullptr, nullptr, callno(), tmo);
        const __amdgpu_buffer_rsrc_t rs = __builtin_amdgcn_make_buffer_rsrc(me + WS_IN + (size_t)((ds - 2u) * SRC_STRIDE + it * ITEM + sl * 8192u), 0, 8192u, 0x00020000);
        asm volatile("" ::: "memory");
#pragma unroll
        for (unsigned i = 0; i < 8; ++i) v[c][i] = __builtin_amdgcn_raw_buffer_load_b128(rs, lane * 16u, i * 1024u, AUX_SYS);
      }
      char* dst = peer + WS_IN + (size_t)(src * SRC_STRIDE + it * ITEM + sl * 8192u);
      const __amdgpu_buffer_rsrc_t rd = __builtin_amdgcn_make_buffer_rsrc(dst, 0, 8192u, 0x00020000);
      wait_until(tstart + tslot(k, d >= 4u) + u * (TAU / 8));
#pragma unroll
      for (unsigned i = 0; i < 8; ++i) __builtin_amdgcn_raw_buffer_store_b128(v[c][i], rd, lane * 16u, i * 1024u, AUX_SYS);
      VM0();
      const unsigned cn = callno();
      if (lane == 0)
        __hip_atomic_store((gu32*)(peer + WS_FIN + (src * 32u + it) * 128u + u * 4u), cn, __ATOMIC_RELAXED, __HIP_MEMORY_SCOPE_SYSTEM);
    }
  };
  if (t < 16u) red[t] = 0u;
  if (d >= 2u && l < 16u)
    for (unsigned kk = wv; kk < (l < 8u ? 16u : 8u); kk += 8u) {
      if (kk < 8u && (kk < NFIRST) != (l < 8u)) continue;
      if (kk >= 8u) wait_until(tstart + T0 + NFIRST * TAU + 80);
      push(std::integral_constant<unsigned, 1>(), kk, 16u, 8u);
    }

  if (wv >= 4) {
    if ((job.plan & 15u) == 3u || (d >= 2u && l >= 8u)) wait_until(tstart + DELAY);
    for (unsigned plan = job.plan | 0xf0000u; (plan & 15u) != 15u; plan >>= 4) {
      const unsigned kind = plan & 15u;
      if (kind == 5u || kind == 6u) { push(std::integral_constant<unsigned, 4>(), (kind == 5u ? 12u : 28u) + wv, d >= 4u ? NCR : kind == 5u ? 32u : NIN, 4u); continue; }
      const unsigned call = callno();
      const bool staged = kind == 1u || kind >= 7u;
      const unsigned mask = kind == 7u ? job.early : kind == 8u ? allsrc & ~job.early : allsrc;
      const int s0 = staged ? 8 : 0, s1 = staged ? 8 + 4 * __builtin_popcount(mask) : 8;
      const bool last = staged && kind != 7u && ships;
      if (staged && wv == 4) {
        const unsigned s = lane >> 3;
        const bool act = (mask >> s) & 1u;
        wait_until(tstart + tslot(kind == 7u ? job.pos2 : job.pos, true));
        poll_flags(act ? me + WS_FIN + (s * 32u + xs) * 128u + (lane & 7u) * 4u : nullptr,
                   act ? me + WS_FIN + (s * 32u + 16u + n) * 128u + (lane & 7u) * 4u : nullptr, call, tmo);
      }
      unsigned srcs = 0;
      for (unsigned s = 6u; s-- > 0u;) if ((mask >> s) & 1u) srcs = (srcs << 4) | s;
      const u64 mine = gemm_load((unsigned)(size_t)sm, wv - 4u, lane,
                px + (size_t)(kind == 3u ? 512u * r + 128u * fa : rowbase(r, xs)) * 1024u,
                pw + (kind == 3u ? (j ^ 1u) * 2048u + fn * 128u : j * 2048u + n * 128u) * 2u, me + WS_IN + (size_t)(xs * ITEM),
                me + WS_IN + (size_t)((16u + n) * ITEM), s0, s1, srcs,
                last && d < 2u && !detour && t == 256u ? (gu64*)(me + WS_TICK) : nullptr);
      if (last) {
        if (t == 256u) red[12] = (unsigned)(mine - tstart);
        BAR();
        u32x4 v[8];
#pragma unroll
        for (int i = 0; i < 8; ++i) v[i] = *(__attribute__((address_space(3))) u32x4*)(sm + (wv - 4u) * 8192u + i * 1024u + lane * 16u);
        int go = (int)__builtin_amdgcn_readfirstlane(red[12]);
        if (go > (int)(now() - tstart) + GATE + 64 * PTAU) go = 0;
        if (d >= 4u) go = GATE + (int)(job.turn ^ ((r & 1u) * 3u)) * PTAUX;
        if (detour) go = RGATE + (int)job.turn * PTAU;
        const unsigned a = d < 2u ? xs : 2u * ((r >> 1) & 1u) + (xs & 1u);
        const unsigned slot = ((d < 2u ? (detour ? 3u : 0u) : 1u + (r & 1u)) * 4u + a) * 16u + n;
        char* const to = detour ? (char*)(gchar*)p.wsx[2u + d] : peer;
        const __amdgpu_buffer_rsrc_t rs = __builtin_amdgcn_make_buffer_rsrc(to + WS_P + (size_t)slot * PTILE + (wv - 4u) * 8192u, 0, 8192u, 0x00020000);
        if (go > 0) wait_until(tstart + (unsigned)go + (wv - 4u) * (PTAU / 4));
#pragma unroll
        for (int i = 0; i < 8; ++i) __builtin_amdgcn_raw_buffer_store_b128(v[i], rs, lane * 16u, i * 1024u, AUX_SYS);
        VM0();
        if (lane == 0) __hip_atomic_store((gu32*)(to + WS_FP + slot * 128u + (wv - 4u) * 4u), call, __ATOMIC_RELAXED, __HIP_MEMORY_SCOPE_SYSTEM);
      }
    }
    if (d < 4u) {
      const unsigned call = callno();
      if (detour) {
        const unsigned slot = xs * 16u + (n ^ 1u);
        char* const to = (char*)(gchar*)p.wsx[2u + d];
        poll_flags(lane == 0 ? me + WS_FP + (192u + slot) * 128u + (wv - 4u) * 4u : nullptr, nullptr, call, tmo);
        const __amdgpu_buffer_rsrc_t ri = __builtin_amdgcn_make_buffer_rsrc(me + WS_P + (size_t)(192u + slot) * PTILE + (wv - 4u) * 8192u, 0, 8192u, 0x00020000);
        const __amdgpu_buffer_rsrc_t rs = __builtin_amdgcn_make_buffer_rsrc(to + WS_P + (size_t)slot * PTILE + (wv - 4u) * 8192u, 0, 8192u, 0x00020000);
        u32x4 v[8];
        asm volatile("" ::: "memory");
#pragma unroll
        for (int i = 0; i < 8; ++i) v[i] = __builtin_amdgcn_raw_buffer_load_b128(ri, lane * 16u, i * 1024u, AUX_SYS);
        wait_until(tstart + RGATE + (4u + job.turn) * PTAU + (wv - 4u) * (PTAU / 4));
#pragma unroll
        for (int i = 0; i < 8; ++i) __builtin_amdgcn_raw_buffer_store_b128(v[i], rs, lane * 16u, i * 1024u, AUX_SYS);
        VM0();
        if (lane == 0) __hip_atomic_store((gu32*)(to + WS_FP + slot * 128u + (wv - 4u) * 4u), call, __ATOMIC_RELAXED, __HIP_MEMORY_SCOPE_SYSTEM);
      }
      u64 g = 0, h = t < 256u + NR ? 0ull : (u64)call << 32;
      wait_until(tstart + SWEEP_T);
      const u64 t0 = now();
      unsigned state = 1u;
      for (;;) {
        if ((unsigned)(g >> 32) != call) g = __hip_atomic_load((gu64*)(me + WS_G + (t - 256u) * 8u), __ATOMIC_RELAXED, __HIP_MEMORY_SCOPE_SYSTEM);
        if ((unsigned)(h >> 32) != call) h = __hip_atomic_load((gu64*)(me + WS_G + t * 8u), __ATOMIC_RELAXED, __HIP_MEMORY_SCOPE_SYSTEM);
        const unsigned tm = __hip_atomic_load(tmo, __ATOMIC_RELAXED, __HIP_MEMORY_SCOPE_AGENT);
        const bool gok = (unsigned)(g >> 32) == call, hok = (unsigned)(h >> 32) == call;
        const unsigned m = wave_max(max(gok ? (unsigned)g : 0u, hok ? (unsigned)h : 0u));
        if (lane == 0) __hip_atomic_store(red + wv, m, __ATOMIC_RELAXED, __HIP_MEMORY_SCOPE_WORKGROUP);
        if (__builtin_amdgcn_readfirstlane(tm) == call) { state = 2u; break; }
        if (__all(gok && hok)) break;
        if (now() - t0 > DEADLINE) {
          __hip_atomic_store(tmo, call, __ATOMIC_RELAXED, __HIP_MEMORY_SCOPE_AGENT);
          state = 2u;
          break;
        }
        asm volatile("" ::: "memory");
      }
      asm volatile("" ::: "memory");
      if (lane == 0) __hip_atomic_store(red + 4u + wv, state, __ATOMIC_RELAXED, __HIP_MEMORY_SCOPE_WORKGROUP);
    }
  } else {
    f32x4 acc[4][4];
#pragma unroll
    for (int i = 0; i < 4; ++i)
#pragma unroll
      for (int k = 0; k < 4; ++k) acc[i][k] = f32x4{0.f, 0.f, 0.f, 0.f};
    const bool collector = vb == COLLECTOR;
    char* gdst = p.wsx[lane & 7u] + WS_G + (collector ? 256u + r : r * 32u + (d & 2u) * 8u + (fn & 3u) * 4u + fa) * 8u;
    asm volatile("" : "+v"(gdst));
    const bool parks = (job.plan & 15u) == 3u;
    unsigned pend = d < 2u ? 3u : 1u;
    const __amdgpu_buffer_rsrc_t rpark = __builtin_amdgcn_make_buffer_rsrc(me + WS_PARK + (size_t)((d & 1u) * 32u + l) * 65536u, 0, 65536u, 0x00020000);
    for (unsigned plan = job.plan | 0xf0000u; (plan & 15u) != 15u; plan >>= 4) {
      const unsigned kind = plan & 15u;
      if (kind == 5u || kind == 6u) continue;
      const bool staged = kind == 1u || kind >= 7u;
      const unsigned mask = kind == 7u ? job.early : kind == 8u ? allsrc & ~job.early : allsrc;
      const int s0 = staged ? 8 : 0, s1 = staged ? 8 + 4 * __builtin_popcount(mask) : 8;
      const bool last = staged && kind != 7u && ships;
      if (!staged) {
#pragma unroll
        for (int i = 0; i < 4; ++i)
#pragma unroll
          for (int k = 0; k < 4; ++k) acc[i][k] = f32x4{0.f, 0.f, 0.f, 0.f};
      }
      gemm_math(acc, sm, wv, lane, s0, s1);
      if (kind == 3u && parks) {
#pragma unroll
        for (int i = 0; i < 16; ++i)
          __builtin_amdgcn_raw_buffer_store_b128(__builtin_bit_cast(u32x4, acc[i >> 2][i & 3]), rpark, ln0, i * 4096u, 0);
      }
      if (last) {
#pragma unroll
        for (int i = 0; i < 8; ++i) {
          const f32x4 c0 = acc[i >> 1][(i & 1) * 2], c1 = acc[i >> 1][(i & 1) * 2 + 1];
          *(__attribute__((address_space(3))) u32x4*)(sm + wv * 8192u + i * 1024u + lane * 16u) =
              u32x4{pack_bf16(c0[0], c0[1]), pack_bf16(c0[2], c0[3]), pack_bf16(c1[0], c1[1]), pack_bf16(c1[2], c1[3])};
        }
        BAR();
        if (parks) {
#pragma unroll
          for (int i = 0; i < 16; ++i)
            acc[i >> 2][i & 3] = __builtin_bit_cast(f32x4, __builtin_amdgcn_raw_buffer_load_b128(rpark, ln0, i * 4096u, 0));
        }
      }
    }
    const unsigned call = callno();
    if (t == 0) __hip_atomic_store(ctr, call, __ATOMIC_RELAXED, __HIP_MEMORY_SCOPE_AGENT);
    if (collector && wv == 0) {
      u64 g = lane < 48u ? 0ull : (u64)call << 32, h = g;
      wait_until(tstart + SWEEP_T - 400);
      const u64 t0 = now();
      for (unsigned it = 1;; ++it) {
        if ((unsigned)(g >> 32) != call) g = __hip_atomic_load((gu64*)(me + WS_GL + lane * 8u), __ATOMIC_RELAXED, __HIP_MEMORY_SCOPE_SYSTEM);
        if ((unsigned)(h >> 32) != call) h = __hip_atomic_load((gu64*)(me + WS_GL + 512u + lane * 8u), __ATOMIC_RELAXED, __HIP_MEMORY_SCOPE_SYSTEM);
        if (__all((unsigned)(g >> 32) == call && (unsigned)(h >> 32) == call)) break;
        if ((it & 63u) == 0 && give_up(tmo, call, t0)) break;
      }
      const unsigned m = wave_max(max((unsigned)g, (unsigned)h));
      if (lane < NR) __hip_atomic_store((gu64*)gdst, ((u64)call << 32) | m, __ATOMIC_RELAXED, __HIP_MEMORY_SCOPE_SYSTEM);
    }
    if (d < 4u) {
      const unsigned slot = ((d < 2u ? 4u : 0u) + fa) * 16u + fn;
      unsigned got = 0u;
      while (pend) {
        const u64 t0 = now();
        for (unsigned it = 1; !got; ++it) {
          unsigned f = 0;
          if (lane < 2u && ((pend >> lane) & 1u))
            f = __hip_atomic_load((gu32*)(me + WS_FP + (slot + lane * 64u) * 128u + wv * 4u), __ATOMIC_RELAXED, __HIP_MEMORY_SCOPE_SYSTEM);
          got = (unsigned)__ballot(f >= call) & pend;
          if (got) break;
          if ((it & 63u) == 0 && give_up(tmo, call, t0)) { got = pend; break; }
          __builtin_amdgcn_s_sleep(PSLEEP);
        }
        got = __builtin_amdgcn_readfirstlane(got);
        const unsigned w = got & 1u ? 0u : 1u;
        const __amdgpu_buffer_rsrc_t rs = __builtin_amdgcn_make_buffer_rsrc(me + WS_P + (size_t)(slot + w * 64u) * PTILE + wv * 8192u, 0, 8192u, 0x00020000);
        u32x4 v[8];
        asm volatile("" ::: "memory");
#pragma unroll
        for (int i = 0; i < 8; ++i) v[i] = __builtin_amdgcn_raw_buffer_load_b128(rs, lane * 16u, i * 1024u, AUX_SYS);
        add_partial(acc, v);
        pend &= ~(1u << w);
        got &= pend;
      }
      float mf = 0.f;
#pragma unroll
      for (int i = 0; i < 4; ++i)
#pragma unroll
        for (int k = 0; k < 4; ++k)
#pragma unroll
          for (int e = 0; e < 4; ++e) mf = fmaxf(mf, fabsf(acc[i][k][e]));
      const unsigned mw = wave_max(__float_as_uint(mf));
      unsigned arr = 0;
      if (lane == 0) {
        __hip_atomic_fetch_max(red, mw, __ATOMIC_RELAXED, __HIP_MEMORY_SCOPE_WORKGROUP);
        arr = __hip_atomic_fetch_add(red + 1, 1u, __ATOMIC_RELAXED, __HIP_MEMORY_SCOPE_WORKGROUP);
      }
      if (__builtin_amdgcn_readfirstlane(arr) == 3u && lane < (fn >= 12u ? NR : 1u)) {
        const u64 g = ((u64)call << 32) | __hip_atomic_load(red, __ATOMIC_RELAXED, __HIP_MEMORY_SCOPE_WORKGROUP);
        __hip_atomic_store((gu64*)(fn >= 12u ? gdst : me + WS_GL + ((d & 2u) * 32u + fn * 4u + fa) * 8u), g, __ATOMIC_RELAXED, __HIP_MEMORY_SCOPE_SYSTEM);
      }
      const unsigned jc = d < 2u ? (j ^ 1u) : j;
      const unsigned pp = lane >> 4;
      const __amdgpu_buffer_rsrc_t ro = __builtin_amdgcn_make_buffer_rsrc(a_out, 0, 512u * 4096u, 0x00020000);
      const unsigned oo = (128u * fa + 64u * (wv >> 1) + (lane & 15u)) * 4096u + jc * 2048u + fn * 128u + 64u * (wv & 1u) + 16u * pp;
      unsigned used = 0xffffffffu;
      for (;;) {
        const u32x4 sv = *(volatile __attribute__((address_space(3))) u32x4*)(red + 8);
        const u32x4 mv = *(volatile __attribute__((address_space(3))) u32x4*)(red + 4);
        const unsigned s0 = sv[0], s1 = sv[1], s2 = sv[2], s3 = sv[3], m0 = mv[0], m1 = mv[1], m2 = mv[2], m3 = mv[3];
        const bool fin = __builtin_amdgcn_readfirstlane(min(min(s0, s1), min(s2, s3))) != 0u;
        const bool bad = __builtin_amdgcn_readfirstlane(max(max(s0, s1), max(s2, s3))) == 2u;
        unsigned cur = __builtin_amdgcn_readfirstlane(max(max(m0, m1), max(m2, m3)));
        const bool seen = __builtin_amdgcn_readfirstlane(min(min(m0, m1), min(m2, m3))) != 0u;
        if (bad) cur = 0xfffffffeu;
        if (cur != used && (fin || (seen && cur >= mw))) {
          const float amax = __uint_as_float(cur);
          const float pre = amax < 1e-30f ? 0x1p64f : 1.0f;
          const float inv = amax > 0.f ? 127.0f / (amax * pre) : 0.f;
          const float magic = 12582912.0f;
          if (used != 0xffffffffu) VM0();
#pragma unroll
          for (int mb = 0; mb < 4; ++mb) {
            unsigned dw[4];
#pragma unroll
            for (int nb = 0; nb < 4; ++nb) {
              unsigned b[4];
#pragma unroll
              for (int e = 0; e < 4; ++e)
                b[e] = __float_as_uint(__builtin_amdgcn_fmed3f(__builtin_fmaf(acc[mb][nb][e] * pre, inv, magic), magic - 127.f, magic + 127.f));
              const unsigned lo = __builtin_amdgcn_perm(b[1], b[0], 0x0c0c0400u), hi = __builtin_amdgcn_perm(b[3], b[2], 0x04000c0cu);
              dw[nb] = bad ? 0x80808080u : (lo | hi);
            }
            const bool odd = pp & 1u;
            const u32x4 o = {odd ? dw[1] : dw[0], odd ? dw[0] : dw[1], odd ? dw[3] : dw[2], odd ? dw[2] : dw[3]};
            __builtin_amdgcn_raw_buffer_store_b128(o, ro, oo + mb * 65536u, 0, AUX_SYS);
          }
          used = cur;
        }
        if (fin) break;
        __builtin_amdgcn_s_sleep(1);
      }
    }
  }
}

void make_jobs(Job* job) {
  constexpr int RESTART = 100, SRC_T = 124;
  int slot[6][32];
  for (int s = 0; s < 6; ++s) {
    for (int it = 0; it < 32; ++it) slot[s][it] = 0;
    for (int k = 0; k < (s < 2 ? NIN : NCR); ++k)
      if (ord_item(k, s + 2) < F_) slot[s][ord_item(k, s + 2)] = k;
  }
  for (int k = 0; k < NIN; ++k) {
    const int e = (int)ord_item(k, 2) - F_;
    for (int s = 2; s < 6 && e >= 0; ++s)
      if ((((s + 2) >> 1) & 1) != (e & 1)) slot[s][32 - NF + e] = k;
  }
  for (int d = 0; d < 8; ++d) {
    Job jb[32];
    int key[32], key2[32], early[32], srt[32], idx[32], txs[32], tn[32];
    for (int m = 0; m < 32; ++m) {
      txs[m] = d < 4 ? (d < 2 ? 0 : 4) + (m & 3) : 8 + 2 * (d - 4) + (m >> 4);
      tn[m] = d < 4 ? 2 * (m >> 2) + (d & 1) : (m & 15);
      int at[6], lastk = 0, cnt = 0;
      for (int s = 0; s < 6; ++s) {
        at[s] = slot[s][txs[m]] > slot[s][16 + tn[m]] ? slot[s][txs[m]] : slot[s][16 + tn[m]];
        if (d >= 4 && s == d - 2) at[s] = -1;
        if (at[s] > lastk) lastk = at[s];
        cnt += at[s] >= 0;
      }
      int best = lastk * TAU + cnt * SRC_T;
      early[m] = 0;
      key2[m] = lastk;
      for (int c = 0; c < lastk; ++c) {
        int mk = 0, na = 0, ka = 0;
        for (int s = 0; s < 6; ++s)
          if (at[s] >= 0 && at[s] <= c) { mk |= 1 << s; ++na; if (at[s] > ka) ka = at[s]; }
        if (!na) continue;
        const int fa = ka * TAU + RESTART + na * SRC_T;
        const int fb = (fa > lastk * TAU ? fa : lastk * TAU) + RESTART + (cnt - na) * SRC_T;
        if (fb < best) { best = fb; early[m] = mk; key2[m] = ka; }
      }
      key[m] = lastk * 64 + m;
      srt[m] = (d == 2 || d == 3 ? key2[m] * TAU : best) * 64 + m;
      idx[m] = m;
    }
    for (int a = 0; a < 32; ++a)
      for (int b = a + 1; b < 32; ++b)
        if (srt[idx[b]] < srt[idx[a]]) { const int tmp = idx[a]; idx[a] = idx[b]; idx[b] = tmp; }
    for (int l = 0; l < 32; ++l) {
      const int m = idx[(l + 16) & 31];
      jb[l].tile = (unsigned char)(txs[m] | (tn[m] << 4));
      jb[l].early = (unsigned char)early[m];
      jb[l].pos = (unsigned char)(key[m] / 64);
      const int fm = d < 2 ? idx[31 - ((l + 16) & 31)] : m;
      jb[l].fin = (unsigned char)((txs[fm] & 3) | (tn[fm] << 2));
      const bool sp = early[m] != 0;
      if (d >= 2 && l >= 8 && l < 16) jb[l].plan = sp ? 0x8750 : 0xf150;
      else if (d >= 2 && d < 4 && l >= 16 && l < 24) jb[l].plan = sp ? 0x8670 : 0xf160;
      else if (d >= 2) jb[l].plan = sp ? 0xf870 : 0xff10;
      else if (tn[m] < EARLY_N) jb[l].plan = sp ? 0x3870 : 0xf310;
      else jb[l].plan = sp ? 0x8703 : 0xf103;
      jb[l].pos2 = (unsigned char)key2[m];
      jb[l].turn = (unsigned char)((l + 16) & 31);
      if (d < 2) {
        jb[l].turn = (unsigned char)(tn[m] == RELAY_N + d ? txs[m] & 3 : 255);
      }
      job[l * 8 + d] = jb[l];
    }
  }
}

}

extern "C" void kernel_launch(int rank, int nranks, ncclComm_t comm, hipStream_t stream, void* const* d_in,
                              const int* in_sizes, int n_in, void* d_out, int out_size, const int* in_dtypes,
                              int out_dtype, void* const* d_out_all, void* const* d_ws_all, size_t ws_bytes) {
  if (nranks != NR || (unsigned)rank >= (unsigned)NR || n_in < 2 || ws_bytes < WS_END || out_size != 512 * 4096 ||
      in_sizes[0] != 4096 * 512 || in_sizes[1] != 512 * 4096) {
    fprintf(stderr, "kernel_launch: unexpected problem shape\n");
    return;
  }
  Params p;
  memset(&p, 0, sizeof p);
  p.x = (const char*)d_in[0];
  p.w = (const char*)d_in[1];
  p.out = (signed char*)d_out;
  for (int i = 0; i < NR; ++i) p.wsx[i] = (char*)d_ws_all[rank ^ i];
  p.rank = rank;
  make_jobs(p.job);
  hipLaunchKernelGGL(fused_gemm_rs_q8, dim3(NWG), dim3(NT), 0, stream, p);
  HIP_CHECK(hipGetLastError());
}
```
